```python
import numpy as np

kernel_name = "gpu_a2a_gemm_m4096_k4096_n4096_bf16_none"


ABSMAX_RTOL = 0.008

COMPUTE_DTYPE = "bfloat16"
EPILOGUE = "none"
IN_QUANT = None
OUT_QUANT = None
IN_DTYPES = {"x": "bfloat16", "w_mat": "bfloat16"}
WORLD_SIZE = 8

def _to_bf16(x):
    u = np.ascontiguousarray(x, dtype=np.float32).view(np.uint32)
    return ((u + 0x7FFF + ((u >> 16) & 1)) & 0xFFFF0000).view(np.float32)

def setup_inputs():

    x = _to_bf16(np.random.randn(4096, 4096).astype(np.float32))
    w_mat = _to_bf16((np.random.randn(4096, 4096) / np.sqrt(4096)).astype(np.float32))
    return {"x": x, "w_mat": w_mat}

def _epilogue(y):
    return y

def reference(x, w_mat):
    y = (x @ w_mat).astype(np.float32)
    return _epilogue(y)
```

```cpp
#include <hip/hip_runtime.h>
#include <rccl/rccl.h>
#include <cstdio>
#include <cstdint>

#define HIP_CHECK(expr) do { hipError_t _e = (expr); if (_e != hipSuccess) { \
    fprintf(stderr, "HIP error %s at %s:%d\n", hipGetErrorString(_e), __FILE__, __LINE__); \
    return; } } while (0)

namespace a2a {
#define LAS __attribute__((address_space(3)))
#define GAS __attribute__((address_space(1)))
typedef unsigned u32x4 __attribute__((ext_vector_type(4)));
typedef unsigned u32x2 __attribute__((ext_vector_type(2)));
typedef float f32x4 __attribute__((ext_vector_type(4)));
typedef __bf16 bf16x8 __attribute__((ext_vector_type(8)));
typedef __bf16 bf16x4 __attribute__((ext_vector_type(4)));
typedef __bf16 bf16x2 __attribute__((ext_vector_type(2)));
typedef float f32x2 __attribute__((ext_vector_type(2)));

constexpr int NR = 8;
constexpr int MLOC = 512, KTOT = 4096, NTOT = 4096, KC = 512;
constexpr int NWG = 256, NTHREADS = 7 * 64;
constexpr int NSTEP = 64;
#define A2A_NSLOT 16
#define A2A_RA 4
#define A2A_WINF 4
#define A2A_FSTRIDE 32
constexpr int NSLOT = A2A_NSLOT;
constexpr int RA = A2A_RA;
constexpr int WINF = A2A_WINF;
constexpr int SLOT_BYTES = 8192;
constexpr int CTL_OFF = NSLOT * SLOT_BYTES;
constexpr int LDS_BYTES = CTL_OFF + 64;
constexpr size_t WS_CNT = 0;
constexpr size_t WS_TMO = 32768;
constexpr size_t WS_FLAG = 65536;
constexpr size_t WS_STG = 1u << 20;
constexpr size_t STG_PAR = 4u << 20;
constexpr size_t WS_NEED = WS_STG + 2 * STG_PAR;
constexpr unsigned long long DEADLINE = 50000000ull;
constexpr int NPUSH = 7 * 16;

struct Params {
    const char* x; const char* w; char* out;
    char* ws[NR];
    int rank;
};

#define STAMP(i) do {} while (0)

__device__ __forceinline__ __amdgpu_buffer_rsrc_t mkrsrc(const void* ptr, unsigned bytes) {
    return __builtin_amdgcn_make_buffer_rsrc((void*)ptr, 0, bytes, 0x00020000);
}
__device__ __forceinline__ void step_src(int t, int rank, int rot, int& src, int& slice) {
    if (t < 8) { src = rank; slice = t; }
    else { const int u = t - 8; slice = u / 7; int i = u - slice * 7 + rot; if (i >= 7) i -= 7; src = (rank + 1 + i) & 7; }
}
__device__ __forceinline__ unsigned lds_ld(LAS unsigned* a) { return __hip_atomic_load(a, __ATOMIC_RELAXED, __HIP_MEMORY_SCOPE_WORKGROUP); }
__device__ __forceinline__ void lds_st(LAS unsigned* a, unsigned v) { __hip_atomic_store(a, v, __ATOMIC_RELAXED, __HIP_MEMORY_SCOPE_WORKGROUP); }
__device__ __forceinline__ unsigned pack2(float a, float b) { f32x2 v = {a, b}; return __builtin_bit_cast(unsigned, __builtin_convertvector(v, bf16x2)); }

__global__ __launch_bounds__(NTHREADS, 2) void a2a_gemm(const Params p) {
    __shared__ __attribute__((aligned(16))) char smem[LDS_BYTES];
    LAS char* lds = (LAS char*)smem;
    LAS unsigned* ctl = (LAS unsigned*)(lds + CTL_OFF);
    const int bid = blockIdx.x, lane = threadIdx.x & 63;
    const int wid = __builtin_amdgcn_readfirstlane(threadIdx.x >> 6);
    const int rank = p.rank;
    const int loc = bid >> 3, mt = loc & 3, nt = (bid & 7) * 8 + (loc >> 2);
    const int rot = (mt * 2 + (loc >> 2)) % 7;
    char* const myws = p.ws[rank];

    if (wid == 6 && lane < 8) ctl[lane] = lane == 0 ? 8u : 0u;
    if (wid == 0) STAMP(0);

    if (wid < 4) {
        const auto rx = mkrsrc(p.x, 4096 * KC * 2);
        const auto rs = mkrsrc(myws + WS_STG, 2 * STG_PAR);
        const unsigned vx0 = (unsigned)(rank * 512 + mt * 128 + wid * 32 + (lane & 15)) * 1024u + (lane >> 4) * 16;
        const unsigned vx1 = vx0 + 16 * 1024;
        const unsigned vs0 = (unsigned)(mt * 4 + wid) * 4096u + lane * 16, vs1 = vs0 + 1024;
        u32x4 ar[RA][4];
        bf16x8 wf[2][2][4];
        f32x4 acc[2][4];
#pragma unroll
        for (int i = 0; i < 2; ++i)
#pragma unroll
            for (int j = 0; j < 4; ++j) acc[i][j] = (f32x4){0.f, 0.f, 0.f, 0.f};
        auto issue_a = [&](__amdgpu_buffer_rsrc_t r_, unsigned v0, unsigned v1, unsigned so, unsigned ks, u32x4 (&r)[4]) {
            r[0] = __builtin_amdgcn_raw_buffer_load_b128(r_, v0, so, 17);
            r[1] = __builtin_amdgcn_raw_buffer_load_b128(r_, v1, so, 17);
            r[2] = __builtin_amdgcn_raw_buffer_load_b128(r_, v0, so + ks, 17);
            r[3] = __builtin_amdgcn_raw_buffer_load_b128(r_, v1, so + ks, 17);
        };
#pragma unroll
        for (int d = 0; d < RA - 1; ++d) { issue_a(rx, vx0, vx1, d * 128, 64, ar[d]); __builtin_amdgcn_sched_barrier(0); }
        asm volatile("s_waitcnt lgkmcnt(0)" ::: "memory");
        __builtin_amdgcn_s_barrier();
        asm volatile("" ::: "memory");

        const int q = (lane >> 2) & 3, g = lane >> 4, f = (q >> 1) | ((g & 1) << 1);
        const unsigned wb = (unsigned)(8 * g + q) * 128u + (lane & 3) * 8;
        unsigned a_known = 8, w_known = 0, par_off = 0;
        int ir = rot, slice = 0;
        auto wait_w = [&](int t) {
            if ((unsigned)t >= w_known) {
                unsigned v;
                while ((v = __builtin_amdgcn_readfirstlane(lds_ld(ctl + 1))) <= (unsigned)t) __builtin_amdgcn_s_sleep(1);
                w_known = v;
                asm volatile("" ::: "memory");
            }
        };
        auto read_w = [&](int t, bf16x8 (&w)[2][4]) {
            LAS char* sl = lds + (t & (NSLOT - 1)) * SLOT_BYTES + wb;
#pragma unroll
            for (int ks = 0; ks < 2; ++ks)
#pragma unroll
                for (int cb = 0; cb < 4; ++cb) {
                    LAS char* a = sl + ((cb ^ f) * 32) + ks * 4096;
                    const bf16x4 lo = __builtin_amdgcn_ds_read_tr16_b64_v4bf16((LAS bf16x4*)a);
                    const bf16x4 hi = __builtin_amdgcn_ds_read_tr16_b64_v4bf16((LAS bf16x4*)(a + 512));
                    w[ks][cb] = __builtin_shufflevector(lo, hi, 0, 1, 2, 3, 4, 5, 6, 7);
                }
        };
        wait_w(0);
        read_w(0, wf[0]);
        if (wid == 0) STAMP(1);
        constexpr int UNR = (RA % 2) ? 2 * RA : RA;
        static_assert(NSTEP % UNR == 0, "unroll");
        for (int t0 = 0; t0 < NSTEP; t0 += UNR) {
#pragma unroll
            for (int d = 0; d < UNR; ++d) {
                const int t = t0 + d, ta = t + RA - 1;
                u32x4 (&rn)[4] = ar[(d + RA - 1) % RA];
                if ((unsigned)ta >= a_known) {
                    unsigned v;
                    while ((v = __builtin_amdgcn_readfirstlane(lds_ld(ctl + 0))) <= (unsigned)ta) __builtin_amdgcn_s_sleep(1);
                    a_known = v;
                    asm volatile("" ::: "memory");
                    par_off = __builtin_amdgcn_readfirstlane(lds_ld(ctl + 2));
                }
                {
                    const bool loc_ = ta < 8, tail = ta >= NSTEP;
                    const unsigned so = loc_ ? ta * 128u : tail ? 0u : par_off + ((rank + 1 + ir) & 7) * (512u * 1024u) + slice * 65536u;
                    issue_a(loc_ ? rx : rs, loc_ ? vx0 : tail ? 0x7f000000u : vs0, loc_ ? vx1 : tail ? 0x7f000000u : vs1, so, loc_ ? 64u : 2048u, rn);
                    if (!loc_) { ir = ir == 6 ? 0 : ir + 1; if (ir == rot) ++slice; }
                }
                const int tw = t + 1 < NSTEP ? t + 1 : t;
                wait_w(tw);
                __builtin_amdgcn_sched_barrier(0);
                read_w(tw, wf[(d + 1) & 1]);
#pragma unroll
                for (int ks = 0; ks < 2; ++ks)
#pragma unroll
                    for (int rb = 0; rb < 2; ++rb)
#pragma unroll
                        for (int cb = 0; cb < 4; ++cb)
                            acc[rb][cb] = __builtin_amdgcn_mfma_f32_16x16x32_bf16(wf[d & 1][ks][cb], __builtin_bit_cast(bf16x8, ar[d % RA][ks * 2 + rb]), acc[rb][cb], 0, 0, 0);
#pragma unroll
                for (int i = 0; i < 16; ++i) {
                    __builtin_amdgcn_sched_group_barrier(0x8, 1, 0);
                    __builtin_amdgcn_sched_group_barrier(0x100, 1, 0);
                }
                __builtin_amdgcn_sched_barrier(0);
                if (lane == 0) lds_st(ctl + 4 + wid, (unsigned)t + 1);
            }
        }
        const bool bad = __builtin_amdgcn_readfirstlane(lds_ld(ctl + 3)) != 0;
        const auto ro = mkrsrc(p.out, MLOC * NTOT * 2);
        const unsigned vo = (unsigned)(mt * 128 + wid * 32 + (lane & 15)) * 8192u + nt * 128 + (g & 1) * 32 + (g >> 1) * 16;
#pragma unroll
        for (int rb = 0; rb < 2; ++rb)
#pragma unroll
            for (int j = 0; j < 2; ++j) {
                const f32x4 a = acc[rb][2 * j], b = acc[rb][2 * j + 1];
                const auto s0 = __builtin_amdgcn_permlane16_swap(pack2(a[0], a[1]), pack2(b[0], b[1]), false, false);
                const auto s1 = __builtin_amdgcn_permlane16_swap(pack2(a[2], a[3]), pack2(b[2], b[3]), false, false);
                u32x4 o = {s0[0], s1[0], s0[1], s1[1]};
                if (bad) o = (u32x4){0x7fc07fc0u, 0x7fc07fc0u, 0x7fc07fc0u, 0x7fc07fc0u};
                __builtin_amdgcn_raw_buffer_store_b128(o, ro, vo + j * 64, rb * 16 * 8192, 0);
            }
    } else if (wid == 4) {
        const auto rw = mkrsrc(p.w, KTOT * NTOT * 2);
        const unsigned vb = (unsigned)(lane >> 3) * 8192u + nt * 128;
        const unsigned ve = vb + (((lane & 7) ^ (((lane >> 4) & 1) * 2)) * 16);
        const unsigned vod = vb + (((lane & 7) ^ ((((lane >> 4) & 1) | 2) * 2)) * 16);
        auto issue = [&](int t) {
            int src, slice; step_src(t, rank, rot, src, slice);
            const unsigned so = (unsigned)(src * 512 + slice * 64) * 8192u;
            LAS char* sl = lds + (t & (NSLOT - 1)) * SLOT_BYTES;
#pragma unroll
            for (int pc = 0; pc < 8; ++pc)
                __builtin_amdgcn_raw_ptr_buffer_load_lds(rw, (LAS void*)(sl + pc * 1024), 16, (pc & 1) ? vod : ve, so + pc * 65536u, 0, 0);
        };
        for (int t = 0; t < WINF; ++t) issue(t);
        asm volatile("s_waitcnt lgkmcnt(0)" ::: "memory");
        __builtin_amdgcn_s_barrier();
        asm volatile("" ::: "memory");
        const unsigned a_full = (unsigned)(size_t)(ctl + 1), a_free = (unsigned)(size_t)(ctl + 4);
        auto publish = [&](unsigned v) {
            asm volatile("s_mov_b64 exec, 1\n\tds_write_b32 %0, %1\n\ts_mov_b64 exec, -1" :: "v"(a_full), "v"(v) : "memory");
        };
        unsigned free_known = NSLOT, pub = 0;
        for (int t = WINF; t < NSTEP; ++t) {
            if ((unsigned)t >= free_known) {
                while (true) {
                    u32x4 v;
                    asm volatile("ds_read_b128 %0, %1\n\ts_waitcnt lgkmcnt(0)" : "=&v"(v) : "v"(a_free) : "memory");
                    const unsigned m = __builtin_amdgcn_readfirstlane(min(min(v[0], v[1]), min(v[2], v[3])));
                    if (m + NSLOT > (unsigned)t) { free_known = m + NSLOT; break; }
                    if (pub < (unsigned)t) {
                        asm volatile("s_waitcnt vmcnt(0)" ::: "memory");
                        pub = t; publish(pub);
                    } else __builtin_amdgcn_s_sleep(2);
                }
            }
            issue(t);
            asm volatile("s_waitcnt vmcnt(%0)" :: "i"(8 * WINF) : "memory");
            if (pub < (unsigned)(t + 1 - WINF)) { pub = t + 1 - WINF; publish(pub); }
        }
        asm volatile("s_waitcnt vmcnt(0)" ::: "memory");
        publish(NSTEP);
        STAMP(3);
    } else if (wid == 5) {
        if (bid < NPUSH) {
            const int di = bid % 7, rg = bid / 7, dest = (rank + 1 + di) & 7;
            GAS unsigned* cnt = (GAS unsigned*)(myws + WS_CNT + bid * 128);
            const unsigned c0 = __hip_atomic_load(cnt, __ATOMIC_RELAXED, __HIP_MEMORY_SCOPE_AGENT);
            const auto rx = mkrsrc(p.x, 4096 * KC * 2);
            const unsigned vx0 = (unsigned)(dest * 512 + rg * 32 + (lane & 15)) * 1024u + (lane >> 4) * 16;
            const unsigned vx1 = vx0 + 16 * 1024;
            u32x4 v[8][4];
#pragma unroll
            for (int s = 0; s < 8; ++s) {
                v[s][0] = __builtin_amdgcn_raw_buffer_load_b128(rx, vx0 + s * 128, 0, 0);
                v[s][2] = __builtin_amdgcn_raw_buffer_load_b128(rx, vx0 + s * 128 + 64, 0, 0);
                v[s][1] = __builtin_amdgcn_raw_buffer_load_b128(rx, vx1 + s * 128, 0, 0);
                v[s][3] = __builtin_amdgcn_raw_buffer_load_b128(rx, vx1 + s * 128 + 64, 0, 0);
            }
            __builtin_amdgcn_s_barrier();
            __builtin_amdgcn_s_waitcnt(0x0F70);
            __builtin_amdgcn_sched_barrier(0);
            STAMP(4);
            const unsigned call = __builtin_amdgcn_readfirstlane(c0) + 1;
            char* const dws = p.ws[dest];
            char* dst = dws + WS_STG + (call & 1) * STG_PAR + rank * (512 * 1024) + rg * 4096 + lane * 16;
            GAS unsigned* flag = (GAS unsigned*)(dws + WS_FLAG + (size_t)(((rg >> 2) * 8 + rank) * 4 + (rg & 3)) * A2A_FSTRIDE);
            auto push = [&](int s) {
                char* d = dst + s * 65536;
                asm volatile("global_store_dwordx4 %0, %1, off sc0 sc1\n\t"
                             "global_store_dwordx4 %0, %2, off offset:1024 sc0 sc1\n\t"
                             "global_store_dwordx4 %0, %3, off offset:2048 sc0 sc1\n\t"
                             "global_store_dwordx4 %0, %4, off offset:3072 sc0 sc1\n\ts_nop 1"
                             :: "v"(d), "v"(v[s][0]), "v"(v[s][1]), "v"(v[s][2]), "v"(v[s][3]) : "memory");
            };
            auto publish = [&](int s) {
                const unsigned val = call * 16 + s + 1;
                asm volatile("s_mov_b64 exec, 1\n\tglobal_store_dword %0, %1, off sc0 sc1\n\ts_mov_b64 exec, -1" :: "v"(flag), "v"(val) : "memory");
            };
            push(0); push(1);
            asm volatile("s_waitcnt vmcnt(4)" ::: "memory");
#pragma unroll
            for (int s = 0; s < 6; ++s) {
                publish(s); push(s + 2);
                asm volatile("s_waitcnt vmcnt(5)" ::: "memory");
            }
            publish(6);
            asm volatile("s_waitcnt vmcnt(1)" ::: "memory");
            publish(7);
            if (lane == 0) __hip_atomic_store(cnt, call, __ATOMIC_RELAXED, __HIP_MEMORY_SCOPE_AGENT);
            STAMP(5);
        } else {
            __builtin_amdgcn_s_barrier();
        }
    } else {
        GAS unsigned* cnt = (GAS unsigned*)(myws + WS_CNT + bid * 128 + 64);
        GAS unsigned* tmo = (GAS unsigned*)(myws + WS_TMO);
        const unsigned c0 = __hip_atomic_load(cnt, __ATOMIC_RELAXED, __HIP_MEMORY_SCOPE_AGENT);
        asm volatile("s_waitcnt lgkmcnt(0)" ::: "memory");
        __builtin_amdgcn_s_barrier();
        asm volatile("" ::: "memory");
        const unsigned call = __builtin_amdgcn_readfirstlane(c0) + 1;
        if (lane == 0) { __hip_atomic_store(cnt, call, __ATOMIC_RELAXED, __HIP_MEMORY_SCOPE_AGENT); lds_st(ctl + 2, (call & 1) * (unsigned)STG_PAR); }
        int i = (lane >> 2) + rot; if (i >= 7) i -= 7;
        const int src = (rank + 1 + i) & 7;
        GAS unsigned* flag = lane < 28 ? (GAS unsigned*)(myws + WS_FLAG + (size_t)((mt * 8 + src) * 4 + (lane & 3)) * A2A_FSTRIDE) : tmo;
        const unsigned long long full = (1ull << 28) - 1, t0 = __builtin_amdgcn_s_memrealtime();
        int j = 0; unsigned ready = 8; bool bad = false;
        while (true) {
            unsigned v = 0;
            if (lane <= 28) v = __hip_atomic_load(flag, __ATOMIC_RELAXED, __HIP_MEMORY_SCOPE_SYSTEM);
            if (__builtin_amdgcn_readlane(v, 28) == call) { bad = true; break; }
            const int prog = (int)(v - call * 16);
            unsigned long long m;
            while (j < 8 && (m = __ballot(prog > j) & full) == full) {
                ++j;
            }
            if (j == 8) break;
            const unsigned r = 8 + 7 * j + (__builtin_ctzll(~m) >> 2);
            if (r != ready) { ready = r; if (lane == 0) lds_st(ctl + 0, r); }
            if (__builtin_amdgcn_s_memrealtime() - t0 > DEADLINE) {
                if (lane == 0) __hip_atomic_store(tmo, call, __ATOMIC_RELAXED, __HIP_MEMORY_SCOPE_AGENT);
                bad = true; break;
            }
            __builtin_amdgcn_s_sleep(2);
        }
        if (lane == 0) { if (bad) lds_st(ctl + 3, 1u); lds_st(ctl + 0, 1000u); }
    }
}
}

extern "C" void kernel_launch(int rank, int nranks, ncclComm_t comm,
                              hipStream_t stream,
                              void* const* d_in, const int* in_sizes, int n_in,
                              void* d_out, int out_size,
                              const int* in_dtypes, int out_dtype,
                              void* const* d_out_all,
                              void* const* d_ws_all, size_t ws_bytes) {
    using namespace a2a;
    if (nranks != NR || (unsigned)rank >= (unsigned)NR || n_in < 2 || ws_bytes < WS_NEED ||
        in_sizes[0] != 4096 * KC || in_sizes[1] != KTOT * NTOT || out_size != MLOC * NTOT) {
        fprintf(stderr, "a2a_gemm: unexpected problem shape\n");
        return;
    }
    Params p;
    p.x = (const char*)d_in[0]; p.w = (const char*)d_in[1]; p.out = (char*)d_out;
    for (int i = 0; i < NR; ++i) p.ws[i] = (char*)d_ws_all[i];
    p.rank = rank;
    hipLaunchKernelGGL(a2a_gemm, dim3(NWG), dim3(NTHREADS), 0, stream, p);
    HIP_CHECK(hipGetLastError());
}
```

```cpp
#include <hip/hip_runtime.h>
#include <rccl/rccl.h>
#include <cstdio>
#include <cstdint>
#include <type_traits>
#include <utility>

#define HIP_CHECK(expr) do { hipError_t _e = (expr); if (_e != hipSuccess) { \
    fprintf(stderr, "HIP error %s at %s:%d\n", hipGetErrorString(_e), __FILE__, __LINE__); \
    return; } } while (0)

namespace a2a {
#define LAS __attribute__((address_space(3)))
#define GAS __attribute__((address_space(1)))
typedef unsigned u32x4 __attribute__((ext_vector_type(4)));
typedef float f32x4 __attribute__((ext_vector_type(4)));
typedef float f32x16 __attribute__((ext_vector_type(16)));
typedef __bf16 bf16x8 __attribute__((ext_vector_type(8)));
typedef __bf16 bf16x4 __attribute__((ext_vector_type(4)));
typedef __bf16 bf16x2 __attribute__((ext_vector_type(2)));
typedef float f32x2 __attribute__((ext_vector_type(2)));

constexpr int NR = 8;
constexpr int MLOC = 512, KTOT = 4096, NTOT = 4096, KC = 512;
constexpr int NWG = 256, NTHREADS = 10 * 64;
#define A2A_N2 8
#define A2A_LSLEEP 0
#define A2A_PSLEEP 2
#define A2A_STAG 4
#define A2A_T0 40
#define A2A_TAU 34
#define A2A_AHEAD 4
#define A2A_XBF16 0
#define A2A_XAUX 16
#define A2A_X1 4
#define A2A_X2 2
#define A2A_FSTRIDE 64
constexpr int N2 = A2A_N2;
constexpr int N1 = (64 - N2) / 2;
constexpr int NT = N1 + N2;
static_assert(N2 % 2 == 0 && N1 >= 4, "N2");
constexpr int NSA = 4, NSW = 4, SLOT = 16384, W_OFF = NSA * SLOT;
static_assert(N1 % 4 == 0 && N2 >= 2, "the wide loop is unrolled by the ring depth");
constexpr int SIG_OFF = W_OFF + NSW * SLOT;
constexpr int SIG_FULL = 0;
constexpr int SIG_FREE = 4;
constexpr int SIG_RDY = 12;
constexpr int SIG_X = 13;
constexpr int NSIG = 14;
constexpr int MISC_OFF = SIG_OFF + NSIG * 512;
constexpr int LDS_BYTES = MISC_OFF + 64;
static_assert(LDS_BYTES <= 163840, "LDS");
constexpr size_t WS_CNT = 0;
constexpr size_t WS_TMO = 32768;
constexpr size_t WS_FLAG = 65536;
constexpr size_t WS_XFLAG = 131072;
constexpr size_t WS_STG = 1u << 20;
constexpr size_t STG_PAR = 4u << 20;
constexpr size_t WS_XBUF = WS_STG + 2 * STG_PAR;
constexpr size_t XBUF_BYTES = 8u << 20;
constexpr size_t WS_NEED = WS_XBUF + XBUF_BYTES;
constexpr unsigned long long DEADLINE = 50000000ull;
constexpr int NPUSH = 7 * 32;

struct Params {
    const char* x; const char* w; char* out;
    char* ws[NR];
    int rank;
};

#define STAMP(i) do {} while (0)

__device__ __forceinline__ __amdgpu_buffer_rsrc_t mkrsrc(const void* ptr, unsigned bytes) {
    return __builtin_amdgcn_make_buffer_rsrc((void*)ptr, 0, bytes, 0x00020000);
}
__device__ __forceinline__ unsigned lds_ld(LAS unsigned* a) { return __hip_atomic_load(a, __ATOMIC_RELAXED, __HIP_MEMORY_SCOPE_WORKGROUP); }
__device__ __forceinline__ void lds_st(LAS unsigned* a, unsigned v) { __hip_atomic_store(a, v, __ATOMIC_RELAXED, __HIP_MEMORY_SCOPE_WORKGROUP); }
__device__ __forceinline__ int aswz(int i) { return ((i << 1) ^ ((i >> 1) * 3)) & 3; }
__device__ __forceinline__ unsigned pack2(float a, float b) { f32x2 v = {a, b}; return __builtin_bit_cast(unsigned, __builtin_convertvector(v, bf16x2)); }
__device__ __forceinline__ unsigned asm_lds_ld(unsigned addr) {
    unsigned v;
    asm volatile("ds_read_b32 %0, %1\n\ts_waitcnt lgkmcnt(0)" : "=&v"(v) : "v"(addr) : "memory");
    return v;
}
__device__ __forceinline__ void signal(u32x4 r, unsigned lds_addr) {
    unsigned keep;
    asm volatile("s_mov_b32 %0, m0\n\ts_mov_b32 m0, %1\n\ts_nop 0\n\tbuffer_load_dword %2, %3, 0 offen lds\n\ts_mov_b32 m0, %0"
                 : "=&s"(keep) : "s"(lds_addr), "v"(0x7ffffff0u), "s"(r) : "memory");
}
template <int N, class F, int... I> __device__ __forceinline__ void static_for_impl(F f, std::integer_sequence<int, I...>) { (f(std::integral_constant<int, I>{}), ...); }
template <int N, class F> __device__ __forceinline__ void static_for(F f) { static_for_impl<N>(f, std::make_integer_sequence<int, N>{}); }
__device__ __forceinline__ int step_table(int lane, int rank, int mt, int h) {
    const int j = lane >> 3, pi = 2 * ((((lane >> 1) & 3) - mt) & 3) + (lane & 1), gi = 8 + 7 * j + pi;
    const bool wide = pi < 7 && gi < 2 * N1 && (gi & 1) == h, narrow = pi < 7 && gi >= 2 * N1;
    const unsigned long long below = (1ull << lane) - 1;
    const int pos = wide ? 4 + __builtin_popcountll(__builtin_amdgcn_ballot_w64(wide) & below)
                         : N1 + __builtin_popcountll(__builtin_amdgcn_ballot_w64(narrow) & below);
    const int v = __builtin_amdgcn_ds_permute((wide || narrow ? pos : 63) * 4, (((rank + 1 + pi) & 7) << 3) | j);
    return lane < 4 ? (rank << 3) | (2 * lane + h) : v;
}
#define START_BARRIER() do { asm volatile("s_waitcnt lgkmcnt(0)" ::: "memory"); __builtin_amdgcn_s_barrier(); asm volatile("" ::: "memory"); } while (0)
#define SB() __builtin_amdgcn_sched_barrier(0)

__global__ __launch_bounds__(NTHREADS, 3) void a2a_gemm(const Params p) {
    __shared__ __attribute__((aligned(16))) char smem[LDS_BYTES];
    LAS char* lds = (LAS char*)smem;
    const unsigned lds0 = (unsigned)(size_t)lds, sig0 = lds0 + SIG_OFF;
    LAS unsigned* misc = (LAS unsigned*)(lds + MISC_OFF);
    auto sig_at = [&](int arr, int e) { return sig0 + arr * 512 + (63 - e) * 4; };
    const int bid = blockIdx.x, lane = threadIdx.x & 63;
    const int wid = __builtin_amdgcn_readfirstlane(threadIdx.x >> 6);
    const int rank = p.rank;
    const int h = bid & 1, loc = bid >> 3, mt = loc & 3, nt = ((bid >> 1) & 3) * 8 + (loc >> 2);
    char* const myws = p.ws[rank];
    const u32x4 rz = {(unsigned)(size_t)p.x, (unsigned)((size_t)p.x >> 32) & 0xffffu, 16u, 0x00020000u};

    if (wid == 0) STAMP(0);
    if (wid < 4) {
        for (int i = threadIdx.x; i < (NSIG * 512 + 64) / 16; i += 256) *(LAS u32x4*)(lds + SIG_OFF + i * 16) = (u32x4){~0u, ~0u, ~0u, ~0u};
    }
    if (wid < 4) {
        START_BARRIER();
        __builtin_amdgcn_s_setprio(3);
        const int wm = wid >> 1, wn = wid & 1, q = (lane >> 2) & 3, g = lane >> 4, r16 = lane & 15, f = (q >> 1) | ((g & 1) << 1);
        const auto rxb = mkrsrc(myws + WS_XBUF, XBUF_BYTES);
        const unsigned a0 = lds0 + (wm * 64 + r16) * 64 + ((g ^ aswz(r16 >> 2)) * 16);
        const unsigned w0 = lds0 + W_OFF + (8 * g + q) * 128 + (((wn * 2) ^ f) * 32) + (lane & 3) * 8, w1 = w0 ^ 32;
        unsigned fbase = sig_at(SIG_FULL + (lane & 1) * 2, 4), free_a = sig_at(SIG_FREE + wid, 0);
        auto wait_zero = [&](unsigned a) {
            while (__builtin_amdgcn_ballot_w64(*(volatile LAS unsigned*)(size_t)a != 0) != 0) __builtin_amdgcn_s_sleep(1);
            asm volatile("" ::: "memory");
        };
        bf16x8 af[2][4], wf[2][4];
        bf16x4 wl;
        f32x4 acc[4][4];
        unsigned fv;
#pragma unroll
        for (int i = 0; i < 4; ++i) { af[1][i] = (bf16x8)(__bf16)0.f; wf[1][i] = (bf16x8)(__bf16)0.f; }
#pragma unroll
        for (int i = 0; i < 16; ++i) acc[i >> 2][i & 3] = (f32x4){0.f, 0.f, 0.f, 0.f};
#define FULL_AT(U) (fbase + ((U) & 1) * 512 + (4 - ((U) >> 1)) * 4)
#define FV() do { if constexpr (more) { fv = *(volatile LAS unsigned*)(size_t)FULL_AT(U + 1); SB(); } } while (0)
#define LGKM0() do { __builtin_amdgcn_s_waitcnt(0xC07F); SB(); } while (0)
#define HAVE() do { if constexpr (more) { miss = __builtin_amdgcn_ballot_w64(fv != 0); asm volatile("" : "+s"(miss)); SB(); } } while (0)
#define FREE() do { if constexpr (sig) { \
            if constexpr (H) asm volatile("s_mov_b32 m0, %0\n\ts_sub_u32 %0, %0, 2052\n\tbuffer_load_dword %1, %2, 0 offen lds" : "+s"(free_a) : "v"(0x7ffffff0u), "s"(rz) : "memory", "scc"); \
            else asm volatile("s_mov_b32 m0, %0\n\ts_add_u32 %0, %0, 2048\n\tbuffer_load_dword %1, %2, 0 offen lds" : "+s"(free_a) : "v"(0x7ffffff0u), "s"(rz) : "memory", "scc"); \
            SB(); } } while (0)
#define RDW(cb, t2) do { const bf16x4 w_ = __builtin_amdgcn_ds_read_tr16_b64_v4bf16((LAS bf16x4*)(size_t)(((cb & 1) ? w1 : w0) + S * SLOT + H * 4096 + (cb >> 1) * 8192 + t2 * 512)); \
            if (t2 == 0) wl = w_; else wf[H][cb] = __builtin_shufflevector(wl, w_, 0, 1, 2, 3, 4, 5, 6, 7); SB(); } while (0)
#define RDA(rb) do { af[H][rb] = *(LAS bf16x8*)(size_t)(a0 + S * SLOT + H * 8192 + rb * 1024); SB(); } while (0)
#define MM(rb, cb) do { acc[rb][cb] = __builtin_amdgcn_mfma_f32_16x16x32_bf16(wf[H ^ 1][cb], af[H ^ 1][rb], acc[rb][cb], 0, 0, 0); SB(); } while (0)
#define MMEND() do { asm volatile("s_nop 7\n\ts_nop 1" ::: "memory"); SB(); } while (0)
        auto half = [&](auto uc, auto pw, auto wide, auto sig, auto more) {
            constexpr int U = decltype(uc)::value, S = U >> 1, H = U & 1;
            unsigned long long miss = 0;
            if constexpr (wide) {
                MM(0, 0); RDW(0, 0); MM(0, 1); RDW(0, 1); RDA(0); MM(0, 2); RDW(1, 0); MM(0, 3); RDW(1, 1);
                MM(1, 0); RDW(2, 0); RDW(2, 1); MM(1, 1); RDW(3, 0); MM(1, 2); RDW(3, 1); MM(1, 3); RDA(1);
                MM(2, 0); RDA(2); MM(2, 1); RDA(3); FV(); MM(2, 2); MM(2, 3);
                MM(3, 0); MM(3, 1); LGKM0(); HAVE(); MM(3, 2); FREE(); MM(3, 3);
            } else if constexpr (pw) {
                MM(0, 0); RDW(0, 0); MM(0, 1); RDW(0, 1); MM(0, 2); RDA(0); MM(0, 3); RDW(1, 0);
                MM(1, 0); RDW(1, 1); MM(1, 1); RDA(1); MM(1, 2); RDA(2); MM(1, 3); RDA(3);
                MM(2, 0); MM(2, 1); FV(); MM(2, 2); MM(2, 3);
                MM(3, 0); MM(3, 1); LGKM0(); HAVE(); MM(3, 2); FREE(); MM(3, 3);
            } else {
                MM(0, 0); RDW(0, 0); RDW(0, 1); MM(0, 1); RDA(0); RDA(1); MM(1, 0); RDW(1, 0); RDW(1, 1); MM(1, 1); RDA(2); RDA(3); FV();
                MM(2, 0); MM(2, 1); MM(3, 0); LGKM0(); HAVE(); FREE(); MM(3, 1);
            }
            if constexpr (more) {
                if (__builtin_expect(miss != 0, 0)) wait_zero(FULL_AT(U + 1));
                if constexpr (U == 7) { fbase -= 16; asm volatile("" : "+v"(fbase)); }
            }
        };
        using T_ = std::integral_constant<bool, true>; using F_ = std::integral_constant<bool, false>;
        __builtin_amdgcn_s_waitcnt(0x0F70);
        wait_zero(FULL_AT(0));
        if (wid == 0) STAMP(1);
#pragma unroll 1
        for (int i = 0; i < N1 / 4; ++i) {
            static_for<8>([&](auto uc) { half(uc, T_{}, T_{}, T_{}, T_{}); });
        }
        constexpr int NX = A2A_XBF16 ? 4 : 8, X1 = A2A_X1, X2 = N2 - A2A_X2;
        static_assert(1 <= X1 && X1 <= X2 && X2 <= N2, "exchange points");
        u32x4 xp[NX];
        static_for<N2>([&](auto ic) {
            constexpr int i = decltype(ic)::value;
            using Sig = std::integral_constant<bool, (i + 4 < N2)>;
            half(std::integral_constant<int, (i & 3) * 2>{}, std::integral_constant<bool, i == 0>{}, F_{}, Sig{}, T_{});
            if constexpr (i == 0) {
                MMEND();
                if (wid == 0) STAMP(6);
#pragma unroll
                for (int k = 0; k < NX; ++k) {
                    const u32x4 v = __builtin_bit_cast(u32x4, acc[k >> 1][2 + (k & 1)]);
                    __builtin_amdgcn_raw_buffer_store_b128(v, rxb, ((bid ^ 1) * 4 + wid) * (NX * 1024) + lane * 16 + k * 1024, 0, A2A_XAUX);
                }
                SB();
            }
            half(std::integral_constant<int, (i & 3) * 2 + 1>{}, F_{}, F_{}, Sig{}, std::integral_constant<bool, (i + 1 < N2)>{});
            if constexpr (i + 1 == X1) {
                constexpr int nsig = 2 * (X1 < N2 - 4 ? X1 : N2 - 4) - 1;
                asm volatile("s_waitcnt vmcnt(%0)" :: "i"(nsig > 0 ? nsig : 0) : "memory");
                const unsigned call = __builtin_amdgcn_readfirstlane(lds_ld(misc)) + 1;
                if (lane == 0) __hip_atomic_store((GAS unsigned*)(myws + WS_XFLAG + (bid ^ 1) * 128 + wid * 4), call, __ATOMIC_RELAXED, __HIP_MEMORY_SCOPE_AGENT);
                if (wid == 0) STAMP(14);
                SB();
            }
            if constexpr (i + 1 == X2) {
                wait_zero(sig_at(SIG_X, 0));
                if (wid == 0) STAMP(7);
#pragma unroll
                for (int k = 0; k < NX; ++k) xp[k] = __builtin_amdgcn_raw_buffer_load_b128(rxb, lane * 16 + k * 1024, (bid * 4 + wid) * (NX * 1024), 16);
                SB();
            }
        });
#pragma unroll
        for (int rb = 0; rb < 4; ++rb) { constexpr int H = 0; MM(rb, 0); MM(rb, 1); }
        MMEND();
        const bool bad = __builtin_amdgcn_readfirstlane(lds_ld(misc + 1)) == 0;
        const auto ro = mkrsrc(p.out, MLOC * NTOT * 2);
        const unsigned vo = (unsigned)(mt * 128 + wm * 64 + r16) * 8192u + nt * 256 + h * 128 + wn * 64 + (g & 1) * 32 + (g >> 1) * 16;
#pragma unroll
        for (int rb = 0; rb < 4; ++rb) {
            const f32x4 a = acc[rb][0] + __builtin_bit_cast(f32x4, xp[2 * rb]), b = acc[rb][1] + __builtin_bit_cast(f32x4, xp[2 * rb + 1]);
            const auto s0 = __builtin_amdgcn_permlane16_swap(pack2(a[0], a[1]), pack2(b[0], b[1]), false, false);
            const auto s1 = __builtin_amdgcn_permlane16_swap(pack2(a[2], a[3]), pack2(b[2], b[3]), false, false);
            u32x4 o = {s0[0], s1[0], s0[1], s1[1]};
            if (bad) o = (u32x4){0x7fc07fc0u, 0x7fc07fc0u, 0x7fc07fc0u, 0x7fc07fc0u};
            __builtin_amdgcn_raw_buffer_store_b128(o, ro, vo + rb * 16 * 8192, 0, 0);
        }
    } else if (wid < 8) {
        const int j = wid & 1;
        unsigned full_a = sig_at(SIG_FULL + wid - 4, 0);
        auto wait_free = [&](int e) {
            const unsigned a = sig_at(SIG_FREE + j * 4 + (lane & 3), e);
            while (__builtin_amdgcn_ballot_w64(asm_lds_ld(a) != 0) != 0) __builtin_amdgcn_s_sleep(A2A_LSLEEP);
        };
        const int tab = step_table(lane, rank, mt, h);
#define LD4(rsrc, ldsp, v0, v1, so, stride, aux) do { \
            __builtin_amdgcn_raw_ptr_buffer_load_lds(rsrc, (LAS void*)(ldsp), 16, v0, (so), 0, aux); \
            __builtin_amdgcn_raw_ptr_buffer_load_lds(rsrc, (LAS void*)(ldsp), 16, v1, (so) + ((stride) - 1024), 1024, aux); \
            __builtin_amdgcn_raw_ptr_buffer_load_lds(rsrc, (LAS void*)(ldsp), 16, v0, (so) + 2 * ((stride) - 1024), 2048, aux); \
            __builtin_amdgcn_raw_ptr_buffer_load_lds(rsrc, (LAS void*)(ldsp), 16, v1, (so) + 3 * ((stride) - 1024), 3072, aux); } while (0)
        if (wid < 6) {
            const unsigned ch = ((lane & 3) ^ aswz(lane >> 4)) * 16;
            const auto rx = mkrsrc(p.x, 4096 * KC * 2);
            const unsigned vx = (unsigned)(rank * 512 + mt * 128 + (lane >> 2)) * 1024u + j * 64 + ch;
            START_BARRIER();
            if (j) __builtin_amdgcn_s_sleep(A2A_STAG);
#pragma unroll 1
            for (int t = 0; t < 4; ++t) {
                LAS char* const sl = lds + t * SLOT + j * 8192;
                LD4(rx, sl, vx, vx, (2 * t + h) * 128, 16384, 0);
                LD4(rx, sl + 4096, vx, vx, (2 * t + h) * 128 + 65536, 16384, 0);
                signal(rz, full_a); full_a -= 4;
                __builtin_amdgcn_s_sleep(A2A_STAG);
            }
            const auto rs = mkrsrc(myws + WS_STG, 2 * STG_PAR);
            const unsigned vs = (unsigned)j * 32768u + mt * 8192 + (lane >> 2) * 64 + ch;
            unsigned par = 0;
#pragma unroll 1
            for (int t = 4; t < NT; ++t) {
                LAS char* const sl = lds + (t & 3) * SLOT + j * 8192;
                while (__builtin_amdgcn_readfirstlane(asm_lds_ld(sig_at(SIG_RDY, t))) != 0) __builtin_amdgcn_s_sleep(1);
                if (t == 4) par = ((__builtin_amdgcn_readfirstlane(asm_lds_ld(lds0 + MISC_OFF)) + 1) & 1) * (unsigned)STG_PAR;
                unsigned so = par + __builtin_amdgcn_readlane(tab, t) * 65536u;
                asm volatile("" : "+s"(so));
                wait_free(t - 4);
                LD4(rs, sl, vs, vs, so, 1024, 17);
                LD4(rs, sl + 4096, vs, vs, so + 4096, 1024, 17);
                signal(rz, full_a); full_a -= 4;
            }
        } else {
            const auto rw = mkrsrc(p.w, KTOT * NTOT * 2);
            const unsigned vwb = (unsigned)(j * 32 + (lane >> 3)) * 8192u + nt * 256;
            const unsigned vwe = vwb + (((lane & 7) ^ (((lane >> 4) & 1) * 2)) * 16), vwo = vwb + (((lane & 7) ^ ((((lane >> 4) & 1) | 2) * 2)) * 16);
            START_BARRIER();
            if (j) __builtin_amdgcn_s_sleep(A2A_STAG);
#pragma unroll 1
            for (int t = 0; t < NT; ++t) {
                LAS char* const sl = lds + W_OFF + (t & 3) * SLOT + j * 4096;
                unsigned so = __builtin_amdgcn_readlane(tab, t) * (64u * 8192u) + h * 128;
                asm volatile("" : "+s"(so));
                if (t >= 4) wait_free(t - 4);
                LD4(rw, sl, vwe, vwo, so, 65536, 0);
                if (t < N1) LD4(rw, sl + 8192, vwe, vwo, so ^ 128, 65536, 0);
                signal(rz, full_a); full_a -= 4;
                if (t < 3) __builtin_amdgcn_s_sleep(A2A_STAG);
            }
        }
        if (wid == 4) STAMP(3);
    } else if (wid == 8) {
        if (bid < NPUSH) {
            __builtin_amdgcn_s_setprio(3);
            const unsigned tstart = (unsigned)__builtin_amdgcn_s_memrealtime();
            const int slot = bid / 7, di = bid - slot * 7, dest = (rank + 1 + di) & 7, slice = slot >> 2, tm = (slot - ((6 - di) >> 1)) & 3;
            GAS unsigned* cnt = (GAS unsigned*)(myws + WS_CNT + bid * 128);
            const auto rx = mkrsrc(p.x, 4096 * KC * 2);
            const unsigned vx = (unsigned)(dest * 512 + tm * 128 + ((lane >> 2) & 7)) * 1024u + slice * 128 + (lane >> 5) * 64 + (lane & 3) * 16;
            char* const dws = p.ws[dest];
            if (slot >= 2) {
                __builtin_amdgcn_s_barrier();
                while ((int)((unsigned)__builtin_amdgcn_s_memrealtime() - tstart) < (slot - 2) * A2A_TAU) __builtin_amdgcn_s_sleep(2);
            }
            const unsigned c0 = __hip_atomic_load(cnt, __ATOMIC_RELAXED, __HIP_MEMORY_SCOPE_AGENT);
            u32x4 v[16];
#pragma unroll
            for (int i = 0; i < 16; ++i) v[i] = __builtin_amdgcn_raw_buffer_load_b128(rx, vx, i * 8192, 0);
            if (slot < 2) __builtin_amdgcn_s_barrier();
            __builtin_amdgcn_s_waitcnt(0x0F70);
            SB();
            const unsigned call = __builtin_amdgcn_readfirstlane(c0) + 1;
            char* const dst = dws + WS_STG + (call & 1) * STG_PAR + (rank * 8 + slice) * 65536 + (lane >> 5) * 32768 + tm * 8192 + (lane & 31) * 16 + 4096;
            while ((int)((unsigned)__builtin_amdgcn_s_memrealtime() - tstart) < A2A_T0 + slot * A2A_TAU) __builtin_amdgcn_s_sleep(1);
            static_for<16>([&](auto ic) {
                constexpr int i = decltype(ic)::value;
                char* const d = dst; const u32x4 vi = v[i];
                asm volatile("global_store_dwordx4 %0, %1, off offset:%2 sc0 sc1\n\ts_nop 1" :: "v"(d), "v"(vi), "i"(i * 512 - 4096) : "memory");
            });
            asm volatile("s_waitcnt vmcnt(0)" ::: "memory");
            if (lane == 0) {
                __hip_atomic_store((GAS unsigned*)(dws + WS_FLAG + ((tm * 8 + slice) * 8 + rank) * A2A_FSTRIDE), call, __ATOMIC_RELAXED, __HIP_MEMORY_SCOPE_SYSTEM);
                __hip_atomic_store(cnt, call, __ATOMIC_RELAXED, __HIP_MEMORY_SCOPE_AGENT);
            }
        } else {
            __builtin_amdgcn_s_barrier();
        }
    } else {
        __builtin_amdgcn_s_setprio(3);
        GAS unsigned* cnt = (GAS unsigned*)(myws + WS_CNT + bid * 128 + 64);
        GAS unsigned* tmo = (GAS unsigned*)(myws + WS_TMO);
        const auto rc = mkrsrc(myws, 65536);
        const unsigned c0 = __hip_atomic_load(cnt, __ATOMIC_RELAXED, __HIP_MEMORY_SCOPE_AGENT);
        const int tab = step_table(lane, rank, mt, h);
        START_BARRIER();
        if (lane == 0) __builtin_amdgcn_raw_ptr_buffer_load_lds(rc, (LAS void*)misc, 4, (unsigned)(WS_CNT + bid * 128 + 64), 0, 0, 16);
        asm volatile("s_waitcnt vmcnt(0)" ::: "memory");
        const unsigned call = __builtin_amdgcn_readfirstlane(c0) + 1;
        if (lane == 0) __hip_atomic_store(cnt, call, __ATOMIC_RELAXED, __HIP_MEMORY_SCOPE_AGENT);
        static_assert(NT + 4 <= 63, "lanes");
        GAS unsigned* flag = lane < NT ? (GAS unsigned*)(myws + WS_FLAG + ((mt * 8 + (tab & 7)) * 8 + (tab >> 3)) * A2A_FSTRIDE)
                           : lane < NT + 4 ? (GAS unsigned*)(myws + WS_XFLAG + bid * 128 + (lane - NT) * 4) : tmo;
        const unsigned long long t0 = __builtin_amdgcn_s_memrealtime();
        unsigned long long seen = 15;
        int ready = 4;
        while (true) {
            const int near = ready < NT ? ready - A2A_AHEAD : NT - A2A_X2 - 1;
            while (__builtin_amdgcn_readfirstlane(asm_lds_ld(sig_at(SIG_FULL, near))) != 0) __builtin_amdgcn_s_sleep(2);
            unsigned v = 0;
            if (lane == 63 || (lane < ready + 12 && lane < NT + 4 && !((seen >> lane) & 1))) v = __hip_atomic_load(flag, __ATOMIC_RELAXED, __HIP_MEMORY_SCOPE_SYSTEM);
            if (__builtin_amdgcn_readlane(v, 63) == call || __builtin_amdgcn_s_memrealtime() - t0 > DEADLINE) {
                if (lane == 0) { __hip_atomic_store(tmo, call, __ATOMIC_RELAXED, __HIP_MEMORY_SCOPE_AGENT); lds_st(misc + 1, 0u); }
                asm volatile("s_waitcnt lgkmcnt(0)" ::: "memory");
                break;
            }
            seen |= __builtin_amdgcn_ballot_w64(v >= call && lane < 63);
            const int r = __builtin_ctzll(~seen);
            if (r != ready) {
                ready = r; signal(rz, sig_at(SIG_RDY, r - 1));
            }
            if (r == NT + 4) break;
            __builtin_amdgcn_s_sleep(A2A_PSLEEP);
        }
        signal(rz, sig_at(SIG_RDY, 63));
        signal(rz, sig_at(SIG_X, 0));
    }
}
}

extern "C" void kernel_launch(int rank, int nranks, ncclComm_t comm,
                              hipStream_t stream,
                              void* const* d_in, const int* in_sizes, int n_in,
                              void* d_out, int out_size,
                              const int* in_dtypes, int out_dtype,
                              void* const* d_out_all,
                              void* const* d_ws_all, size_t ws_bytes) {
    using namespace a2a;
    if (nranks != NR || (unsigned)rank >= (unsigned)NR || n_in < 2 || ws_bytes < WS_NEED ||
        in_sizes[0] != 4096 * KC || in_sizes[1] != KTOT * NTOT || out_size != MLOC * NTOT) {
        fprintf(stderr, "a2a_gemm: unexpected problem shape\n");
        return;
    }
    Params p;
    p.x = (const char*)d_in[0]; p.w = (const char*)d_in[1]; p.out = (char*)d_out;
    for (int i = 0; i < NR; ++i) p.ws[i] = (char*)d_ws_all[i];
    p.rank = rank;
    hipLaunchKernelGGL(a2a_gemm, dim3(NWG), dim3(NTHREADS), 0, stream, p);
    HIP_CHECK(hipGetLastError());
}
```

```cpp
#include <hip/hip_runtime.h>
#include <rccl/rccl.h>
#include <cstdio>
#include <cstdint>
#include <type_traits>
#include <utility>

#define HIP_CHECK(expr) do { hipError_t _e = (expr); if (_e != hipSuccess) { \
    fprintf(stderr, "HIP error %s at %s:%d\n", hipGetErrorString(_e), __FILE__, __LINE__); \
    return; } } while (0)

namespace a2a {
#define LAS __attribute__((address_space(3)))
#define GAS __attribute__((address_space(1)))
typedef unsigned u32x4 __attribute__((ext_vector_type(4)));
typedef float f32x4 __attribute__((ext_vector_type(4)));
typedef float f32x16 __attribute__((ext_vector_type(16)));
typedef __bf16 bf16x8 __attribute__((ext_vector_type(8)));
typedef __bf16 bf16x4 __attribute__((ext_vector_type(4)));
typedef __bf16 bf16x2 __attribute__((ext_vector_type(2)));
typedef float f32x2 __attribute__((ext_vector_type(2)));

constexpr int NR = 8;
constexpr int MLOC = 512, KTOT = 4096, NTOT = 4096, KC = 512;
#define A2A_NPF 4
#define A2A_G0 14
constexpr int NWG = 256, NTHREADS = (8 + A2A_NPF) * 64;
#define A2A_N2 8
#define A2A_LSLEEP 0
#define A2A_PSLEEP 4
#define A2A_PINF 0
#define A2A_PAUSE -1
#define A2A_GATE 220
#define A2A_COOP 0
#define A2A_LATE 64
#define A2A_STAG 2
#define A2A_HOLD 0
#define A2A_HOLD2 6
#define A2A_PWIN 8
#define A2A_T0 110
#define A2A_TAU 34
#define A2A_AHEAD 6
#define A2A_MAP 3
#define A2A_WAUX 0
#define A2A_PAUX 0
#define A2A_OAUX 0
#define A2A_XBF16 1
#define A2A_XAUX 16
#define A2A_X1 3
#define A2A_X2 2
#define A2A_FSTRIDE 128
constexpr int N2 = A2A_N2;
constexpr int N1 = (64 - N2) / 2;
constexpr int NT = N1 + N2;
static_assert(N2 % 2 == 0 && N1 >= 4, "N2");
constexpr int NSA = 4, NSW = 4, SLOT = 16384, W_OFF = NSA * SLOT;
static_assert(N1 % 4 == 0 && N2 >= 2, "the wide loop is unrolled by the ring depth");
constexpr int SIG_OFF = W_OFF + NSW * SLOT;
constexpr int SIG_FULL = 0;
constexpr int SIG_FREE = 2;
constexpr int NSIG = 10;
constexpr int MISC_OFF = SIG_OFF + NSIG * 512;
constexpr int FLG_OFF = MISC_OFF + 64;
constexpr int LDS_BYTES = FLG_OFF + 256;
static_assert(LDS_BYTES <= 163840, "LDS");
constexpr size_t WS_CNT = 0;
constexpr size_t WS_TMO = 32768;
constexpr size_t WS_FLAG = 65536;
constexpr size_t WS_XFLAG = 131072;
constexpr size_t WS_STG = 1u << 20;
constexpr size_t STG_PAR = 4u << 20;
constexpr size_t WS_XBUF = WS_STG + 2 * STG_PAR;
constexpr size_t XBUF_BYTES = 8u << 20;
constexpr size_t WS_NEED = WS_XBUF + XBUF_BYTES;
constexpr unsigned long long DEADLINE = 50000000ull;
constexpr int NPUSH = 7 * 32;

struct Params {
    const char* x; const char* w; char* out;
    char* ws[NR];
    int rank;
};

#define STAMP(i) do {} while (0)

__device__ __forceinline__ __amdgpu_buffer_rsrc_t mkrsrc(const void* ptr, unsigned bytes) {
    return __builtin_amdgcn_make_buffer_rsrc((void*)ptr, 0, bytes, 0x00020000);
}
__device__ __forceinline__ unsigned lds_ld(LAS unsigned* a) { return __hip_atomic_load(a, __ATOMIC_RELAXED, __HIP_MEMORY_SCOPE_WORKGROUP); }
__device__ __forceinline__ void lds_st(LAS unsigned* a, unsigned v) { __hip_atomic_store(a, v, __ATOMIC_RELAXED, __HIP_MEMORY_SCOPE_WORKGROUP); }
__device__ __forceinline__ int aswz(int i) { return ((i << 1) ^ ((i >> 1) * 3)) & 3; }
__device__ __forceinline__ unsigned pack2(float a, float b) { f32x2 v = {a, b}; return __builtin_bit_cast(unsigned, __builtin_convertvector(v, bf16x2)); }
__device__ __forceinline__ unsigned asm_lds_ld(unsigned addr) {
    unsigned v;
    asm volatile("ds_read_b32 %0, %1\n\ts_waitcnt lgkmcnt(0)" : "=&v"(v) : "v"(addr) : "memory");
    return v;
}
__device__ __forceinline__ void signal(u32x4 r, unsigned lds_addr) {
    unsigned keep;
    asm volatile("s_mov_b32 %0, m0\n\ts_mov_b32 m0, %1\n\ts_nop 0\n\tbuffer_load_dword %2, %3, 0 offen lds\n\ts_mov_b32 m0, %0"
                 : "=&s"(keep) : "s"(lds_addr), "v"(0x7ffffff0u), "s"(r) : "memory");
}
template <int N, class F, int... I> __device__ __forceinline__ void static_for_impl(F f, std::integer_sequence<int, I...>) { (f(std::integral_constant<int, I>{}), ...); }
template <int N, class F> __device__ __forceinline__ void static_for(F f) { static_for_impl<N>(f, std::make_integer_sequence<int, N>{}); }
__device__ __forceinline__ int step_table(int lane, int rank, int mt, int h) {
    const int j = lane >> 3, pi = 2 * ((((lane >> 1) & 3) - mt) & 3) + (lane & 1), gi = 8 + 7 * j + pi;
    const bool wide = pi < 7 && gi < 2 * N1 && (gi & 1) == h, narrow = pi < 7 && gi >= 2 * N1;
    const unsigned long long below = (1ull << lane) - 1;
    const int pos = wide ? 4 + __builtin_popcountll(__builtin_amdgcn_ballot_w64(wide) & below)
                         : N1 + __builtin_popcountll(__builtin_amdgcn_ballot_w64(narrow) & below);
    const int v = __builtin_amdgcn_ds_permute((wide || narrow ? pos : 63) * 4, (((rank + 1 + pi) & 7) << 3) | j);
    return lane < 4 ? (rank << 3) | (2 * lane + h) : v;
}
#define START_BARRIER() do { asm volatile("s_waitcnt lgkmcnt(0)" ::: "memory"); __builtin_amdgcn_s_barrier(); asm volatile("" ::: "memory"); } while (0)
#define SB() __builtin_amdgcn_sched_barrier(0)

__global__ __launch_bounds__(NTHREADS, 3) void a2a_gemm(const Params p) {
    __shared__ __attribute__((aligned(16))) char smem[LDS_BYTES];
    LAS char* lds = (LAS char*)smem;
    const unsigned lds0 = (unsigned)(size_t)lds, sig0 = lds0 + SIG_OFF;
    LAS unsigned* misc = (LAS unsigned*)(lds + MISC_OFF);
    auto sig_at = [&](int arr, int e) { return sig0 + arr * 512 + (63 - e) * 4; };
    const int bid = blockIdx.x, lane = threadIdx.x & 63;
    const int wid = __builtin_amdgcn_readfirstlane(threadIdx.x >> 6);
    const int rank = p.rank;
    const int h = bid & 1, loc = bid >> 3, mt = loc & 3, nt = (loc >> 2) * 4 + ((bid >> 1) & 3);
    char* const myws = p.ws[rank];
    const u32x4 rz = {(unsigned)(size_t)p.x, (unsigned)((size_t)p.x >> 32) & 0xffffu, 16u, 0x00020000u};

    if (wid == 0) STAMP(0);
    if (wid < 4) {
        for (int i = threadIdx.x; i < (NSIG * 512 + 64) / 16; i += 256) *(LAS u32x4*)(lds + SIG_OFF + i * 16) = (u32x4){~0u, ~0u, ~0u, ~0u};
        if (wid == 3) *(LAS unsigned*)(lds + FLG_OFF + lane * 4) = 0u;
    }
    if (wid < 4) {
        START_BARRIER();
        __builtin_amdgcn_s_setprio(3);
        const int wm = wid >> 1, wn = wid & 1, q = (lane >> 2) & 3, g = lane >> 4, r16 = lane & 15, f = (q >> 1) | ((g & 1) << 1);
        const auto rxb = mkrsrc(myws + WS_XBUF, XBUF_BYTES);
        const unsigned a0 = lds0 + (wm * 64 + r16) * 64 + ((g ^ aswz(r16 >> 2)) * 16);
        const unsigned w0 = lds0 + W_OFF + (8 * g + q) * 128 + (((wn * 2) ^ f) * 32) + (lane & 3) * 8, w1 = w0 ^ 32;
        unsigned fbase = sig_at(SIG_FULL, 4), free_a = sig_at(SIG_FREE + wid, 0);
        auto wait_zero = [&](unsigned a) {
            while (__builtin_amdgcn_ballot_w64(*(volatile LAS unsigned*)(size_t)a != 0) != 0) __builtin_amdgcn_s_sleep(1);
            asm volatile("" ::: "memory");
        };
        bf16x8 af[2][4], wf[2][4];
        bf16x4 wl;
        f32x4 acc[4][4];
        unsigned fv;
#pragma unroll
        for (int i = 0; i < 4; ++i) { af[1][i] = (bf16x8)(__bf16)0.f; wf[1][i] = (bf16x8)(__bf16)0.f; }
#pragma unroll
        for (int i = 0; i < 16; ++i) acc[i >> 2][i & 3] = (f32x4){0.f, 0.f, 0.f, 0.f};
#define FULL_AT(U) (fbase + ((U) & 1) * 512 + (4 - ((U) >> 1)) * 4)
#define FV() do { if constexpr (more) { fv = *(volatile LAS unsigned*)(size_t)FULL_AT(U + 1); SB(); } } while (0)
#define LGKM0() do { __builtin_amdgcn_s_waitcnt(0xC07F); SB(); } while (0)
#define HAVE() do { if constexpr (more) { miss = __builtin_amdgcn_ballot_w64(fv != 0); asm volatile("" : "+s"(miss)); SB(); } } while (0)
#define FREE() do { if constexpr (sig) { \
            if constexpr (H) asm volatile("s_mov_b32 m0, %0\n\ts_sub_u32 %0, %0, 2052\n\tbuffer_load_dword %1, %2, 0 offen lds" : "+s"(free_a) : "v"(0x7ffffff0u), "s"(rz) : "memory", "scc"); \
            else asm volatile("s_mov_b32 m0, %0\n\ts_add_u32 %0, %0, 2048\n\tbuffer_load_dword %1, %2, 0 offen lds" : "+s"(free_a) : "v"(0x7ffffff0u), "s"(rz) : "memory", "scc"); \
            SB(); } } while (0)
#define RDW(cb, t2) do { const bf16x4 w_ = __builtin_amdgcn_ds_read_tr16_b64_v4bf16((LAS bf16x4*)(size_t)(((cb & 1) ? w1 : w0) + S * SLOT + H * 4096 + (cb >> 1) * 8192 + t2 * 512)); \
            if (t2 == 0) wl = w_; else wf[H][cb] = __builtin_shufflevector(wl, w_, 0, 1, 2, 3, 4, 5, 6, 7); SB(); } while (0)
#define RDA(rb) do { af[H][rb] = *(LAS bf16x8*)(size_t)(a0 + S * SLOT + H * 8192 + rb * 1024); SB(); } while (0)
#define MM(rb, cb) do { acc[rb][cb] = __builtin_amdgcn_mfma_f32_16x16x32_bf16(wf[H ^ 1][cb], af[H ^ 1][rb], acc[rb][cb], 0, 0, 0); SB(); } while (0)
#define MMEND() do { asm volatile("s_nop 7\n\ts_nop 1" ::: "memory"); SB(); } while (0)
        auto half = [&](auto uc, auto pw, auto wide, auto sig, auto more) {
            constexpr int U = decltype(uc)::value, S = U >> 1, H = U & 1;
            unsigned long long miss = 0;
            if constexpr (wide) {
                MM(0, 0); RDW(0, 0); MM(0, 1); RDW(0, 1); RDA(0); MM(0, 2); RDW(1, 0); MM(0, 3); RDW(1, 1);
                MM(1, 0); RDW(2, 0); RDW(2, 1); MM(1, 1); RDW(3, 0); MM(1, 2); RDW(3, 1); MM(1, 3); RDA(1);
                MM(2, 0); RDA(2); MM(2, 1); RDA(3); FV(); MM(2, 2); MM(2, 3);
                MM(3, 0); MM(3, 1); LGKM0(); HAVE(); MM(3, 2); FREE(); MM(3, 3);
            } else if constexpr (pw) {
                MM(0, 0); RDW(0, 0); MM(0, 1); RDW(0, 1); MM(0, 2); RDA(0); MM(0, 3); RDW(1, 0);
                MM(1, 0); RDW(1, 1); MM(1, 1); RDA(1); MM(1, 2); RDA(2); MM(1, 3); RDA(3);
                MM(2, 0); MM(2, 1); FV(); MM(2, 2); MM(2, 3);
                MM(3, 0); MM(3, 1); LGKM0(); HAVE(); MM(3, 2); FREE(); MM(3, 3);
            } else {
                MM(0, 0); RDW(0, 0); RDW(0, 1); MM(0, 1); RDA(0); RDA(1); MM(1, 0); RDW(1, 0); RDW(1, 1); MM(1, 1); RDA(2); RDA(3); FV();
                MM(2, 0); MM(2, 1); MM(3, 0); LGKM0(); HAVE(); FREE(); MM(3, 1);
            }
            if constexpr (more) {
                if (__builtin_expect(miss != 0, 0)) wait_zero(FULL_AT(U + 1));
                if constexpr (U == 7) { fbase -= 16; asm volatile("" : "+v"(fbase)); }
            }
        };
        using T_ = std::integral_constant<bool, true>; using F_ = std::integral_constant<bool, false>;
        __builtin_amdgcn_s_waitcnt(0x0F70);
        wait_zero(FULL_AT(0));
        if (wid == 0) STAMP(1);
#pragma unroll 1
        for (int i = 0; i < N1 / 4; ++i) {
            static_for<8>([&](auto uc) { half(uc, T_{}, T_{}, T_{}, T_{}); });
        }
        constexpr int NX = A2A_XBF16 ? 4 : 8, X1 = A2A_X1, X2 = N2 - A2A_X2;
        static_assert(1 <= X1 && X1 <= X2 && X2 <= N2, "exchange points");
        u32x4 xp[NX];
        unsigned call;
        static_for<N2>([&](auto ic) {
            constexpr int i = decltype(ic)::value;
            using Sig = std::integral_constant<bool, (i + 4 < N2)>;
            half(std::integral_constant<int, (i & 3) * 2>{}, std::integral_constant<bool, i == 0>{}, F_{}, Sig{}, T_{});
            if constexpr (i == 0) {
                MMEND();
                if (wid == 0) STAMP(6);
#pragma unroll
                for (int k = 0; k < NX; ++k) {
                    const f32x4 a = acc[k][2], b = acc[k][3];
                    const u32x4 v = {pack2(a[0], a[1]), pack2(a[2], a[3]), pack2(b[0], b[1]), pack2(b[2], b[3])};
                    __builtin_amdgcn_raw_buffer_store_b128(v, rxb, ((bid ^ 1) * 4 + wid) * (NX * 1024) + lane * 16 + k * 1024, 0, A2A_XAUX);
                }
                SB();
            }
            half(std::integral_constant<int, (i & 3) * 2 + 1>{}, F_{}, F_{}, Sig{}, std::integral_constant<bool, (i + 1 < N2)>{});
            if constexpr (i + 1 == X1) {
                constexpr int nsig = 2 * (X1 < N2 - 4 ? X1 : N2 - 4) - 1;
                asm volatile("s_waitcnt vmcnt(%0)" :: "i"(nsig > 0 ? nsig : 0) : "memory");
                call = __builtin_amdgcn_readfirstlane(lds_ld(misc)) + 1;
                if (lane == 0) __hip_atomic_store((GAS unsigned*)(myws + WS_XFLAG + (bid ^ 1) * 128 + wid * 4), call, __ATOMIC_RELAXED, __HIP_MEMORY_SCOPE_AGENT);
                if (wid == 0) STAMP(14);
                SB();
            }
            if constexpr (i + 1 == X2) {
                while ((unsigned)__builtin_amdgcn_readfirstlane(lds_ld((LAS unsigned*)(lds + FLG_OFF) + NT + wid)) < call) __builtin_amdgcn_s_sleep(1);
                asm volatile("" ::: "memory");
                if (wid == 0) STAMP(7);
#pragma unroll
                for (int k = 0; k < NX; ++k) xp[k] = __builtin_amdgcn_raw_buffer_load_b128(rxb, lane * 16 + k * 1024, (bid * 4 + wid) * (NX * 1024), 16);
                SB();
            }
        });
#pragma unroll
        for (int rb = 0; rb < 4; ++rb) { constexpr int H = 0; MM(rb, 0); MM(rb, 1); }
        MMEND();
        const bool bad = __builtin_amdgcn_readfirstlane(lds_ld(misc + 1)) == 0;
        const auto ro = mkrsrc(p.out, MLOC * NTOT * 2);
        const unsigned vo = (unsigned)(mt * 128 + wm * 64 + r16) * 8192u + nt * 256 + h * 128 + wn * 64 + (g & 1) * 32 + (g >> 1) * 16;
#pragma unroll
        for (int rb = 0; rb < 4; ++rb) {
            const u32x4 x_ = xp[rb];
            const f32x4 a = acc[rb][0] + __builtin_bit_cast(f32x4, (u32x4){x_[0] << 16, x_[0] & 0xffff0000u, x_[1] << 16, x_[1] & 0xffff0000u});
            const f32x4 b = acc[rb][1] + __builtin_bit_cast(f32x4, (u32x4){x_[2] << 16, x_[2] & 0xffff0000u, x_[3] << 16, x_[3] & 0xffff0000u});
            const auto s0 = __builtin_amdgcn_permlane16_swap(pack2(a[0], a[1]), pack2(b[0], b[1]), false, false);
            const auto s1 = __builtin_amdgcn_permlane16_swap(pack2(a[2], a[3]), pack2(b[2], b[3]), false, false);
            u32x4 o = {s0[0], s1[0], s0[1], s1[1]};
            if (bad) o = (u32x4){0x7fc07fc0u, 0x7fc07fc0u, 0x7fc07fc0u, 0x7fc07fc0u};
            __builtin_amdgcn_raw_buffer_store_b128(o, ro, vo + rb * 16 * 8192, 0, A2A_OAUX);
        }
    } else if (wid < 6) {
        const int j = wid & 1;
        unsigned tpause = (unsigned)__builtin_amdgcn_s_memrealtime() + A2A_T0 + bid / 7 * A2A_TAU - A2A_PAUSE;
        bool pending = A2A_PAUSE >= 0 && bid < NPUSH;
        unsigned full_a = sig_at(SIG_FULL + j, 0);
        const int tab = step_table(lane, rank, mt, h);
#define LD4(rsrc, ldsp, v0, v1, so, stride, aux) do { \
            __builtin_amdgcn_raw_ptr_buffer_load_lds(rsrc, (LAS void*)(ldsp), 16, v0, (so), 0, aux); \
            __builtin_amdgcn_raw_ptr_buffer_load_lds(rsrc, (LAS void*)(ldsp), 16, v1, (so) + ((stride) - 1024), 1024, aux); \
            __builtin_amdgcn_raw_ptr_buffer_load_lds(rsrc, (LAS void*)(ldsp), 16, v0, (so) + 2 * ((stride) - 1024), 2048, aux); \
            __builtin_amdgcn_raw_ptr_buffer_load_lds(rsrc, (LAS void*)(ldsp), 16, v1, (so) + 3 * ((stride) - 1024), 3072, aux); } while (0)
        const auto rw = mkrsrc(p.w, KTOT * NTOT * 2);
        const unsigned vwb = (unsigned)(j * 32 + (lane >> 3)) * 8192u + nt * 256;
        const unsigned vwe = vwb + (((lane & 7) ^ (((lane >> 4) & 1) * 2)) * 16), vwo = vwb + (((lane & 7) ^ ((((lane >> 4) & 1) | 2) * 2)) * 16);
        auto load_w = [&](int t, unsigned so) {
            LAS char* const sl = lds + W_OFF + (t & 3) * SLOT + j * 4096;
            LD4(rw, sl, vwe, vwo, so, 65536, A2A_WAUX);
            if (t < N1) LD4(rw, sl + 8192, vwe, vwo, so ^ 128, 65536, A2A_WAUX);
        };
        const unsigned ch = ((lane & 3) ^ aswz(lane >> 4)) * 16;
        const auto rx = mkrsrc(p.x, 4096 * KC * 2);
        const unsigned vx = (unsigned)(rank * 512 + mt * 128 + (lane >> 2)) * 1024u + j * 64 + ch;
        START_BARRIER();
#pragma unroll 1
        for (int t = 0; t < 4; ++t) {
            LAS char* const sl = lds + t * SLOT + j * 8192;
            if (t == 3) asm volatile("s_waitcnt vmcnt(40)" ::: "memory");
            load_w(t, (rank * 8 + 2 * t + h) * (64u * 8192u) + h * 128);
            LD4(rx, sl, vx, vx, (2 * t + h) * 128, 16384, 0);
            LD4(rx, sl + 4096, vx, vx, (2 * t + h) * 128 + 65536, 16384, 0);
            signal(rz, full_a); full_a -= 4;
        }
        const auto rs = mkrsrc(myws + WS_STG, 2 * STG_PAR);
        const unsigned vs = (unsigned)j * 32768u + mt * 8192 + (lane >> 2) * 64 + ch;
        unsigned c0;
        while ((c0 = __builtin_amdgcn_readfirstlane(asm_lds_ld(lds0 + MISC_OFF))) == ~0u) __builtin_amdgcn_s_sleep(1);
        asm volatile("" : "+s"(c0));
        const unsigned call = c0 + 1, par = (call & 1) * (unsigned)STG_PAR;
        unsigned watch = (lane & 4) ? lds0 + FLG_OFF + 16 : sig_at(SIG_FREE + j * 4 + (lane & 3), 0);
        const unsigned flip = (lane & 4) ? 0u : ~0u, dw = (lane & 4) ? 4 : -4;
#pragma unroll 1
        for (int t = 4; t < NT; ++t, watch += dw) {
            const unsigned tile = __builtin_amdgcn_readlane(tab, t);
            unsigned sw = tile * (64u * 8192u) + h * 128, sa = tile * 65536u + par, busy;
            asm volatile("s_waitcnt vmcnt(40)" : "+s"(sw), "+s"(sa) :: "memory");
            const unsigned now = 0;
            while ((busy = (unsigned)__builtin_amdgcn_ballot_w64((asm_lds_ld(watch) ^ flip) < call)) & 15) {}
            if (pending && (int)(now - tpause) >= 0) {
                while (__builtin_amdgcn_readfirstlane(asm_lds_ld(lds0 + MISC_OFF + 8)) != 0) __builtin_amdgcn_s_sleep(1);
                pending = false;
            }
            load_w(t, sw);
            if (busy & 16) {
                while ((unsigned)__builtin_amdgcn_ballot_w64(asm_lds_ld(watch) < call) & 16) __builtin_amdgcn_s_sleep(1);
            }
            LAS char* const sl = lds + (t & 3) * SLOT + j * 8192;
            LD4(rs, sl, vs, vs, sa, 1024, 17);
            LD4(rs, sl + 4096, vs, vs, sa + 4096, 1024, 17);
            signal(rz, full_a); full_a -= 4;
        }
        if (wid == 4) STAMP(3);
    } else if (wid == 6) {
        if (bid < NPUSH) {
            __builtin_amdgcn_s_setprio(3);
            const unsigned tstart = (unsigned)__builtin_amdgcn_s_memrealtime();
            const int slot = bid / 7, di = bid - slot * 7, dest = (rank + 1 + di) & 7, slice = slot >> 2, tm = (slot - ((6 - di) >> 1)) & 3;
            GAS unsigned* cnt = (GAS unsigned*)(myws + WS_CNT + bid * 128);
            const auto rx = mkrsrc(p.x, 4096 * KC * 2);
            const unsigned vx = (unsigned)(dest * 512 + tm * 128 + ((lane >> 2) & 7)) * 1024u + slice * 128 + (lane >> 5) * 64 + (lane & 3) * 16;
            char* const dws = p.ws[dest];
            if (slot >= A2A_LATE) {
                __builtin_amdgcn_s_barrier();
                while ((int)((unsigned)__builtin_amdgcn_s_memrealtime() - tstart) < A2A_T0 + (slot - 5) * A2A_TAU) __builtin_amdgcn_s_sleep(4);
            }
            const unsigned c0 = __hip_atomic_load(cnt, __ATOMIC_RELAXED, __HIP_MEMORY_SCOPE_AGENT);
            u32x4 v[16];
#pragma unroll
            for (int i = 0; i < 16; ++i) v[i] = __builtin_amdgcn_raw_buffer_load_b128(rx, vx, i * 8192, A2A_PAUX);
            if (slot >= A2A_HOLD2 && slot < A2A_LATE) __builtin_amdgcn_s_barrier();
            __builtin_amdgcn_s_waitcnt(0x0F70);
            SB();
            const unsigned call = __builtin_amdgcn_readfirstlane(c0) + 1;
            char* const dst = dws + WS_STG + (call & 1) * STG_PAR + (rank * 8 + slice) * 65536 + (lane >> 5) * 32768 + tm * 8192 + (lane & 31) * 16 + 4096;
            while ((int)((unsigned)__builtin_amdgcn_s_memrealtime() - tstart) < A2A_T0 + slot * A2A_TAU) __builtin_amdgcn_s_sleep(1);
            static_for<16>([&](auto ic) {
                constexpr int i = decltype(ic)::value;
                char* const d = dst; const u32x4 vi = v[i];
                asm volatile("global_store_dwordx4 %0, %1, off offset:%2 sc0 sc1\n\ts_nop 1" :: "v"(d), "v"(vi), "i"(i * 512 - 4096) : "memory");
            });
            if (slot >= A2A_HOLD && slot < A2A_HOLD2) __builtin_amdgcn_s_barrier();
            asm volatile("s_waitcnt vmcnt(0)" ::: "memory");
            if (lane == 0) {
                __hip_atomic_store((GAS unsigned*)(dws + WS_FLAG + ((tm * 8 + slice) * 8 + rank) * A2A_FSTRIDE), call, __ATOMIC_RELAXED, __HIP_MEMORY_SCOPE_SYSTEM);
                __hip_atomic_store(cnt, call, __ATOMIC_RELAXED, __HIP_MEMORY_SCOPE_AGENT);
            }
            static_assert(A2A_HOLD == 0, "the word below is initialised in front of the barrier");
            if (A2A_PAUSE >= 0 && lane == 0) lds_st(misc + 2, 0u);
            if (slot < A2A_HOLD) __builtin_amdgcn_s_barrier();
        } else {
            __builtin_amdgcn_s_barrier();
        }
    } else if (wid >= 8) {
        unsigned vj = 0;
        __builtin_amdgcn_s_barrier();
        const char* const wq = p.w + (size_t)(mt * 16 + (wid - 8) * 4) * 8192 + nt * 256;
        unsigned junk = 0;
#pragma unroll 1
        for (int gi = A2A_G0 + h; gi < 64; gi += gi + 2 < 2 * N1 ? 2 : gi < 2 * N1 ? 2 * N1 - gi : 1) {
            const int u = gi - 8, slice = u / 7, src = (rank + 1 + u - slice * 7) & 7;
            const char* const a = wq + (size_t)(src * 8 + slice) * (64 * 8192);
#define PF2(r) "s_load_dword %0, %1, " #r "*8192\n\ts_load_dword %0, %1, " #r "*8192+64\n\t"
            asm volatile(PF2(0) PF2(1) PF2(2) PF2(3) "s_nop 0" : "+s"(junk) : "s"(a + h * 128) : "memory");
            if (gi < 2 * N1) asm volatile(PF2(0) PF2(1) PF2(2) PF2(3) "s_nop 0" : "+s"(junk) : "s"(a + (h ^ 1) * 128) : "memory");
        }
        asm volatile("s_waitcnt vmcnt(0) lgkmcnt(0)" : "+s"(junk), "+v"(vj) :: "memory");
    } else {
        __builtin_amdgcn_s_setprio(3);
        const unsigned long long t0 = __builtin_amdgcn_s_memrealtime();
        GAS unsigned* cnt = (GAS unsigned*)(myws + WS_CNT + bid * 128 + 64);
        GAS unsigned* tmo = (GAS unsigned*)(myws + WS_TMO);
        const auto rc = mkrsrc(myws, WS_STG);
        const unsigned c0 = __hip_atomic_load(cnt, __ATOMIC_RELAXED, __HIP_MEMORY_SCOPE_AGENT);
        const int tab = step_table(lane, rank, mt, h);
        START_BARRIER();
        if (lane == 0) __builtin_amdgcn_raw_ptr_buffer_load_lds(rc, (LAS void*)misc, 4, (unsigned)(WS_CNT + bid * 128 + 64), 0, 0, 16);
        asm volatile("s_waitcnt vmcnt(0)" ::: "memory");
        const unsigned call = __builtin_amdgcn_readfirstlane(c0) + 1;
        if (lane == 0) __hip_atomic_store(cnt, call, __ATOMIC_RELAXED, __HIP_MEMORY_SCOPE_AGENT);
        static_assert(NT + 4 <= 63, "lanes");
        const unsigned foff = lane < NT ? WS_FLAG + ((mt * 8 + (tab & 7)) * 8 + (tab >> 3)) * A2A_FSTRIDE
                            : lane < NT + 4 ? WS_XFLAG + bid * 128 + (lane - NT) * 4 : WS_TMO;
        const unsigned due = lane < NT ? (unsigned)t0 + A2A_T0 + A2A_GATE + ((tab & 7) * 4 + ((mt + ((((tab >> 3) - rank - 1) & 7) >> 1)) & 3)) * A2A_TAU : (unsigned)t0;
        const unsigned long long given = 15;
        bool bad = false;
        for (unsigned n = 1;; ++n) {
            const unsigned v = asm_lds_ld(lds0 + FLG_OFF + lane * 4);
            const unsigned long long seen = __builtin_amdgcn_ballot_w64(v >= call) | given;
            const int ready = __builtin_ctzll(~seen);
            const unsigned long long now = __builtin_amdgcn_s_memrealtime();
            bad = __builtin_amdgcn_readlane(v, 63) == call || now - t0 > DEADLINE;
            if (bad || ready >= NT + 4) break;
            const int near = ready < NT ? ready - A2A_AHEAD : NT - A2A_X2 - 1;
            if (near < 0 || __builtin_amdgcn_readfirstlane(asm_lds_ld(sig_at(SIG_FULL, near))) == 0)
            {
                if ((lane == 63 && (n & 15) == 0) || (lane < (ready < NT ? min(NT, ready + A2A_PWIN) : NT + 4) && !((seen >> lane) & 1) && (int)((unsigned)now - due) >= 0))
                    __builtin_amdgcn_raw_ptr_buffer_load_lds(rc, (LAS void*)(lds + FLG_OFF), 4, foff, 0, 0, 17);
                asm volatile("s_waitcnt vmcnt(%0)" :: "i"(A2A_PINF) : "memory");
            }
            __builtin_amdgcn_s_sleep(A2A_PSLEEP);
        }
        if (bad) {
            asm volatile("s_waitcnt vmcnt(0)" ::: "memory");
            if (lane == 0) { __hip_atomic_store(tmo, call, __ATOMIC_RELAXED, __HIP_MEMORY_SCOPE_AGENT); lds_st(misc + 1, 0u); }
            lds_st((LAS unsigned*)(lds + FLG_OFF) + lane, call);
        }
    }
}
}

extern "C" void kernel_launch(int rank, int nranks, ncclComm_t comm,
                              hipStream_t stream,
                              void* const* d_in, const int* in_sizes, int n_in,
                              void* d_out, int out_size,
                              const int* in_dtypes, int out_dtype,
                              void* const* d_out_all,
                              void* const* d_ws_all, size_t ws_bytes) {
    using namespace a2a;
    if (nranks != NR || (unsigned)rank >= (unsigned)NR || n_in < 2 || ws_bytes < WS_NEED ||
        in_sizes[0] != 4096 * KC || in_sizes[1] != KTOT * NTOT || out_size != MLOC * NTOT) {
        fprintf(stderr, "a2a_gemm: unexpected problem shape\n");
        return;
    }
    Params p;
    p.x = (const char*)d_in[0]; p.w = (const char*)d_in[1]; p.out = (char*)d_out;
    for (int i = 0; i < NR; ++i) p.ws[i] = (char*)d_ws_all[i];
    p.rank = rank;
    hipLaunchKernelGGL(a2a_gemm, dim3(NWG), dim3(NTHREADS), 0, stream, p);
    HIP_CHECK(hipGetLastError());
}
```

```cpp
#include <hip/hip_runtime.h>
#include <rccl/rccl.h>
#include <cstdio>
#include <cstdint>
#include <type_traits>
#include <utility>

#define HIP_CHECK(expr) do { hipError_t _e = (expr); if (_e != hipSuccess) { \
    fprintf(stderr, "HIP error %s at %s:%d\n", hipGetErrorString(_e), __FILE__, __LINE__); \
    return; } } while (0)

namespace a2a {
#define LAS __attribute__((address_space(3)))
#define GAS __attribute__((address_space(1)))
typedef unsigned u32x4 __attribute__((ext_vector_type(4)));
typedef float f32x4 __attribute__((ext_vector_type(4)));
typedef __bf16 bf16x8 __attribute__((ext_vector_type(8)));
typedef __bf16 bf16x4 __attribute__((ext_vector_type(4)));
typedef __bf16 bf16x2 __attribute__((ext_vector_type(2)));
typedef float f32x2 __attribute__((ext_vector_type(2)));

#define A2A_NPF 4
#define A2A_G0 14
#define A2A_N2 4
#define A2A_X2 1
#define A2A_T0 90
#define A2A_TAU 33
#define A2A_HOLD 9
#define A2A_GATE0 170
#define A2A_GATE 220
#define A2A_AHEAD 6
#define A2A_PWIN 8
#define A2A_PSLEEP 4
#define A2A_LSLEEP 0
#define A2A_CPRIO 3
#define A2A_LPRIO 0

constexpr int NR = 8;
constexpr int MLOC = 512, KTOT = 4096, NTOT = 4096, KC = 512;
constexpr int NWG = 256, NWAVE = 8 + A2A_NPF, NTHREADS = NWAVE * 64;
constexpr int N2 = A2A_N2, N1 = (64 - N2) / 2, NT = N1 + N2;
static_assert(N2 % 2 == 0 && N2 >= 2 && N1 >= 8 && NT <= 63, "N2");
constexpr int SLOT = 16384, W_OFF = 4 * SLOT;
constexpr int SIG_OFF = 8 * SLOT;
constexpr int SIG_FULL = 0;
constexpr int SIG_FREE = 2;
constexpr int NSIG = 10;
constexpr int MISC_OFF = SIG_OFF + NSIG * 512;
constexpr int FLG_OFF = MISC_OFF + 64;
constexpr int LDS_BYTES = FLG_OFF + 256;
static_assert(LDS_BYTES <= 163840 && LDS_BYTES > 81920, "one workgroup per CU");
constexpr size_t WS_CNT = 0;
constexpr size_t WS_TMO = 32768;
constexpr size_t WS_FLAG = 65536;
constexpr size_t WS_STG = 1u << 20;
constexpr size_t STG_PAR = 4u << 20;
constexpr size_t WS_XBUF = WS_STG + 2 * STG_PAR;
constexpr size_t XBUF_BYTES = 6u << 20;
constexpr size_t WS_NEED = WS_XBUF + XBUF_BYTES;
constexpr unsigned long long DEADLINE = 50000000ull;
constexpr int NPUSH = 7 * 32;

struct Params {
    const char* x; const char* w; char* out;
    char* ws;
    char* peer[NR - 1];
    int rank; int pad;
};

#define STAMP(i) do {} while (0)

__device__ __forceinline__ __amdgpu_buffer_rsrc_t mkrsrc(const void* ptr, unsigned bytes) {
    return __builtin_amdgcn_make_buffer_rsrc((void*)ptr, 0, bytes, 0x00020000);
}
__device__ __forceinline__ unsigned lds_ld(LAS unsigned* a) { return __hip_atomic_load(a, __ATOMIC_RELAXED, __HIP_MEMORY_SCOPE_WORKGROUP); }
__device__ __forceinline__ void lds_st(LAS unsigned* a, unsigned v) { __hip_atomic_store(a, v, __ATOMIC_RELAXED, __HIP_MEMORY_SCOPE_WORKGROUP); }
__device__ __forceinline__ int aswz(int i) { return ((i << 1) ^ ((i >> 1) * 3)) & 3; }
__device__ __forceinline__ unsigned pack2(float a, float b) { f32x2 v = {a, b}; return __builtin_bit_cast(unsigned, __builtin_convertvector(v, bf16x2)); }
__device__ __forceinline__ unsigned asm_lds_ld(unsigned addr) {
    unsigned v;
    asm volatile("ds_read_b32 %0, %1\n\ts_waitcnt lgkmcnt(0)" : "=&v"(v) : "v"(addr) : "memory");
    return v;
}
__device__ __forceinline__ void signal(u32x4 r, unsigned lds_addr) {
    unsigned keep;
    asm volatile("s_mov_b32 %0, m0\n\ts_mov_b32 m0, %1\n\ts_nop 0\n\tbuffer_load_dword off, %2, 0 lds\n\ts_mov_b32 m0, %0"
                 : "=&s"(keep) : "s"(lds_addr), "s"(r) : "memory");
}
template <int N, class F, int... I> __device__ __forceinline__ void static_for_impl(F f, std::integer_sequence<int, I...>) { (f(std::integral_constant<int, I>{}), ...); }
template <int N, class F> __device__ __forceinline__ void static_for(F f) { static_for_impl<N>(f, std::make_integer_sequence<int, N>{}); }
__device__ __forceinline__ int step_table(int lane, int rank, int mt, int h) {
    const int j = lane >> 3, pi = 2 * ((((lane >> 1) & 3) - mt) & 3) + (lane & 1), gi = 8 + 7 * j + pi;
    const bool wide = pi < 7 && gi < 2 * N1 && (gi & 1) == h, narrow = pi < 7 && gi >= 2 * N1;
    const unsigned long long below = (1ull << lane) - 1, mw = __builtin_amdgcn_ballot_w64(wide), mn = __builtin_amdgcn_ballot_w64(narrow);
    const int pos = wide ? 4 + __builtin_popcountll(mw & below) : narrow ? N1 + __builtin_popcountll(mn & below) : 63;
    const int v = __builtin_amdgcn_ds_permute(pos * 4, (((rank + 1 + pi) & 7) << 3) | j);
    return lane < 4 ? (rank << 3) | (2 * lane + h) : v;
}
#define START_BARRIER() do { asm volatile("s_waitcnt lgkmcnt(0)" ::: "memory"); __builtin_amdgcn_s_barrier(); asm volatile("" ::: "memory"); } while (0)
#define SB() __builtin_amdgcn_sched_barrier(0)

__global__ __launch_bounds__(NTHREADS, (NWAVE + 3) / 4) void a2a_gemm(const Params p) {
    __shared__ __attribute__((aligned(16))) char smem[LDS_BYTES];
    LAS char* lds = (LAS char*)smem;
    const unsigned lds0 = (unsigned)(size_t)lds, sig0 = lds0 + SIG_OFF;
    LAS unsigned* misc = (LAS unsigned*)(lds + MISC_OFF);
    auto sig_at = [&](int arr, int e) { return sig0 + arr * 512 + (63 - e) * 4; };
    const int bid = blockIdx.x, lane = threadIdx.x & 63;
    const int wid = __builtin_amdgcn_readfirstlane(threadIdx.x >> 6);
    const int rank = p.rank;
    const int h = bid & 1, mt = (bid >> 3) & 3, nt = (bid >> 5) * 4 + ((bid >> 1) & 3);
    char* const myws = p.ws;
    const u32x4 rz = {(unsigned)(size_t)p.x, (unsigned)((size_t)p.x >> 32) & 0xffffu, 0u, 0x00020000u};

    if (wid < 4) {
        if (wid == 0) STAMP(0);
        for (int i = threadIdx.x; i < (NSIG * 512 + 64) / 16; i += 256) *(LAS u32x4*)(lds + SIG_OFF + i * 16) = (u32x4){~0u, ~0u, ~0u, ~0u};
        if (wid == 3) *(LAS unsigned*)(lds + FLG_OFF + lane * 4) = 0u;
        START_BARRIER();
        __builtin_amdgcn_s_setprio(A2A_CPRIO);
        const int wm = wid >> 1, wn = wid & 1, q = (lane >> 2) & 3, g = lane >> 4, r16 = lane & 15, f = (q >> 1) | ((g & 1) << 1);
        const unsigned a0 = lds0 + (wm * 64 + r16) * 64 + ((g ^ aswz(r16 >> 2)) * 16);
        const unsigned w0 = lds0 + W_OFF + (8 * g + q) * 128 + (((wn * 2) ^ f) * 32) + (lane & 3) * 8, w1 = w0 ^ 32;
        unsigned fbase = sig_at(SIG_FULL, 4);
        unsigned free_a = sig_at(SIG_FREE + wid, 0);
        char* outp = p.out;
        asm volatile("" : "+s"(outp));
        auto wait_zero = [&](unsigned a) {
            while (__builtin_amdgcn_ballot_w64(*(volatile LAS unsigned*)(size_t)a != 0) != 0) __builtin_amdgcn_s_sleep(1);
            asm volatile("" ::: "memory");
        };
        bf16x8 af[2][4], wf[2][4];
        bf16x4 wl;
        f32x4 acc[4][4];
        unsigned fv;
#pragma unroll
        for (int i = 0; i < 4; ++i) { af[1][i] = (bf16x8)(__bf16)0.f; wf[1][i] = (bf16x8)(__bf16)0.f; }
#pragma unroll
        for (int i = 0; i < 16; ++i) acc[i >> 2][i & 3] = (f32x4){0.f, 0.f, 0.f, 0.f};
#define FULL_AT(U) (fbase + ((U) & 1) * 512 + (4 - ((U) >> 1)) * 4)
#define FV() do { if constexpr (more) { fv = *(volatile LAS unsigned*)(size_t)FULL_AT(U + 1); SB(); } } while (0)
#define LGKM0() do { __builtin_amdgcn_s_waitcnt(0xC07F); SB(); } while (0)
#define HAVE() do { if constexpr (more) { miss = __builtin_amdgcn_ballot_w64(fv != 0); asm volatile("" : "+s"(miss)); SB(); } } while (0)
#define M0SET() do { if constexpr (sig) { \
            if constexpr (H) asm volatile("s_mov_b32 m0, %0\n\ts_sub_u32 %0, %0, 2052" : "+s"(free_a) :: "memory", "scc"); \
            else asm volatile("s_mov_b32 m0, %0\n\ts_add_u32 %0, %0, 2048" : "+s"(free_a) :: "memory", "scc"); \
            SB(); } } while (0)
#define FREE() do { if constexpr (sig) { const u32x4 rz_ = rz; asm volatile("buffer_load_dword off, %0, 0 lds" :: "s"(rz_) : "memory"); SB(); } } while (0)
#define RDW(cb, t2) do { const bf16x4 w_ = __builtin_amdgcn_ds_read_tr16_b64_v4bf16((LAS bf16x4*)(size_t)(((cb & 1) ? w1 : w0) + S * SLOT + H * 4096 + (cb >> 1) * 8192 + t2 * 512)); \
            if (t2 == 0) wl = w_; else wf[H][cb] = __builtin_shufflevector(wl, w_, 0, 1, 2, 3, 4, 5, 6, 7); SB(); } while (0)
#define RDA(rb) do { af[H][rb] = *(LAS bf16x8*)(size_t)(a0 + S * SLOT + H * 8192 + rb * 1024); SB(); } while (0)
#define MM(rb, cb) do { f32x4& c_ = acc[rb][cb]; const bf16x8 w_ = wf[H ^ 1][cb], a_ = af[H ^ 1][rb]; \
            asm volatile("v_mfma_f32_16x16x32_bf16 %0, %1, %2, %0" : "+v"(c_) : "v"(w_), "v"(a_)); SB(); } while (0)
#define MMEND() do { asm volatile("s_nop 7\n\ts_nop 1" ::: "memory"); SB(); } while (0)
        auto half = [&](auto uc, auto pw, auto wide, auto sig, auto more) {
            constexpr int U = decltype(uc)::value, S = U >> 1, H = U & 1;
            unsigned long long miss = 0;
            if constexpr (wide) {
                MM(0, 0); RDW(0, 0); MM(0, 1); RDW(0, 1); RDA(0); MM(0, 2); RDW(1, 0); MM(0, 3); RDW(1, 1);
                MM(1, 0); RDW(2, 0); RDW(2, 1); MM(1, 1); RDW(3, 0); MM(1, 2); RDW(3, 1); MM(1, 3); RDA(1);
                MM(2, 0); RDA(2); MM(2, 1); RDA(3); FV(); MM(2, 2); M0SET(); MM(2, 3);
                MM(3, 0); MM(3, 1); LGKM0(); HAVE(); MM(3, 2); FREE(); MM(3, 3);
            } else if constexpr (pw) {
                MM(0, 0); RDW(0, 0); MM(0, 1); RDW(0, 1); MM(0, 2); RDA(0); MM(0, 3); RDW(1, 0);
                MM(1, 0); RDW(1, 1); MM(1, 1); RDA(1); MM(1, 2); RDA(2); MM(1, 3); RDA(3);
                MM(2, 0); MM(2, 1); FV(); MM(2, 2); M0SET(); MM(2, 3);
                MM(3, 0); MM(3, 1); LGKM0(); HAVE(); MM(3, 2); FREE(); MM(3, 3);
            } else {
                MM(0, 0); RDW(0, 0); RDW(0, 1); MM(0, 1); RDA(0); RDA(1); MM(1, 0); RDW(1, 0); RDW(1, 1); MM(1, 1); RDA(2); RDA(3); FV();
                MM(2, 0); M0SET(); MM(2, 1); MM(3, 0); LGKM0(); HAVE(); FREE(); MM(3, 1);
            }
            if constexpr (more) {
                if (__builtin_expect(miss != 0, 0)) wait_zero(FULL_AT(U + 1));
                if constexpr (U == 7) { fbase -= 16; asm volatile("" : "+v"(fbase)); }
            }
        };
        using T_ = std::integral_constant<bool, true>;
        __builtin_amdgcn_s_waitcnt(0x0F70);
        wait_zero(FULL_AT(0));
#pragma unroll 1
        for (int i = 0; i < N1 / 4; ++i) {
            static_for<8>([&](auto uc) { half(uc, T_{}, T_{}, T_{}, T_{}); });
        }
        constexpr int NX = 6, R1 = N1 % 4, X2 = N2 - A2A_X2;
        static_assert(1 <= X2 && X2 <= N2, "exchange point");
        const auto rxb = mkrsrc(myws + WS_XBUF, XBUF_BYTES);
        u32x4 xp[NX];
        unsigned call;
        int tid = threadIdx.x;
        auto fetch = [&]() {
            asm volatile("" ::: "memory");
#pragma unroll
            for (int k = 0; k < NX; ++k) xp[k] = __builtin_amdgcn_raw_buffer_load_b128(rxb, (bid * 4 + wid) * (NX * 1024) + (tid & 63) * 16 + k * 1024, 0, 16);
        };
        static_for<R1 + N2>([&](auto ic) {
            constexpr int s_ = decltype(ic)::value, i = s_ - R1;
            using Sig = std::integral_constant<bool, (i + 4 < N2)>;
            using Wide = std::integral_constant<bool, (i < 0)>;
            half(std::integral_constant<int, (s_ & 3) * 2>{}, std::integral_constant<bool, (i <= 0)>{}, Wide{}, Sig{}, T_{});
            if constexpr (i == 0) {
                MMEND();
                if (wid == 0) STAMP(6);
                asm volatile("" : "+v"(tid));
                call = __builtin_amdgcn_readfirstlane(lds_ld(misc)) + 1;
                unsigned pk[18];
#pragma unroll
                for (int k = 0; k < 16; ++k) pk[k] = pack2(acc[k >> 2][2 + ((k >> 1) & 1)][(k & 1) * 2], acc[k >> 2][2 + ((k >> 1) & 1)][(k & 1) * 2 + 1]);
                pk[16] = pk[17] = 0;
#pragma unroll
                for (int k = 0; k < NX; ++k)
                    __builtin_amdgcn_raw_buffer_store_b128((u32x4){pk[3 * k], pk[3 * k + 1], pk[3 * k + 2], call}, rxb, ((bid ^ 1) * 4 + wid) * (NX * 1024) + (tid & 63) * 16 + k * 1024, 0, 16);
                SB();
            }
            half(std::integral_constant<int, (s_ & 3) * 2 + 1>{}, Wide{}, Wide{}, Sig{}, std::integral_constant<bool, (i + 1 < N2)>{});
            if constexpr (i + 1 == X2) { fetch(); SB(); }
        });
        unsigned vo = (unsigned)(mt * 128 + wm * 64 + (tid & 15)) * 8192u + nt * 256 + h * 128 + wn * 64 + ((tid >> 4) & 1) * 32 + ((tid >> 5) & 1) * 16;
        bool bad = __builtin_amdgcn_readfirstlane(lds_ld(misc + 1)) == 0;
        asm volatile("" : "+v"(vo));
        SB();
#pragma unroll
        for (int rb = 0; rb < 4; ++rb) { constexpr int H = 0; MM(rb, 0); MM(rb, 1); }
        MMEND();
        auto stale = [&]() { return __builtin_amdgcn_ballot_w64(((xp[0][3] ^ call) | (xp[1][3] ^ call) | (xp[2][3] ^ call) | (xp[3][3] ^ call) | (xp[4][3] ^ call) | (xp[5][3] ^ call)) != 0) != 0; };
        asm volatile("" :: "v"(xp[5]));
        if (stale()) {
            GAS unsigned* tmo = (GAS unsigned*)(myws + WS_TMO);
            const unsigned long long t0 = __builtin_amdgcn_s_memrealtime();
            for (unsigned n = 1;; ++n) {
                fetch();
                if (!stale()) break;
                if ((n & 15) == 0 && ((unsigned)__builtin_amdgcn_readfirstlane(__hip_atomic_load(tmo, __ATOMIC_RELAXED, __HIP_MEMORY_SCOPE_AGENT)) == call
                                      || __builtin_amdgcn_s_memrealtime() - t0 > DEADLINE)) {
                    __hip_atomic_store(tmo, call, __ATOMIC_RELAXED, __HIP_MEMORY_SCOPE_AGENT);
                    bad = true; break;
                }
                __builtin_amdgcn_s_sleep(1);
            }
        }
        if (wid == 0) STAMP(7);
        const auto ro = mkrsrc(outp, MLOC * NTOT * 2);
#pragma unroll
        for (int rb = 0; rb < 4; ++rb) {
            unsigned x_[4];
#pragma unroll
            for (int k = 0; k < 4; ++k) x_[k] = xp[(rb * 4 + k) / 3][(rb * 4 + k) % 3];
            const f32x4 a = acc[rb][0] + __builtin_bit_cast(f32x4, (u32x4){x_[0] << 16, x_[0] & 0xffff0000u, x_[1] << 16, x_[1] & 0xffff0000u});
            const f32x4 b = acc[rb][1] + __builtin_bit_cast(f32x4, (u32x4){x_[2] << 16, x_[2] & 0xffff0000u, x_[3] << 16, x_[3] & 0xffff0000u});
            const auto s0 = __builtin_amdgcn_permlane16_swap(pack2(a[0], a[1]), pack2(b[0], b[1]), false, false);
            const auto s1 = __builtin_amdgcn_permlane16_swap(pack2(a[2], a[3]), pack2(b[2], b[3]), false, false);
            u32x4 o = {s0[0], s1[0], s0[1], s1[1]};
            if (bad) o = (u32x4){0x7fc07fc0u, 0x7fc07fc0u, 0x7fc07fc0u, 0x7fc07fc0u};
            __builtin_amdgcn_raw_buffer_store_b128(o, ro, vo + rb * 16 * 8192, 0, 0);
        }
    } else if (wid < 6) {
        __builtin_amdgcn_s_setprio(A2A_LPRIO);
        const int j = wid & 1;
        unsigned full_a = sig_at(SIG_FULL + j, 0);
        const int tab = step_table(lane, rank, mt, h);
#define LD4(rsrc, ldsp, v0, v1, so, stride, aux) do { \
            __builtin_amdgcn_raw_ptr_buffer_load_lds(rsrc, (LAS void*)(ldsp), 16, v0, (so), 0, aux); \
            __builtin_amdgcn_raw_ptr_buffer_load_lds(rsrc, (LAS void*)(ldsp), 16, v1, (so) + ((stride) - 1024), 1024, aux); \
            __builtin_amdgcn_raw_ptr_buffer_load_lds(rsrc, (LAS void*)(ldsp), 16, v0, (so) + 2 * ((stride) - 1024), 2048, aux); \
            __builtin_amdgcn_raw_ptr_buffer_load_lds(rsrc, (LAS void*)(ldsp), 16, v1, (so) + 3 * ((stride) - 1024), 3072, aux); } while (0)
        const auto rw = mkrsrc(p.w, KTOT * NTOT * 2);
        const unsigned vwb = (unsigned)(j * 32 + (lane >> 3)) * 8192u + nt * 256;
        const unsigned vwe = vwb + (((lane & 7) ^ (((lane >> 4) & 1) * 2)) * 16), vwo = vwb + (((lane & 7) ^ ((((lane >> 4) & 1) | 2) * 2)) * 16);
        const unsigned ch = ((lane & 3) ^ aswz(lane >> 4)) * 16;
        const auto rx = mkrsrc(p.x, 4096 * KC * 2);
        const unsigned vx = (unsigned)(rank * 512 + mt * 128 + (lane >> 2)) * 1024u + j * 64 + ch;
        LAS char* const la = lds + j * 8192;
        LAS char* const lw = lds + W_OFF + j * 4096;
        const int hold = bid / 7 < A2A_HOLD ? max(bid / 7 - 1, 0) / 2 : 4;
        START_BARRIER();
#pragma unroll 1
        for (int t = 0; t < 4; ++t) {
            if (t == hold) while (__builtin_amdgcn_readfirstlane(asm_lds_ld(lds0 + MISC_OFF + 8)) != 0) __builtin_amdgcn_s_sleep(1);
            if (t == 3) asm volatile("s_waitcnt vmcnt(40)" ::: "memory");
            const unsigned sw = (rank * 8 + 2 * t + h) * (64u * 8192u) + h * 128;
            LD4(rw, lw + t * SLOT, vwe, vwo, sw, 65536, 0);
            LD4(rw, lw + t * SLOT + 8192, vwe, vwo, sw ^ 128, 65536, 0);
            LD4(rx, la + t * SLOT, vx, vx, (2 * t + h) * 128, 16384, 0);
            LD4(rx, la + t * SLOT + 4096, vx, vx, (2 * t + h) * 128 + 65536, 16384, 0);
            signal(rz, full_a); full_a -= 4;
        }
        int tab4 = tab << ((lane & 3) * 8);
        tab4 |= __builtin_amdgcn_update_dpp(0, tab4, 0xB1, 0xf, 0xf, true);
        tab4 |= __builtin_amdgcn_update_dpp(0, tab4, 0x4E, 0xf, 0xf, true);
        const u32x4 rw4 = {(unsigned)(size_t)p.w, (unsigned)((size_t)p.w >> 32) & 0xffffu, KTOT * NTOT * 2, 0x00020000u};
        const unsigned vw1 = vwo + (65536 - 1024), vw2 = vwe + 2 * (65536 - 1024), vw3 = vwo + 3 * (65536 - 1024);
        const size_t sb = (size_t)(myws + WS_STG) + j * 32768 + mt * 8192;
        const u32x4 rs4 = {(unsigned)sb, ((unsigned)(sb >> 32) & 0xffffu) | (16u << 16), (unsigned)(2 * STG_PAR) - (j * 32768 + mt * 8192), 0x00800000u};
        unsigned c0;
        while ((c0 = __builtin_amdgcn_readfirstlane(asm_lds_ld(lds0 + MISC_OFF))) == ~0u) __builtin_amdgcn_s_sleep(1);
        asm volatile("" : "+s"(c0));
        const unsigned call = c0 + 1, par = (call & 1) * (unsigned)STG_PAR;
        unsigned watch = ((lane & 4) ? lds0 + FLG_OFF + (63 - 4) * 4 : sig_at(SIG_FREE + j * 4 + (lane & 3), 0)) - 12;
        const int thr = (lane & 4) ? (int)call : 0;
        auto probe = [&](auto sc) {
            int v; const unsigned wa = watch;
            asm volatile("ds_read_b32 %0, %1 offset:%2\n\ts_waitcnt lgkmcnt(0)" : "=&v"(v) : "v"(wa), "i"(12 - 4 * decltype(sc)::value) : "memory");
            return (unsigned)__builtin_amdgcn_ballot_w64(v < thr);
        };
#define LW(v, so, o) "buffer_load_dwordx4 " v ", %5, " so " offen" o " lds\n\t"
#define LA(so, o) "buffer_load_dwordx4 off, %2, " so o " sc0 sc1 lds\n\t"
        auto step = [&](auto sc, unsigned tile, bool wide) {
            constexpr int S = decltype(sc)::value;
            unsigned sw = tile * (64u * 8192u) + h * 128, sa = tile * 65536u + par, busy;
            asm volatile("s_waitcnt vmcnt(40)" : "+s"(sw), "+s"(sa) :: "memory");
            while ((busy = probe(sc)) & 15) {
                if (A2A_LSLEEP) __builtin_amdgcn_s_sleep(A2A_LSLEEP);
            }
            const unsigned v0 = vwe, v1 = vw1, v2 = vw2, v3 = vw3, mw = lds0 + W_OFF + j * 4096 + S * SLOT, ma = lds0 + j * 8192 + S * SLOT;
            const u32x4 rw_ = rw4, rs_ = rs4, rz_ = rz;
            asm volatile("s_mov_b32 m0, %6\n\ts_nop 0\n\t" LW("%1", "%0", "") LW("%2", "%0", " offset:1024") LW("%3", "%0", " offset:2048") LW("%4", "%0", " offset:3072")
                         :: "s"(sw), "v"(v0), "v"(v1), "v"(v2), "v"(v3), "s"(rw_), "s"(mw) : "memory");
            if (wide)
                asm volatile("s_add_u32 m0, m0, 0x2000\n\ts_xor_b32 %0, %0, 0x80\n\t" LW("%1", "%0", "") LW("%2", "%0", " offset:1024") LW("%3", "%0", " offset:2048") LW("%4", "%0", " offset:3072")
                             : "+s"(sw) : "v"(v0), "v"(v1), "v"(v2), "v"(v3), "s"(rw_) : "memory", "scc");
            if (__builtin_expect(busy & 16, 0)) while (probe(sc) & 16) __builtin_amdgcn_s_sleep(1);
            unsigned fa = full_a;
            asm volatile(
                         "s_mov_b32 m0, %4\n\ts_nop 0\n\t" LA("%0", "") LA("%0", " offset:1024") LA("%0", " offset:2048") LA("%0", " offset:3072")
                         "s_add_u32 m0, m0, 0x1000\n\ts_add_u32 %0, %0, 0x1000\n\t" LA("%0", "") LA("%0", " offset:1024") LA("%0", " offset:2048") LA("%0", " offset:3072")
                         "s_mov_b32 m0, %1\n\ts_sub_u32 %1, %1, 4\n\tbuffer_load_dword off, %3, 0 lds"
                         : "+s"(sa), "+s"(fa) : "s"(rs_), "s"(rz_), "s"(ma) : "memory", "scc");
            full_a = fa;
        };
#pragma unroll 1
        for (int t = 4; t < NT; t += 4) {
            const unsigned t4 = __builtin_amdgcn_readlane(tab4, t);
            static_for<4>([&](auto sc) {
                constexpr int S = decltype(sc)::value;
                if (S < NT % 4 || NT % 4 == 0 || t + S < NT) step(sc, (t4 >> (8 * S)) & 63, t + S < N1);
            });
            watch -= 16;
        }
    } else if (wid == 6) {
        if (bid < NPUSH) {
            __builtin_amdgcn_s_setprio(3);
            const unsigned tstart = (unsigned)__builtin_amdgcn_s_memrealtime();
            const int slot = bid / 7, di = bid - slot * 7, dest = (rank + 1 + di) & 7, slice = slot >> 2, tm = (slot - ((6 - di) >> 1)) & 3;
            GAS unsigned* cnt = (GAS unsigned*)(myws + WS_CNT + bid * 128);
            const auto rx = mkrsrc(p.x, 4096 * KC * 2);
            const unsigned vx = (unsigned)(dest * 512 + tm * 128 + ((lane >> 2) & 7)) * 1024u + slice * 128 + (lane >> 5) * 64;
            const unsigned vx0 = vx + ((lane & 3) ^ aswz((lane >> 4) & 1)) * 16, vx1 = vx + ((lane & 3) ^ aswz(2 + ((lane >> 4) & 1))) * 16;
            u32x4 v[16];
#pragma unroll
            for (int i = 0; i < 16; ++i) v[i] = __builtin_amdgcn_raw_buffer_load_b128(rx, (i & 1) ? vx1 : vx0, i * 8192, 0);
            const unsigned c0 = __hip_atomic_load(cnt, __ATOMIC_RELAXED, __HIP_MEMORY_SCOPE_AGENT);
            __builtin_amdgcn_s_barrier();
            char* const dws = p.peer[di];
            __builtin_amdgcn_s_waitcnt(0x0F70);
            SB();
            const unsigned call = __builtin_amdgcn_readfirstlane(c0) + 1;
            char* const dst = dws + WS_STG + (call & 1) * STG_PAR + (rank * 8 + slice) * 65536 + (lane >> 5) * 32768 + tm * 8192 + (lane & 31) * 16 + 4096;
            asm volatile("" :: "v"(dst));
            while ((int)((unsigned)__builtin_amdgcn_s_memrealtime() - tstart) < A2A_T0 + slot * A2A_TAU) __builtin_amdgcn_s_sleep(1);
            static_for<16>([&](auto ic) {
                constexpr int i = decltype(ic)::value;
                char* const d = dst; const u32x4 vi = v[i];
                asm volatile("global_store_dwordx4 %0, %1, off offset:%2 sc0 sc1\n\ts_nop 1" :: "v"(d), "v"(vi), "i"(i * 512 - 4096) : "memory");
            });
            if (slot < A2A_HOLD && lane == 0) lds_st(misc + 2, 0u);
            asm volatile("s_waitcnt vmcnt(0)" ::: "memory");
            if (lane == 0) {
                __hip_atomic_store((GAS unsigned*)(dws + WS_FLAG + ((tm * 8 + slice) * 8 + rank) * 128), call, __ATOMIC_RELAXED, __HIP_MEMORY_SCOPE_SYSTEM);
                __hip_atomic_store(cnt, call, __ATOMIC_RELAXED, __HIP_MEMORY_SCOPE_AGENT);
            }
        } else {
            __builtin_amdgcn_s_barrier();
        }
    } else if (wid == 7) {
        __builtin_amdgcn_s_setprio(3);
        const unsigned long long t0 = __builtin_amdgcn_s_memrealtime();
        GAS unsigned* cnt = (GAS unsigned*)(myws + WS_CNT + bid * 128 + 64);
        const auto rc = mkrsrc(myws, WS_STG);
        const unsigned c0 = __hip_atomic_load(cnt, __ATOMIC_RELAXED, __HIP_MEMORY_SCOPE_AGENT);
        const int tab = step_table(lane, rank, mt, h);
        const int st = 63 - lane, rt = __builtin_amdgcn_ds_bpermute(st * 4, tab);
        const unsigned foff = st < NT ? WS_FLAG + ((mt * 8 + (rt & 7)) * 8 + (rt >> 3)) * 128 : WS_TMO;
        const int fslot = (rt & 7) * 4 + ((mt + ((((rt >> 3) - rank - 1) & 7) >> 1)) & 3);
        const unsigned due = (unsigned)t0 + A2A_T0 + (fslot < A2A_HOLD ? A2A_GATE0 : A2A_GATE) + fslot * A2A_TAU;
        START_BARRIER();
        if (lane == 0) __builtin_amdgcn_raw_ptr_buffer_load_lds(rc, (LAS void*)misc, 4, (unsigned)(WS_CNT + bid * 128 + 64), 0, 0, 16);
        asm volatile("s_waitcnt vmcnt(0)" ::: "memory");
        const unsigned call = __builtin_amdgcn_readfirstlane(c0) + 1;
        if (lane == 0) __hip_atomic_store(cnt, call, __ATOMIC_RELAXED, __HIP_MEMORY_SCOPE_AGENT);
        bool bad = false;
        for (unsigned n = 1;; ++n) {
            const unsigned v = asm_lds_ld(lds0 + FLG_OFF + lane * 4);
            const bool here = v >= call || st < 4;
            const int ready = __builtin_clzll(~__builtin_amdgcn_ballot_w64(here));
            const unsigned long long now = __builtin_amdgcn_s_memrealtime();
            bad = __builtin_amdgcn_readfirstlane(v) == call || now - t0 > DEADLINE;
            if (bad || ready >= NT) break;
            if (ready < A2A_AHEAD || __builtin_amdgcn_readfirstlane(asm_lds_ld(sig_at(SIG_FULL, ready - A2A_AHEAD))) == 0) {
                if ((lane == 0 && (n & 15) == 0) || (st < min(NT, ready + A2A_PWIN) && !here && (int)((unsigned)now - due) >= 0))
                    __builtin_amdgcn_raw_ptr_buffer_load_lds(rc, (LAS void*)(lds + FLG_OFF), 4, foff, 0, 0, 17);
                asm volatile("s_waitcnt vmcnt(0)" ::: "memory");
            }
            __builtin_amdgcn_s_sleep(A2A_PSLEEP);
        }
        if (bad) {
            if (lane == 0) { __hip_atomic_store((GAS unsigned*)(myws + WS_TMO), call, __ATOMIC_RELAXED, __HIP_MEMORY_SCOPE_AGENT); lds_st(misc + 1, 0u); }
            lds_st((LAS unsigned*)(lds + FLG_OFF) + lane, call);
        }
    } else {
        __builtin_amdgcn_s_barrier();
        constexpr int ROWS = 16 / A2A_NPF;
        const char* const wq = p.w + (size_t)(mt * 16 + (wid - 8) * ROWS) * 8192 + nt * 256;
        unsigned junk = 0;
#pragma unroll 1
        for (int gi = A2A_G0 + h; gi < 64; gi += gi + 2 < 2 * N1 ? 2 : gi < 2 * N1 ? 2 * N1 - gi : 1) {
            const int u = gi - 8, slice = u / 7, src = (rank + 1 + u - slice * 7) & 7;
            const char* const a = wq + (size_t)(src * 8 + slice) * (64 * 8192);
#define PF2(r) "s_load_dword %0, %1, " #r "*8192\n\ts_load_dword %0, %1, " #r "*8192+64\n\t"
#pragma unroll
            for (int r = 0; r < ROWS; r += 4) {
                asm volatile(PF2(0) PF2(1) PF2(2) PF2(3) "s_nop 0" : "+s"(junk) : "s"(a + r * 8192 + h * 128) : "memory");
                if (gi < 2 * N1) asm volatile(PF2(0) PF2(1) PF2(2) PF2(3) "s_nop 0" : "+s"(junk) : "s"(a + r * 8192 + (h ^ 1) * 128) : "memory");
            }
        }
        asm volatile("s_waitcnt lgkmcnt(0)" : "+s"(junk) :: "memory");
    }
}
}

extern "C" void kernel_launch(int rank, int nranks, ncclComm_t comm,
                              hipStream_t stream,
                              void* const* d_in, const int* in_sizes, int n_in,
                              void* d_out, int out_size,
                              const int* in_dtypes, int out_dtype,
                              void* const* d_out_all,
                              void* const* d_ws_all, size_t ws_bytes) {
    using namespace a2a;
    if (nranks != NR || (unsigned)rank >= (unsigned)NR || n_in < 2 || ws_bytes < WS_NEED ||
        in_sizes[0] != 4096 * KC || in_sizes[1] != KTOT * NTOT || out_size != MLOC * NTOT) {
        fprintf(stderr, "a2a_gemm: unexpected problem shape\n");
        return;
    }
    Params p;
    p.x = (const char*)d_in[0]; p.w = (const char*)d_in[1]; p.out = (char*)d_out;
    p.ws = (char*)d_ws_all[rank];
    for (int i = 0; i < NR - 1; ++i) p.peer[i] = (char*)d_ws_all[(rank + 1 + i) % NR];
    p.rank = rank; p.pad = 0;
    hipLaunchKernelGGL(a2a_gemm, dim3(NWG), dim3(NTHREADS), 0, stream, p);
    HIP_CHECK(hipGetLastError());
}
```

```cpp
#include <hip/hip_runtime.h>
#include <rccl/rccl.h>
#include <cstdio>
#include <cstdint>
#include <type_traits>
#include <utility>

#define HIP_CHECK(expr) do { hipError_t _e = (expr); if (_e != hipSuccess) { \
    fprintf(stderr, "HIP error %s at %s:%d\n", hipGetErrorString(_e), __FILE__, __LINE__); \
    return; } } while (0)

namespace a2a {
#define LAS __attribute__((address_space(3)))
#define GAS __attribute__((address_space(1)))
typedef unsigned u32x4 __attribute__((ext_vector_type(4)));
typedef float f32x4 __attribute__((ext_vector_type(4)));
typedef __bf16 bf16x8 __attribute__((ext_vector_type(8)));
typedef __bf16 bf16x4 __attribute__((ext_vector_type(4)));
typedef __bf16 bf16x2 __attribute__((ext_vector_type(2)));
typedef float f32x2 __attribute__((ext_vector_type(2)));

#define A2A_NPF 4
#define A2A_G0 14
#define A2A_N2 4
#define A2A_X2 1
#define A2A_T0 90
#define A2A_TAU 34
#define A2A_HOLD 9
#define A2A_GATE0 170
#define A2A_GATE 220
#define A2A_AHEAD 6
#define A2A_PSLEEP 4
#define A2A_LATE 12
#define A2A_XS 2
#define A2A_PMAP 3
#define A2A_S2 12
#define A2A_TAU2 37
#define A2A_PAUSE -1
#define A2A_RESUME 0
#define A2A_CPRIO 3
#define A2A_LPRIO 0

constexpr int NR = 8;
constexpr int MLOC = 512, KTOT = 4096, NTOT = 4096, KC = 512;
constexpr int NWG = 256, NWAVE = 8 + A2A_NPF, NTHREADS = NWAVE * 64;
constexpr int N2 = A2A_N2, N1 = (64 - N2) / 2, NT = N1 + N2;
static_assert(N2 % 2 == 0 && N2 >= 2 && N1 >= 8 && NT <= 60, "N2");
constexpr int SLOT = 16384, W_OFF = 4 * SLOT;
constexpr int SIG_OFF = 8 * SLOT;
constexpr int SIG_FULL = 0;
constexpr int SIG_FREE = 4;
constexpr int NSIG = 12;
constexpr int MISC_OFF = SIG_OFF + NSIG * 512;
constexpr int FLG_OFF = MISC_OFF + 64;
constexpr int LDS_BYTES = FLG_OFF + 256;
static_assert(LDS_BYTES <= 163840 && LDS_BYTES > 81920, "one workgroup per CU");
constexpr size_t WS_CNT = 0;
constexpr size_t WS_TMO = 32768;
constexpr size_t WS_FLAG = 65536;
constexpr size_t WS_STG = 1u << 20;
constexpr size_t STG_PAR = 4u << 20;
constexpr size_t WS_XBUF = WS_STG + 2 * STG_PAR;
constexpr size_t XBUF_BYTES = 6u << 20;
constexpr size_t WS_NEED = WS_XBUF + XBUF_BYTES;
constexpr unsigned long long DEADLINE = 50000000ull;
constexpr int NPUSH = 7 * 32;

struct Params {
    const char* x; const char* w; char* out;
    char* ws;
    char* peer[NR - 1];
    int rank; int pad;
};

#define STAMP(i) do {} while (0)

__device__ __forceinline__ __amdgpu_buffer_rsrc_t mkrsrc(const void* ptr, unsigned bytes) {
    return __builtin_amdgcn_make_buffer_rsrc((void*)ptr, 0, bytes, 0x00020000);
}
__device__ __forceinline__ unsigned lds_ld(LAS unsigned* a) { return __hip_atomic_load(a, __ATOMIC_RELAXED, __HIP_MEMORY_SCOPE_WORKGROUP); }
__device__ __forceinline__ void lds_st(LAS unsigned* a, unsigned v) { __hip_atomic_store(a, v, __ATOMIC_RELAXED, __HIP_MEMORY_SCOPE_WORKGROUP); }
__device__ __forceinline__ int aswz(int i) { return ((i << 1) ^ ((i >> 1) * 3)) & 3; }
__device__ __forceinline__ unsigned pack2(float a, float b) { f32x2 v = {a, b}; return __builtin_bit_cast(unsigned, __builtin_convertvector(v, bf16x2)); }
__device__ __forceinline__ unsigned asm_lds_ld(unsigned addr) {
    unsigned v;
    asm volatile("ds_read_b32 %0, %1\n\ts_waitcnt lgkmcnt(0)" : "=&v"(v) : "v"(addr) : "memory");
    return v;
}
template <int N, class F, int... I> __device__ __forceinline__ void static_for_impl(F f, std::integer_sequence<int, I...>) { (f(std::integral_constant<int, I>{}), ...); }
template <int N, class F> __device__ __forceinline__ void static_for(F f) { static_for_impl<N>(f, std::make_integer_sequence<int, N>{}); }
__device__ __forceinline__ int tile_table(int lane, int rank, int mt, int h) {
    const int j = lane >> 3, pi = 2 * ((((lane >> 1) & 3) - mt) & 3) + (lane & 1), gi = 8 + 7 * j + pi;
    const bool wide = pi < 7 && gi < 2 * N1 && (gi & 1) == h, narrow = pi < 7 && gi >= 2 * N1;
    const unsigned long long below = (1ull << lane) - 1, mw = __builtin_amdgcn_ballot_w64(wide), mn = __builtin_amdgcn_ballot_w64(narrow);
    const int pos = wide ? 4 + __builtin_popcountll(mw & below) : narrow ? N1 + __builtin_popcountll(mn & below) : 63;
    return __builtin_amdgcn_ds_permute((63 - pos) * 4, (((rank + 1 + pi) & 7) << 3) | j);
}
constexpr unsigned long long entries(int from, int to, int parity) {
    unsigned long long m = 0;
    for (int t = from; t < to; ++t) if ((t & 1) == parity) m |= 1ull << (63 - t);
    return m;
}
__device__ __forceinline__ unsigned top4(unsigned long long m) { unsigned hi = (unsigned)(m >> 32); asm("" : "+s"(hi)); return hi >> 28; }
__device__ __forceinline__ int departs(int slot) { return A2A_T0 + slot * A2A_TAU + max(slot - A2A_S2, 0) * (A2A_TAU2 - A2A_TAU); }
#define START_BARRIER() do { asm volatile("s_waitcnt lgkmcnt(0)" ::: "memory"); __builtin_amdgcn_s_barrier(); asm volatile("" ::: "memory"); } while (0)
#define SB() __builtin_amdgcn_sched_barrier(0)

__global__ __launch_bounds__(NTHREADS, (NWAVE + 3) / 4) void a2a_gemm(const Params p) {
    __shared__ __attribute__((aligned(16))) char smem[LDS_BYTES];
    LAS char* lds = (LAS char*)smem;
    const unsigned lds0 = (unsigned)(size_t)lds, sig0 = lds0 + SIG_OFF;
    LAS unsigned* misc = (LAS unsigned*)(lds + MISC_OFF);
    auto sig_at = [&](int arr, int e) { return sig0 + arr * 512 + (63 - e) * 4; };
    const int bid = blockIdx.x, lane = threadIdx.x & 63;
    const int wid = __builtin_amdgcn_readfirstlane(threadIdx.x >> 6);
    const int rank = p.rank;
    const int h = bid & 1, mt = (bid >> 3) & 3, nt = (bid >> 5) * 4 + ((bid >> 1) & 3);
    const int pid = (bid & 7) == 3 ? NPUSH : (bid >> 3) * 7 + (bid & 7) - ((bid & 7) > 3);
    char* const myws = p.ws;
    const u32x4 rz = {(unsigned)(size_t)p.x, (unsigned)((size_t)p.x >> 32) & 0xffffu, 0u, 0x00020000u};

    if (wid < 4) {
        if (wid == 0) STAMP(0);
        for (int i = threadIdx.x; i < (NSIG * 512 + 64) / 16; i += 256) *(LAS u32x4*)(lds + SIG_OFF + i * 16) = (u32x4){~0u, ~0u, ~0u, ~0u};
        if (wid == 3) *(LAS unsigned*)(lds + FLG_OFF + lane * 4) = 0u;
        START_BARRIER();
        __builtin_amdgcn_s_setprio(A2A_CPRIO);
        const int wm = wid >> 1, wn = wid & 1, q = (lane >> 2) & 3, g = lane >> 4, r16 = lane & 15, f = (q >> 1) | ((g & 1) << 1);
        const unsigned a0 = lds0 + (wm * 64 + r16) * 64 + ((g ^ aswz(r16 >> 2)) * 16);
        const unsigned w0 = lds0 + W_OFF + (8 * g + q) * 128 + (((wn * 2) ^ f) * 32) + (lane & 3) * 8, w1 = w0 ^ 32;
        unsigned fbase = sig_at(SIG_FULL, 2);
        unsigned free_a = sig_at(SIG_FREE + wid, 0);
        char* outp = p.out;
        asm volatile("" : "+s"(outp));
        auto wait_zero = [&](unsigned a) {
            while (__builtin_amdgcn_ballot_w64(*(volatile LAS unsigned*)(size_t)a != 0) != 0) __builtin_amdgcn_s_sleep(1);
            asm volatile("" ::: "memory");
        };
        bf16x8 af[2][4], wf[2][4];
        bf16x4 wl;
        f32x4 acc[4][4];
        unsigned fv;
#pragma unroll
        for (int i = 0; i < 4; ++i) { af[1][i] = (bf16x8)(__bf16)0.f; wf[1][i] = (bf16x8)(__bf16)0.f; }
#pragma unroll
        for (int i = 0; i < 16; ++i) acc[i >> 2][i & 3] = (f32x4){0.f, 0.f, 0.f, 0.f};
#define FULL_AT(U) (fbase + ((U) & 3) * 512 + (2 - ((U) >> 2)) * 4)
#define FV() do { if constexpr (more) { fv = *(volatile LAS unsigned*)(size_t)FULL_AT(U + 1); SB(); } } while (0)
#define LGKM0() do { __builtin_amdgcn_s_waitcnt(0xC07F); SB(); } while (0)
#define HAVE() do { if constexpr (more) { miss = __builtin_amdgcn_ballot_w64(fv != 0); asm volatile("" : "+s"(miss)); SB(); } } while (0)
#define M0SET() do { if constexpr (sig) { \
            if constexpr (H) asm volatile("s_mov_b32 m0, %0\n\ts_sub_u32 %0, %0, 2052" : "+s"(free_a) :: "memory", "scc"); \
            else asm volatile("s_mov_b32 m0, %0\n\ts_add_u32 %0, %0, 2048" : "+s"(free_a) :: "memory", "scc"); \
            SB(); } } while (0)
#define FREE() do { if constexpr (sig) { const u32x4 rz_ = rz; asm volatile("buffer_load_dword off, %0, 0 lds" :: "s"(rz_) : "memory"); SB(); } } while (0)
#define RDW(cb, t2) do { const bf16x4 w_ = __builtin_amdgcn_ds_read_tr16_b64_v4bf16((LAS bf16x4*)(size_t)(((cb & 1) ? w1 : w0) + S * SLOT + H * 4096 + (cb >> 1) * 8192 + t2 * 512)); \
            if (t2 == 0) wl = w_; else wf[H][cb] = __builtin_shufflevector(wl, w_, 0, 1, 2, 3, 4, 5, 6, 7); SB(); } while (0)
#define RDA(rb) do { af[H][rb] = *(LAS bf16x8*)(size_t)(a0 + S * SLOT + H * 8192 + rb * 1024); SB(); } while (0)
#define MM(rb, cb) do { f32x4& c_ = acc[rb][cb]; const bf16x8 w_ = wf[H ^ 1][cb], a_ = af[H ^ 1][rb]; \
            asm volatile("v_mfma_f32_16x16x32_bf16 %0, %1, %2, %0" : "+v"(c_) : "v"(w_), "v"(a_)); SB(); } while (0)
#define MMEND() do { asm volatile("s_nop 7\n\ts_nop 1" ::: "memory"); SB(); } while (0)
        auto half = [&](auto uc, auto pw, auto wide, auto sig, auto more) {
            constexpr int U = decltype(uc)::value, S = U >> 1, H = U & 1;
            unsigned long long miss = 0;
            if constexpr (wide) {
                MM(0, 0); RDW(0, 0); MM(0, 1); RDW(0, 1); RDA(0); MM(0, 2); RDW(1, 0); MM(0, 3); RDW(1, 1);
                MM(1, 0); RDW(2, 0); RDW(2, 1); MM(1, 1); RDW(3, 0); MM(1, 2); RDW(3, 1); MM(1, 3); RDA(1);
                MM(2, 0); RDA(2); MM(2, 1); RDA(3); FV(); MM(2, 2); M0SET(); MM(2, 3);
                MM(3, 0); MM(3, 1); LGKM0(); HAVE(); MM(3, 2); FREE(); MM(3, 3);
            } else if constexpr (pw) {
                MM(0, 0); RDW(0, 0); MM(0, 1); RDW(0, 1); MM(0, 2); RDA(0); MM(0, 3); RDW(1, 0);
                MM(1, 0); RDW(1, 1); MM(1, 1); RDA(1); MM(1, 2); RDA(2); MM(1, 3); RDA(3);
                MM(2, 0); MM(2, 1); FV(); MM(2, 2); M0SET(); MM(2, 3);
                MM(3, 0); MM(3, 1); LGKM0(); HAVE(); MM(3, 2); FREE(); MM(3, 3);
            } else {
                MM(0, 0); RDW(0, 0); RDW(0, 1); MM(0, 1); RDA(0); RDA(1); MM(1, 0); RDW(1, 0); RDW(1, 1); MM(1, 1); RDA(2); RDA(3); FV();
                MM(2, 0); M0SET(); MM(2, 1); MM(3, 0); LGKM0(); HAVE(); FREE(); MM(3, 1);
            }
            if constexpr (more) {
                if (__builtin_expect(miss != 0, 0)) wait_zero(FULL_AT(U + 1));
                if constexpr (U == 7) { fbase -= 8; asm volatile("" : "+v"(fbase)); }
            }
        };
        using T_ = std::integral_constant<bool, true>;
        __builtin_amdgcn_s_waitcnt(0x0F70);
        wait_zero(FULL_AT(0));
#pragma unroll 1
        for (int i = 0; i < N1 / 4; ++i) {
            static_for<8>([&](auto uc) { half(uc, T_{}, T_{}, T_{}, T_{}); });
        }
        constexpr int NX = 6, R1 = N1 % 4, X2 = N2 - A2A_X2;
        static_assert(1 <= X2 && X2 <= N2, "exchange point");
        const auto rxb = mkrsrc(myws + WS_XBUF, XBUF_BYTES);
        u32x4 xp[NX];
        unsigned call;
        int tid = threadIdx.x;
        auto fetch = [&]() {
            asm volatile("" ::: "memory");
#pragma unroll
            for (int k = 0; k < NX; ++k) xp[k] = __builtin_amdgcn_raw_buffer_load_b128(rxb, (bid * 4 + wid) * (NX * 1024) + (tid & 63) * 16 + k * 1024, 0, 16);
        };
        static_for<R1 + N2>([&](auto ic) {
            constexpr int s_ = decltype(ic)::value, i = s_ - R1;
            using Sig = std::integral_constant<bool, (i + 4 < N2)>;
            using Wide = std::integral_constant<bool, (i < 0)>;
            half(std::integral_constant<int, (s_ & 3) * 2>{}, std::integral_constant<bool, (i <= 0)>{}, Wide{}, Sig{}, T_{});
            if constexpr (i == 0) {
                MMEND();
                if (wid == 0) STAMP(6);
                asm volatile("" : "+v"(tid));
                call = __builtin_amdgcn_readfirstlane(lds_ld(misc)) + 1;
                unsigned pk[18];
#pragma unroll
                for (int k = 0; k < 16; ++k) pk[k] = pack2(acc[k >> 2][2 + ((k >> 1) & 1)][(k & 1) * 2], acc[k >> 2][2 + ((k >> 1) & 1)][(k & 1) * 2 + 1]);
                pk[16] = pk[17] = 0;
#pragma unroll
                for (int k = 0; k < NX; ++k)
                    __builtin_amdgcn_raw_buffer_store_b128((u32x4){pk[3 * k], pk[3 * k + 1], pk[3 * k + 2], call}, rxb, ((bid ^ 1) * 4 + wid) * (NX * 1024) + (tid & 63) * 16 + k * 1024, 0, 16);
                SB();
            }
            half(std::integral_constant<int, (s_ & 3) * 2 + 1>{}, Wide{}, Wide{}, Sig{}, std::integral_constant<bool, (i + 1 < N2)>{});
            if constexpr (i + 1 == X2) { fetch(); SB(); }
        });
        unsigned vo = (unsigned)(mt * 128 + wm * 64 + (tid & 15)) * 8192u + nt * 256 + h * 128 + wn * 64 + ((tid >> 4) & 1) * 32 + ((tid >> 5) & 1) * 16;
        bool bad = __builtin_amdgcn_readfirstlane(lds_ld(misc + 1)) == 0;
        asm volatile("" : "+v"(vo));
        SB();
#pragma unroll
        for (int rb = 0; rb < 4; ++rb) { constexpr int H = 0; MM(rb, 0); MM(rb, 1); }
        MMEND();
        auto stale = [&]() { return __builtin_amdgcn_ballot_w64(((xp[0][3] ^ call) | (xp[1][3] ^ call) | (xp[2][3] ^ call) | (xp[3][3] ^ call) | (xp[4][3] ^ call) | (xp[5][3] ^ call)) != 0) != 0; };
        asm volatile("" :: "v"(xp[5]));
        if (stale()) {
            GAS unsigned* tmo = (GAS unsigned*)(myws + WS_TMO);
            const unsigned long long t0 = __builtin_amdgcn_s_memrealtime();
            for (unsigned n = 1;; ++n) {
                fetch();
                if (!stale()) break;
                if ((n & 15) == 0 && ((unsigned)__builtin_amdgcn_readfirstlane(__hip_atomic_load(tmo, __ATOMIC_RELAXED, __HIP_MEMORY_SCOPE_AGENT)) == call
                                      || __builtin_amdgcn_s_memrealtime() - t0 > DEADLINE)) {
                    __hip_atomic_store(tmo, call, __ATOMIC_RELAXED, __HIP_MEMORY_SCOPE_AGENT);
                    bad = true; break;
                }
                __builtin_amdgcn_s_sleep(1);
            }
        }
        if (wid == 0) STAMP(7);
        const auto ro = mkrsrc(outp, MLOC * NTOT * 2);
#pragma unroll
        for (int rb = 0; rb < 4; ++rb) {
            unsigned x_[4];
#pragma unroll
            for (int k = 0; k < 4; ++k) x_[k] = xp[(rb * 4 + k) / 3][(rb * 4 + k) % 3];
            const f32x4 a = acc[rb][0] + __builtin_bit_cast(f32x4, (u32x4){x_[0] << 16, x_[0] & 0xffff0000u, x_[1] << 16, x_[1] & 0xffff0000u});
            const f32x4 b = acc[rb][1] + __builtin_bit_cast(f32x4, (u32x4){x_[2] << 16, x_[2] & 0xffff0000u, x_[3] << 16, x_[3] & 0xffff0000u});
            const auto s0 = __builtin_amdgcn_permlane16_swap(pack2(a[0], a[1]), pack2(b[0], b[1]), false, false);
            const auto s1 = __builtin_amdgcn_permlane16_swap(pack2(a[2], a[3]), pack2(b[2], b[3]), false, false);
            u32x4 o = {s0[0], s1[0], s0[1], s1[1]};
            if (bad) o = (u32x4){0x7fc07fc0u, 0x7fc07fc0u, 0x7fc07fc0u, 0x7fc07fc0u};
            __builtin_amdgcn_raw_buffer_store_b128(o, ro, vo + rb * 16 * 8192, 0, 0);
        }
    } else if (wid < 6) {
        __builtin_amdgcn_s_setprio(A2A_LPRIO);
        const int j = wid & 1;
        unsigned full_a = sig_at(SIG_FULL + j * 2, 2);
        const int tab = tile_table(lane, rank, mt, h);
        const u32x4 rw4 = {(unsigned)(size_t)p.w, (unsigned)((size_t)p.w >> 32) & 0xffffu, KTOT * NTOT * 2, 0x00020000u};
        const unsigned vwb = (unsigned)(lane >> 3) * 8192u + nt * 256;
        const unsigned vw0 = vwb + (((lane & 7) ^ (((lane >> 4) & 1) * 2)) * 16), vw1 = vwb + (((lane & 7) ^ ((((lane >> 4) & 1) | 2) * 2)) * 16) + (65536 - 1024);
        const unsigned vw2 = vw0 + 2 * (65536 - 1024), vw3 = vw1 + 2 * (65536 - 1024);
#define LV4(aux) "buffer_load_dwordx4 %1, %5, %0 offen" aux " lds\n\tbuffer_load_dwordx4 %2, %5, %0 offen offset:1024" aux " lds\n\t" \
                 "buffer_load_dwordx4 %3, %5, %0 offen offset:2048" aux " lds\n\tbuffer_load_dwordx4 %4, %5, %0 offen offset:3072" aux " lds\n\t"
        auto load_w = [&](unsigned so, unsigned m, bool wide) {
            asm volatile("s_mov_b32 m0, %6\n\ts_nop 0\n\t" LV4("") :: "s"(so), "v"(vw0), "v"(vw1), "v"(vw2), "v"(vw3), "s"(rw4), "s"(m) : "memory");
            if (wide) asm volatile("s_add_u32 m0, m0, 0x2000\n\ts_xor_b32 %0, %0, 0x80\n\t" LV4("") : "+s"(so) : "v"(vw0), "v"(vw1), "v"(vw2), "v"(vw3), "s"(rw4) : "memory", "scc");
        };
        auto landed = [&](unsigned a) {
            asm volatile("s_mov_b32 m0, %0\n\ts_mov_b64 exec, 1\n\tbuffer_load_dword off, %1, 0 lds\n\ts_mov_b64 exec, -1" :: "s"(a), "s"(rz) : "memory");
        };
        {
            const u32x4 rx4 = {rz[0], rz[1], 4096 * KC * 2, 0x00020000u};
            const unsigned vx0 = (unsigned)(rank * 512 + mt * 128 + (lane >> 2)) * 1024u + ((lane & 3) ^ aswz(lane >> 4)) * 16;
            const unsigned vx1 = vx0 + (16384 - 1024), vx2 = vx0 + 2 * (16384 - 1024), vx3 = vx0 + 3 * (16384 - 1024);
            const int hold = pid / 7 < A2A_HOLD ? max(pid / 7 - 1, 0) / 2 : 4;
            START_BARRIER();
#pragma unroll 1
            for (int t = 0; t < 4; ++t) {
                if (t == hold) while (__builtin_amdgcn_readfirstlane(asm_lds_ld(lds0 + MISC_OFF + 8)) != 0) __builtin_amdgcn_s_sleep(1);
                asm volatile("s_waitcnt vmcnt(40)" ::: "memory");
                load_w((rank * 8 + 2 * t + h) * (64u * 8192u) + j * (32u * 8192u) + h * 128, lds0 + W_OFF + t * SLOT + j * 4096, true);
                unsigned so = (2 * t + h) * 128 + j * 64;
                asm volatile("s_mov_b32 m0, %6\n\ts_nop 0\n\t" LV4("") "s_add_u32 m0, m0, 0x1000\n\ts_add_u32 %0, %0, 0x10000\n\t" LV4("")
                             : "+s"(so) : "v"(vx0), "v"(vx1), "v"(vx2), "v"(vx3), "s"(rx4), "s"(lds0 + t * SLOT + j * 8192) : "memory", "scc");
                landed(sig_at(SIG_FULL + (t & 1) * 2 + j, t >> 1));
            }
        }
        const size_t sb = (size_t)(myws + WS_STG) + mt * 8192;
        const u32x4 rs4 = {(unsigned)sb, ((unsigned)(sb >> 32) & 0xffffu) | (16u << 16), (unsigned)(2 * STG_PAR) - mt * 8192, 0x00800000u};
        unsigned c0;
        while ((c0 = __builtin_amdgcn_readfirstlane(asm_lds_ld(lds0 + MISC_OFF))) == ~0u) __builtin_amdgcn_s_sleep(1);
        asm volatile("" : "+s"(c0));
        const unsigned call = c0 + 1, par = (call & 1) * (unsigned)STG_PAR;
        unsigned watch = lane < 4 ? lds0 + MISC_OFF + lane * 4 : lane < 60 ? lds0 + FLG_OFF + lane * 4 : sig_at(SIG_FREE + (lane & 3), j);
        const int thr = lane >= 4 && lane < 60 ? (int)call : 0;
        const unsigned d1 = lane < 60 ? 0 : 2048, d2 = lane < 60 ? 0 : -2056;
        auto go = [&]() {
            unsigned long long busy;
            do busy = __builtin_amdgcn_ballot_w64((int)asm_lds_ld(watch) < thr); while (top4(busy) != 0 || ((unsigned)busy & 12) == 4);
            return busy;
        };
        unsigned long long left = j ? entries(4, N1, 1) : entries(4, N1, 0);
#define LA4 "buffer_load_dwordx4 off, %1, %0 sc0 sc1 lds\n\tbuffer_load_dwordx4 off, %1, %0 offset:1024 sc0 sc1 lds\n\t" \
            "buffer_load_dwordx4 off, %1, %0 offset:2048 sc0 sc1 lds\n\tbuffer_load_dwordx4 off, %1, %0 offset:3072 sc0 sc1 lds\n\t"
        auto load_a = [&](unsigned so, unsigned m) {
            asm volatile("s_mov_b32 m0, %2\n\ts_nop 0\n\t" LA4 "s_add_u32 m0, m0, 0x1000\n\ts_add_u32 %0, %0, 0x1000\n\t" LA4
                         : "+s"(so) : "s"(rs4), "s"(m) : "memory", "scc");
        };
#pragma unroll 1
        for (int t = 4 + j; t < NT; t += 2) {
            if (left == 0) left = j ? entries(N1, NT, 1) : entries(N1, NT, 0);
            const unsigned slot = lds0 + (t & 3) * SLOT;
            asm volatile("s_waitcnt vmcnt(40)" ::: "memory");
            unsigned long long busy = go();
            int pick = 63 - __builtin_clzll((left & ~busy) != 0 ? left & ~busy : left);
            unsigned tile = __builtin_amdgcn_readlane(tab, pick);
            load_w(tile * (64u * 8192u) + h * 128, slot + W_OFF, t < N1);
            if (__builtin_expect((left & ~busy) == 0, 0)) {
                do { __builtin_amdgcn_s_sleep(1); busy = __builtin_amdgcn_ballot_w64((int)asm_lds_ld(watch) < thr); } while ((left & ~busy) == 0);
                if ((busy >> pick) & 1) {
                    pick = 63 - __builtin_clzll(left & ~busy);
                    tile = __builtin_amdgcn_readlane(tab, pick);
                    asm volatile("s_waitcnt vmcnt(40)" ::: "memory");
                    load_w(tile * (64u * 8192u) + h * 128, slot + W_OFF, t < N1);
                }
            }
            left &= ~(1ull << pick);
            const unsigned sw = tile * (64u * 8192u) + h * 128, sa = tile * 65536u + par;
            load_a(sa, slot);
            landed(full_a);
            asm volatile("s_waitcnt vmcnt(40)" ::: "memory");
            watch += d1;
            go();
            watch += d2;
            load_w(sw + 32u * 8192u, slot + W_OFF + 4096, t < N1);
            load_a(sa + 32768, slot + 8192);
            landed(full_a + 512);
            full_a -= 4;
        }
    } else if (wid == 6) {
        if (pid < NPUSH) {
            __builtin_amdgcn_s_setprio(3);
            const unsigned tstart = (unsigned)__builtin_amdgcn_s_memrealtime();
            const int slot = pid / 7, di = pid - slot * 7, dest = (rank + 1 + di) & 7, slice = slot >> 2, tm = (slot - ((6 - di) >> 1)) & 3;
            GAS unsigned* cnt = (GAS unsigned*)(myws + WS_CNT + bid * 128);
            const auto rx = mkrsrc(p.x, 4096 * KC * 2);
            const unsigned vx = (unsigned)(dest * 512 + tm * 128 + ((lane >> 2) & 7)) * 1024u + slice * 128 + (lane >> 5) * 64;
            const unsigned vx0 = vx + ((lane & 3) ^ aswz((lane >> 4) & 1)) * 16, vx1 = vx + ((lane & 3) ^ aswz(2 + ((lane >> 4) & 1))) * 16;
            if (slot >= A2A_LATE) {
                __builtin_amdgcn_s_barrier();
                while (__builtin_amdgcn_readfirstlane(asm_lds_ld(sig_at(SIG_FULL + (A2A_XS & 1) * 2, A2A_XS >> 1))) != 0) __builtin_amdgcn_s_sleep(2);
            }
            u32x4 v[16];
#pragma unroll
            for (int i = 0; i < 16; ++i) v[i] = __builtin_amdgcn_raw_buffer_load_b128(rx, (i & 1) ? vx1 : vx0, i * 8192, 0);
            const unsigned c0 = __hip_atomic_load(cnt, __ATOMIC_RELAXED, __HIP_MEMORY_SCOPE_AGENT);
            if (slot < A2A_LATE) __builtin_amdgcn_s_barrier();
            char* const dws = p.peer[di];
            __builtin_amdgcn_s_waitcnt(0x0F70);
            SB();
            const unsigned call = __builtin_amdgcn_readfirstlane(c0) + 1;
            char* const dst = dws + WS_STG + (call & 1) * STG_PAR + (rank * 8 + slice) * 65536 + (lane >> 5) * 32768 + tm * 8192 + (lane & 31) * 16 + 4096;
            asm volatile("" :: "v"(dst));
            if (A2A_PAUSE >= 0 && slot >= A2A_HOLD) {
                for (int left; (left = departs(slot) - A2A_PAUSE - (int)((unsigned)__builtin_amdgcn_s_memrealtime() - tstart)) > 0;) {
                    if (left > 80) __builtin_amdgcn_s_sleep(16); else __builtin_amdgcn_s_sleep(1);
                }
                asm volatile("s_mov_b32 m0, %0\n\ts_mov_b64 exec, 1\n\tbuffer_load_dword off, %1, 0 lds\n\ts_mov_b64 exec, -1" :: "s"(lds0 + MISC_OFF + 12), "s"(rz) : "memory");
            }
            for (int left; (left = departs(slot) - (int)((unsigned)__builtin_amdgcn_s_memrealtime() - tstart)) > 0;) {
                if (left > 80) __builtin_amdgcn_s_sleep(16); else __builtin_amdgcn_s_sleep(1);
            }
            static_for<16>([&](auto ic) {
                constexpr int i = decltype(ic)::value;
                char* const d = dst; const u32x4 vi = v[i];
                asm volatile("global_store_dwordx4 %0, %1, off offset:%2 sc0 sc1\n\ts_nop 1" :: "v"(d), "v"(vi), "i"(i * 512 - 4096) : "memory");
            });
            if ((slot < A2A_HOLD || (A2A_PAUSE >= 0 && A2A_RESUME == 0)) && lane == 0) lds_st(misc + 2, 0u);
            asm volatile("s_waitcnt vmcnt(0)" ::: "memory");
            if (lane == 0) {
                __hip_atomic_store((GAS unsigned*)(dws + WS_FLAG + ((tm * 8 + slice) * 8 + rank) * 128), call, __ATOMIC_RELAXED, __HIP_MEMORY_SCOPE_SYSTEM);
                if (A2A_PAUSE >= 0 && A2A_RESUME == 1 && slot >= A2A_HOLD) lds_st(misc + 2, 0u);
                __hip_atomic_store(cnt, call, __ATOMIC_RELAXED, __HIP_MEMORY_SCOPE_AGENT);
            }
        } else {
            __builtin_amdgcn_s_barrier();
        }
    } else if (wid == 7) {
        __builtin_amdgcn_s_setprio(3);
        const unsigned long long t0 = __builtin_amdgcn_s_memrealtime();
        GAS unsigned* cnt = (GAS unsigned*)(myws + WS_CNT + bid * 128 + 64);
        const auto rc = mkrsrc(myws, WS_STG);
        const unsigned c0 = __hip_atomic_load(cnt, __ATOMIC_RELAXED, __HIP_MEMORY_SCOPE_AGENT);
        const int st = 63 - lane, rt = tile_table(lane, rank, mt, h);
        const unsigned foff = st < NT ? WS_FLAG + ((mt * 8 + (rt & 7)) * 8 + (rt >> 3)) * 128 : WS_TMO;
        const int fslot = (rt & 7) * 4 + ((mt + ((((rt >> 3) - rank - 1) & 7) >> 1)) & 3);
        const unsigned due = (unsigned)t0 + departs(fslot) + (fslot < A2A_HOLD ? A2A_GATE0 : A2A_GATE);
        START_BARRIER();
        if (lane == 0) __builtin_amdgcn_raw_ptr_buffer_load_lds(rc, (LAS void*)misc, 4, (unsigned)(WS_CNT + bid * 128 + 64), 0, 0, 16);
        asm volatile("s_waitcnt vmcnt(0)" ::: "memory");
        const unsigned call = __builtin_amdgcn_readfirstlane(c0) + 1;
        if (lane == 0) __hip_atomic_store(cnt, call, __ATOMIC_RELAXED, __HIP_MEMORY_SCOPE_AGENT);
        bool bad = false;
        for (unsigned n = 1;; ++n) {
            const unsigned v = asm_lds_ld(lds0 + FLG_OFF + lane * 4);
            const bool here = v >= call || st < 4 || st >= NT;
            const unsigned long long seen = __builtin_amdgcn_ballot_w64(here), now = __builtin_amdgcn_s_memrealtime();
            bad = __builtin_amdgcn_readfirstlane(v) == call || now - t0 > DEADLINE;
            if (bad || seen == ~0ull) break;
            const int have = min(__builtin_popcountll(seen & entries(0, N1, 0)), __builtin_popcountll(seen & entries(0, N1, 1))) - A2A_AHEAD / 2;
            if (have < 0 || __builtin_amdgcn_readfirstlane(asm_lds_ld(sig_at(SIG_FULL, have))) == 0) {
                if ((lane == 0 && (n & 15) == 0) || (!here && (int)((unsigned)now - due) >= 0))
                    __builtin_amdgcn_raw_ptr_buffer_load_lds(rc, (LAS void*)(lds + FLG_OFF), 4, foff, 0, 0, 17);
                asm volatile("s_waitcnt vmcnt(0)" ::: "memory");
            }
            __builtin_amdgcn_s_sleep(A2A_PSLEEP);
        }
        if (bad) {
            if (lane == 0) { __hip_atomic_store((GAS unsigned*)(myws + WS_TMO), call, __ATOMIC_RELAXED, __HIP_MEMORY_SCOPE_AGENT); lds_st(misc + 1, 0u); }
            lds_st((LAS unsigned*)(lds + FLG_OFF) + lane, call);
        }
    } else {
        __builtin_amdgcn_s_barrier();
        constexpr int ROWS = 16 / A2A_NPF;
        const char* const wq = p.w + (size_t)(mt * 16 + (wid - 8) * ROWS) * 8192 + nt * 256;
        unsigned junk = 0;
#pragma unroll 1
        for (int gi = A2A_G0 + h; gi < 64; gi += gi + 2 < 2 * N1 ? 2 : gi < 2 * N1 ? 2 * N1 - gi : 1) {
            const int u = gi - 8, slice = u / 7, src = (rank + 1 + u - slice * 7) & 7;
            const char* const a = wq + (size_t)(src * 8 + slice) * (64 * 8192);
#define PF2(r) "s_load_dword %0, %1, " #r "*8192\n\ts_load_dword %0, %1, " #r "*8192+64\n\t"
#pragma unroll
            for (int r = 0; r < ROWS; r += 4) {
                asm volatile(PF2(0) PF2(1) PF2(2) PF2(3) "s_nop 0" : "+s"(junk) : "s"(a + r * 8192 + h * 128) : "memory");
                if (gi < 2 * N1) asm volatile(PF2(0) PF2(1) PF2(2) PF2(3) "s_nop 0" : "+s"(junk) : "s"(a + r * 8192 + (h ^ 1) * 128) : "memory");
            }
        }
        asm volatile("s_waitcnt lgkmcnt(0)" : "+s"(junk) :: "memory");
    }
}
}

extern "C" void kernel_launch(int rank, int nranks, ncclComm_t comm,
                              hipStream_t stream,
                              void* const* d_in, const int* in_sizes, int n_in,
                              void* d_out, int out_size,
                              const int* in_dtypes, int out_dtype,
                              void* const* d_out_all,
                              void* const* d_ws_all, size_t ws_bytes) {
    using namespace a2a;
    if (nranks != NR || (unsigned)rank >= (unsigned)NR || n_in < 2 || ws_bytes < WS_NEED ||
        in_sizes[0] != 4096 * KC || in_sizes[1] != KTOT * NTOT || out_size != MLOC * NTOT) {
        fprintf(stderr, "a2a_gemm: unexpected problem shape\n");
        return;
    }
    Params p;
    p.x = (const char*)d_in[0]; p.w = (const char*)d_in[1]; p.out = (char*)d_out;
    p.ws = (char*)d_ws_all[rank];
    for (int i = 0; i < NR - 1; ++i) p.peer[i] = (char*)d_ws_all[(rank + 1 + i) % NR];
    p.rank = rank; p.pad = 0;
    hipLaunchKernelGGL(a2a_gemm, dim3(NWG), dim3(NTHREADS), 0, stream, p);
    HIP_CHECK(hipGetLastError());
}
```

```cpp
#include <hip/hip_runtime.h>
#include <rccl/rccl.h>
#include <cstdio>
#include <cstdint>
#include <type_traits>
#include <utility>

#define HIP_CHECK(expr) do { hipError_t _e = (expr); if (_e != hipSuccess) { \
    fprintf(stderr, "HIP error %s at %s:%d\n", hipGetErrorString(_e), __FILE__, __LINE__); \
    return; } } while (0)

namespace a2a {
#define LAS __attribute__((address_space(3)))
#define GAS __attribute__((address_space(1)))
typedef unsigned u32x4 __attribute__((ext_vector_type(4)));
typedef float f32x4 __attribute__((ext_vector_type(4)));
typedef __bf16 bf16x8 __attribute__((ext_vector_type(8)));
typedef __bf16 bf16x4 __attribute__((ext_vector_type(4)));
typedef __bf16 bf16x2 __attribute__((ext_vector_type(2)));
typedef float f32x2 __attribute__((ext_vector_type(2)));

constexpr int N2 = 4;
constexpr int T0 = 90;
constexpr int TAU = 34, TAU2 = 37;
constexpr int S2 = 12;
constexpr int HOLD = 9;
constexpr int LATE = 12;
constexpr int GATE0 = 170, GATE = 220;
constexpr int G0 = 14;

constexpr int NR = 8;
constexpr int MLOC = 512, KTOT = 4096, NTOT = 4096, KC = 512;
constexpr int NWG = 256, NTHREADS = 12 * 64;
constexpr int N1 = (64 - N2) / 2, NT = N1 + N2;
static_assert(N2 % 2 == 0 && N2 >= 2 && N1 >= 8 && NT <= 60, "N2");
constexpr int SLOT = 16384, W_OFF = 4 * SLOT;
constexpr int SIG_OFF = 8 * SLOT;
constexpr int SIG_FULL = 0;
constexpr int SIG_FREE = 4;
constexpr int NSIG = 12;
constexpr int MISC_OFF = SIG_OFF + NSIG * 512;
constexpr int FLG_OFF = MISC_OFF + 64;
constexpr int LDS_BYTES = FLG_OFF + 256;
static_assert(LDS_BYTES <= 163840 && LDS_BYTES > 81920, "one workgroup per CU");
constexpr size_t WS_CNT = 0;
constexpr size_t WS_TMO = 32768;
constexpr size_t WS_FLAG = 65536;
constexpr size_t WS_STG = 1u << 20;
constexpr size_t STG_PAR = 4u << 20;
constexpr size_t WS_XBUF = WS_STG + 2 * STG_PAR;
constexpr size_t XBUF_BYTES = 6u << 20;
constexpr size_t WS_NEED = WS_XBUF + XBUF_BYTES;
constexpr unsigned long long DEADLINE = 50000000ull;

struct Params {
    const char* x; const char* w; char* out;
    char* ws;
    char* peer[NR - 1];
    int rank; int pad;
};

__device__ __forceinline__ __amdgpu_buffer_rsrc_t mkrsrc(const void* ptr, unsigned bytes) {
    return __builtin_amdgcn_make_buffer_rsrc((void*)ptr, 0, bytes, 0x00020000);
}
__device__ __forceinline__ unsigned lds_ld(LAS unsigned* a) { return __hip_atomic_load(a, __ATOMIC_RELAXED, __HIP_MEMORY_SCOPE_WORKGROUP); }
__device__ __forceinline__ void lds_st(LAS unsigned* a, unsigned v) { __hip_atomic_store(a, v, __ATOMIC_RELAXED, __HIP_MEMORY_SCOPE_WORKGROUP); }
__device__ __forceinline__ int aswz(int i) { return ((i << 1) ^ ((i >> 1) * 3)) & 3; }
__device__ __forceinline__ unsigned pack2(float a, float b) { f32x2 v = {a, b}; return __builtin_bit_cast(unsigned, __builtin_convertvector(v, bf16x2)); }
__device__ __forceinline__ unsigned asm_lds_ld(unsigned addr) {
    unsigned v;
    asm volatile("ds_read_b32 %0, %1\n\ts_waitcnt lgkmcnt(0)" : "=&v"(v) : "v"(addr) : "memory");
    return v;
}
template <int N, class F, int... I> __device__ __forceinline__ void static_for_impl(F f, std::integer_sequence<int, I...>) { (f(std::integral_constant<int, I>{}), ...); }
template <int N, class F> __device__ __forceinline__ void static_for(F f) { static_for_impl<N>(f, std::make_integer_sequence<int, N>{}); }
__device__ __forceinline__ int tile_table(int lane, int rank, int mt, int h) {
    const int j = lane >> 3, pi = 2 * ((((lane >> 1) & 3) - mt) & 3) + (lane & 1), gi = 8 + 7 * j + pi;
    const bool wide = pi < 7 && gi < 2 * N1 && (gi & 1) == h, narrow = pi < 7 && gi >= 2 * N1;
    const unsigned long long below = (1ull << lane) - 1, mw = __builtin_amdgcn_ballot_w64(wide), mn = __builtin_amdgcn_ballot_w64(narrow);
    const int pos = wide ? 4 + __builtin_popcountll(mw & below) : narrow ? N1 + __builtin_popcountll(mn & below) : 63;
    return __builtin_amdgcn_ds_permute((63 - pos) * 4, (((rank + 1 + pi) & 7) << 3) | j);
}
constexpr unsigned long long entries(int from, int to, int parity) {
    unsigned long long m = 0;
    for (int t = from; t < to; ++t) if ((t & 1) == parity) m |= 1ull << (63 - t);
    return m;
}
__device__ __forceinline__ unsigned top4(unsigned long long m) { unsigned hi = (unsigned)(m >> 32); asm("" : "+s"(hi)); return hi >> 28; }
__device__ __forceinline__ int departs(int slot) { return T0 + slot * TAU + max(slot - S2, 0) * (TAU2 - TAU); }
#define START_BARRIER() do { asm volatile("s_waitcnt lgkmcnt(0)" ::: "memory"); __builtin_amdgcn_s_barrier(); asm volatile("" ::: "memory"); } while (0)
#define SB() __builtin_amdgcn_sched_barrier(0)

__global__ __launch_bounds__(NTHREADS, 3) void a2a_gemm(const Params p) {
    __shared__ __attribute__((aligned(16))) char smem[LDS_BYTES];
    LAS char* lds = (LAS char*)smem;
    const unsigned lds0 = (unsigned)(size_t)lds, sig0 = lds0 + SIG_OFF;
    LAS unsigned* misc = (LAS unsigned*)(lds + MISC_OFF);
    auto sig_at = [&](int arr, int e) { return sig0 + arr * 512 + (63 - e) * 4; };
    const int bid = blockIdx.x, lane = threadIdx.x & 63;
    const int wid = __builtin_amdgcn_readfirstlane(threadIdx.x >> 6);
    const int rank = p.rank;
    const int h = bid & 1, mt = (bid >> 3) & 3, nt = (bid >> 5) * 4 + ((bid >> 1) & 3);
    const int slot = (bid & 7) == 3 ? 32 : bid >> 3, di = (bid & 7) - ((bid & 7) > 3);
    char* const myws = p.ws;
    const u32x4 rz = {(unsigned)(size_t)p.x, (unsigned)((size_t)p.x >> 32) & 0xffffu, 0u, 0x00020000u};

    if (wid < 4) {
        for (int i = threadIdx.x; i < (NSIG * 512 + 64) / 16; i += 256) *(LAS u32x4*)(lds + SIG_OFF + i * 16) = (u32x4){~0u, ~0u, ~0u, ~0u};
        if (wid == 3) *(LAS unsigned*)(lds + FLG_OFF + lane * 4) = 0u;
        START_BARRIER();
        __builtin_amdgcn_s_setprio(3);
        const int wm = wid >> 1, wn = wid & 1, q = (lane >> 2) & 3, g = lane >> 4, r16 = lane & 15, f = (q >> 1) | ((g & 1) << 1);
        const unsigned a0 = lds0 + (wm * 64 + r16) * 64 + ((g ^ aswz(r16 >> 2)) * 16);
        const unsigned w0 = lds0 + W_OFF + (8 * g + q) * 128 + (((wn * 2) ^ f) * 32) + (lane & 3) * 8, w1 = w0 ^ 32;
        unsigned fbase = sig_at(SIG_FULL, 2);
        unsigned free_a = sig_at(SIG_FREE + wid, 0);
        char* outp = p.out;
        asm volatile("" : "+s"(outp));
        auto wait_zero = [&](unsigned a) {
            while (__builtin_amdgcn_ballot_w64(*(volatile LAS unsigned*)(size_t)a != 0) != 0) __builtin_amdgcn_s_sleep(1);
            asm volatile("" ::: "memory");
        };
        bf16x8 af[2][4], wf[2][4];
        bf16x4 wl;
        f32x4 acc[4][4];
        unsigned fv;
#pragma unroll
        for (int i = 0; i < 4; ++i) { af[1][i] = (bf16x8)(__bf16)0.f; wf[1][i] = (bf16x8)(__bf16)0.f; }
#pragma unroll
        for (int i = 0; i < 16; ++i) acc[i >> 2][i & 3] = (f32x4){0.f, 0.f, 0.f, 0.f};
#define FULL_AT(U) (fbase + ((U) & 3) * 512 + (2 - ((U) >> 2)) * 4)
#define FV() do { if constexpr (more) { fv = *(volatile LAS unsigned*)(size_t)FULL_AT(U + 1); SB(); } } while (0)
#define LGKM0() do { __builtin_amdgcn_s_waitcnt(0xC07F); SB(); } while (0)
#define HAVE() do { if constexpr (more) { miss = __builtin_amdgcn_ballot_w64(fv != 0); asm volatile("" : "+s"(miss)); SB(); } } while (0)
#define M0SET() do { if constexpr (sig) { \
            if constexpr (H) asm volatile("s_mov_b32 m0, %0\n\ts_sub_u32 %0, %0, 2052" : "+s"(free_a) :: "memory", "scc"); \
            else asm volatile("s_mov_b32 m0, %0\n\ts_add_u32 %0, %0, 2048" : "+s"(free_a) :: "memory", "scc"); \
            SB(); } } while (0)
#define FREE() do { if constexpr (sig) { const u32x4 rz_ = rz; asm volatile("buffer_load_dword off, %0, 0 lds" :: "s"(rz_) : "memory"); SB(); } } while (0)
#define RDW(cb, t2) do { const bf16x4 w_ = __builtin_amdgcn_ds_read_tr16_b64_v4bf16((LAS bf16x4*)(size_t)(((cb & 1) ? w1 : w0) + S * SLOT + H * 4096 + (cb >> 1) * 8192 + t2 * 512)); \
            if (t2 == 0) wl = w_; else wf[H][cb] = __builtin_shufflevector(wl, w_, 0, 1, 2, 3, 4, 5, 6, 7); SB(); } while (0)
#define RDA(rb) do { af[H][rb] = *(LAS bf16x8*)(size_t)(a0 + S * SLOT + H * 8192 + rb * 1024); SB(); } while (0)
#define MM(rb, cb) do { f32x4& c_ = acc[rb][cb]; const bf16x8 w_ = wf[H ^ 1][cb], a_ = af[H ^ 1][rb]; \
            asm volatile("v_mfma_f32_16x16x32_bf16 %0, %1, %2, %0" : "+v"(c_) : "v"(w_), "v"(a_)); SB(); } while (0)
#define MMEND() do { asm volatile("s_nop 7\n\ts_nop 1" ::: "memory"); SB(); } while (0)
        auto half = [&](auto uc, auto pw, auto wide, auto sig, auto more) {
            constexpr int U = decltype(uc)::value, S = U >> 1, H = U & 1;
            unsigned long long miss = 0;
            if constexpr (wide) {
                MM(0, 0); RDW(0, 0); MM(0, 1); RDW(0, 1); RDA(0); MM(0, 2); RDW(1, 0); MM(0, 3); RDW(1, 1);
                MM(1, 0); RDW(2, 0); RDW(2, 1); MM(1, 1); RDW(3, 0); MM(1, 2); RDW(3, 1); MM(1, 3); RDA(1);
                MM(2, 0); RDA(2); MM(2, 1); RDA(3); FV(); MM(2, 2); M0SET(); MM(2, 3);
                MM(3, 0); MM(3, 1); LGKM0(); HAVE(); MM(3, 2); FREE(); MM(3, 3);
            } else if constexpr (pw) {
                MM(0, 0); RDW(0, 0); MM(0, 1); RDW(0, 1); MM(0, 2); RDA(0); MM(0, 3); RDW(1, 0);
                MM(1, 0); RDW(1, 1); MM(1, 1); RDA(1); MM(1, 2); RDA(2); MM(1, 3); RDA(3);
                MM(2, 0); MM(2, 1); FV(); MM(2, 2); M0SET(); MM(2, 3);
                MM(3, 0); MM(3, 1); LGKM0(); HAVE(); MM(3, 2); FREE(); MM(3, 3);
            } else {
                MM(0, 0); RDW(0, 0); RDW(0, 1); MM(0, 1); RDA(0); RDA(1); MM(1, 0); RDW(1, 0); RDW(1, 1); MM(1, 1); RDA(2); RDA(3); FV();
                MM(2, 0); M0SET(); MM(2, 1); MM(3, 0); LGKM0(); HAVE(); FREE(); MM(3, 1);
            }
            if constexpr (more) {
                if (__builtin_expect(miss != 0, 0)) wait_zero(FULL_AT(U + 1));
                if constexpr (U == 7) { fbase -= 8; asm volatile("" : "+v"(fbase)); }
            }
        };
        using T_ = std::integral_constant<bool, true>;
        __builtin_amdgcn_s_waitcnt(0x0F70);
        wait_zero(FULL_AT(0));
#pragma unroll 1
        for (int i = 0; i < N1 / 4; ++i) {
            static_for<8>([&](auto uc) { half(uc, T_{}, T_{}, T_{}, T_{}); });
        }
        constexpr int NX = 6, R1 = N1 % 4;
        const auto rxb = mkrsrc(myws + WS_XBUF, XBUF_BYTES);
        u32x4 xp[NX];
        unsigned call;
        int tid = threadIdx.x;
        auto fetch = [&]() {
            asm volatile("" ::: "memory");
#pragma unroll
            for (int k = 0; k < NX; ++k) xp[k] = __builtin_amdgcn_raw_buffer_load_b128(rxb, (bid * 4 + wid) * (NX * 1024) + (tid & 63) * 16 + k * 1024, 0, 16);
        };
        static_for<R1 + N2>([&](auto ic) {
            constexpr int s_ = decltype(ic)::value, i = s_ - R1;
            using Sig = std::integral_constant<bool, (i + 4 < N2)>;
            using Wide = std::integral_constant<bool, (i < 0)>;
            half(std::integral_constant<int, (s_ & 3) * 2>{}, std::integral_constant<bool, (i <= 0)>{}, Wide{}, Sig{}, T_{});
            if constexpr (i == 0) {
                MMEND();
                asm volatile("" : "+v"(tid));
                call = __builtin_amdgcn_readfirstlane(lds_ld(misc)) + 1;
                unsigned pk[18];
#pragma unroll
                for (int k = 0; k < 16; ++k) pk[k] = pack2(acc[k >> 2][2 + ((k >> 1) & 1)][(k & 1) * 2], acc[k >> 2][2 + ((k >> 1) & 1)][(k & 1) * 2 + 1]);
                pk[16] = pk[17] = 0;
#pragma unroll
                for (int k = 0; k < NX; ++k)
                    __builtin_amdgcn_raw_buffer_store_b128((u32x4){pk[3 * k], pk[3 * k + 1], pk[3 * k + 2], call}, rxb, ((bid ^ 1) * 4 + wid) * (NX * 1024) + (tid & 63) * 16 + k * 1024, 0, 16);
                SB();
            }
            half(std::integral_constant<int, (s_ & 3) * 2 + 1>{}, Wide{}, Wide{}, Sig{}, std::integral_constant<bool, (i + 1 < N2)>{});
            if constexpr (i + 2 == N2) { fetch(); SB(); }
        });
        unsigned vo = (unsigned)(mt * 128 + wm * 64 + (tid & 15)) * 8192u + nt * 256 + h * 128 + wn * 64 + ((tid >> 4) & 1) * 32 + ((tid >> 5) & 1) * 16;
        bool bad = __builtin_amdgcn_readfirstlane(lds_ld(misc + 1)) == 0;
        asm volatile("" : "+v"(vo));
        SB();
#pragma unroll
        for (int rb = 0; rb < 4; ++rb) { constexpr int H = 0; MM(rb, 0); MM(rb, 1); }
        MMEND();
        auto stale = [&]() { return __builtin_amdgcn_ballot_w64(((xp[0][3] ^ call) | (xp[1][3] ^ call) | (xp[2][3] ^ call) | (xp[3][3] ^ call) | (xp[4][3] ^ call) | (xp[5][3] ^ call)) != 0) != 0; };
        asm volatile("" :: "v"(xp[5]));
        if (stale()) {
            GAS unsigned* tmo = (GAS unsigned*)(myws + WS_TMO);
            const unsigned long long t0 = __builtin_amdgcn_s_memrealtime();
            for (unsigned n = 1;; ++n) {
                fetch();
                if (!stale()) break;
                if ((n & 15) == 0 && ((unsigned)__builtin_amdgcn_readfirstlane(__hip_atomic_load(tmo, __ATOMIC_RELAXED, __HIP_MEMORY_SCOPE_AGENT)) == call
                                      || __builtin_amdgcn_s_memrealtime() - t0 > DEADLINE)) {
                    __hip_atomic_store(tmo, call, __ATOMIC_RELAXED, __HIP_MEMORY_SCOPE_AGENT);
                    bad = true; break;
                }
                __builtin_amdgcn_s_sleep(1);
            }
        }
        const auto ro = mkrsrc(outp, MLOC * NTOT * 2);
#pragma unroll
        for (int rb = 0; rb < 4; ++rb) {
            unsigned x_[4];
#pragma unroll
            for (int k = 0; k < 4; ++k) x_[k] = xp[(rb * 4 + k) / 3][(rb * 4 + k) % 3];
            const f32x4 a = acc[rb][0] + __builtin_bit_cast(f32x4, (u32x4){x_[0] << 16, x_[0] & 0xffff0000u, x_[1] << 16, x_[1] & 0xffff0000u});
            const f32x4 b = acc[rb][1] + __builtin_bit_cast(f32x4, (u32x4){x_[2] << 16, x_[2] & 0xffff0000u, x_[3] << 16, x_[3] & 0xffff0000u});
            const auto s0 = __builtin_amdgcn_permlane16_swap(pack2(a[0], a[1]), pack2(b[0], b[1]), false, false);
            const auto s1 = __builtin_amdgcn_permlane16_swap(pack2(a[2], a[3]), pack2(b[2], b[3]), false, false);
            u32x4 o = {s0[0], s1[0], s0[1], s1[1]};
            if (bad) o = (u32x4){0x7fc07fc0u, 0x7fc07fc0u, 0x7fc07fc0u, 0x7fc07fc0u};
            __builtin_amdgcn_raw_buffer_store_b128(o, ro, vo + rb * 16 * 8192, 0, 0);
        }
    } else if (wid < 6) {
        const int j = wid & 1;
        unsigned full_a = sig_at(SIG_FULL + j * 2, 2);
        const int tab = tile_table(lane, rank, mt, h);
        const u32x4 rw4 = {(unsigned)(size_t)p.w, (unsigned)((size_t)p.w >> 32) & 0xffffu, KTOT * NTOT * 2, 0x00020000u};
        const unsigned vwb = (unsigned)(lane >> 3) * 8192u + nt * 256;
        const unsigned vw0 = vwb + (((lane & 7) ^ (((lane >> 4) & 1) * 2)) * 16), vw1 = vwb + (((lane & 7) ^ ((((lane >> 4) & 1) | 2) * 2)) * 16) + (65536 - 1024);
        const unsigned vw2 = vw0 + 2 * (65536 - 1024), vw3 = vw1 + 2 * (65536 - 1024);
#define LV4(aux) "buffer_load_dwordx4 %1, %5, %0 offen" aux " lds\n\tbuffer_load_dwordx4 %2, %5, %0 offen offset:1024" aux " lds\n\t" \
                 "buffer_load_dwordx4 %3, %5, %0 offen offset:2048" aux " lds\n\tbuffer_load_dwordx4 %4, %5, %0 offen offset:3072" aux " lds\n\t"
        auto load_w = [&](unsigned so, unsigned m, bool wide) {
            asm volatile("s_mov_b32 m0, %6\n\ts_nop 0\n\t" LV4("") :: "s"(so), "v"(vw0), "v"(vw1), "v"(vw2), "v"(vw3), "s"(rw4), "s"(m) : "memory");
            if (wide) asm volatile("s_add_u32 m0, m0, 0x2000\n\ts_xor_b32 %0, %0, 0x80\n\t" LV4("") : "+s"(so) : "v"(vw0), "v"(vw1), "v"(vw2), "v"(vw3), "s"(rw4) : "memory", "scc");
        };
        auto landed = [&](unsigned a) {
            asm volatile("s_mov_b32 m0, %0\n\ts_mov_b64 exec, 1\n\tbuffer_load_dword off, %1, 0 lds\n\ts_mov_b64 exec, -1" :: "s"(a), "s"(rz) : "memory");
        };
        {
            const u32x4 rx4 = {rz[0], rz[1], 4096 * KC * 2, 0x00020000u};
            const unsigned vx0 = (unsigned)(rank * 512 + mt * 128 + (lane >> 2)) * 1024u + ((lane & 3) ^ aswz(lane >> 4)) * 16;
            const unsigned vx1 = vx0 + (16384 - 1024), vx2 = vx0 + 2 * (16384 - 1024), vx3 = vx0 + 3 * (16384 - 1024);
            const int hold = slot < HOLD ? max(slot - 1, 0) / 2 : 4;
            START_BARRIER();
#pragma unroll 1
            for (int t = 0; t < 4; ++t) {
                if (t == hold) while (__builtin_amdgcn_readfirstlane(asm_lds_ld(lds0 + MISC_OFF + 8)) != 0) __builtin_amdgcn_s_sleep(1);
                asm volatile("s_waitcnt vmcnt(40)" ::: "memory");
                const int sl_ = 2 * (t == 0 ? 0 : 1 + (t - 1 + mt + (bid >> 5)) % 3) + h;
                load_w((rank * 8 + sl_) * (64u * 8192u) + j * (32u * 8192u) + h * 128, lds0 + W_OFF + t * SLOT + j * 4096, true);
                unsigned so = sl_ * 128 + j * 64;
                asm volatile("s_mov_b32 m0, %6\n\ts_nop 0\n\t" LV4("") "s_add_u32 m0, m0, 0x1000\n\ts_add_u32 %0, %0, 0x10000\n\t" LV4("")
                             : "+s"(so) : "v"(vx0), "v"(vx1), "v"(vx2), "v"(vx3), "s"(rx4), "s"(lds0 + t * SLOT + j * 8192) : "memory", "scc");
                landed(sig_at(SIG_FULL + (t & 1) * 2 + j, t >> 1));
            }
        }
        const size_t sb = (size_t)(myws + WS_STG) + mt * 8192;
        const u32x4 rs4 = {(unsigned)sb, ((unsigned)(sb >> 32) & 0xffffu) | (16u << 16), (unsigned)(2 * STG_PAR) - mt * 8192, 0x00800000u};
        unsigned c0;
        while ((c0 = __builtin_amdgcn_readfirstlane(asm_lds_ld(lds0 + MISC_OFF))) == ~0u) __builtin_amdgcn_s_sleep(1);
        asm volatile("" : "+s"(c0));
        const unsigned call = c0 + 1, par = (call & 1) * (unsigned)STG_PAR;
        unsigned watch = lane < 60 ? lds0 + FLG_OFF + lane * 4 : sig_at(SIG_FREE + (lane & 3), j);
        const int thr = lane < 60 ? (int)call : 0;
        const unsigned d1 = lane < 60 ? 0 : 2048, d2 = lane < 60 ? 0 : -2056;
        auto go = [&]() {
            unsigned long long busy;
            do busy = __builtin_amdgcn_ballot_w64((int)asm_lds_ld(watch) < thr); while (top4(busy) != 0);
            return busy;
        };
        unsigned long long left = j ? entries(4, N1, 1) : entries(4, N1, 0);
#define LA4 "buffer_load_dwordx4 off, %1, %0 sc0 sc1 lds\n\tbuffer_load_dwordx4 off, %1, %0 offset:1024 sc0 sc1 lds\n\t" \
            "buffer_load_dwordx4 off, %1, %0 offset:2048 sc0 sc1 lds\n\tbuffer_load_dwordx4 off, %1, %0 offset:3072 sc0 sc1 lds\n\t"
        auto load_a = [&](unsigned so, unsigned m) {
            asm volatile("s_mov_b32 m0, %2\n\ts_nop 0\n\t" LA4 "s_add_u32 m0, m0, 0x1000\n\ts_add_u32 %0, %0, 0x1000\n\t" LA4
                         : "+s"(so) : "s"(rs4), "s"(m) : "memory", "scc");
        };
#pragma unroll 1
        for (int t = 4 + j; t < NT; t += 2) {
            if (left == 0) left = j ? entries(N1, NT, 1) : entries(N1, NT, 0);
            const unsigned sl = lds0 + (t & 3) * SLOT;
            asm volatile("s_waitcnt vmcnt(40)" ::: "memory");
            unsigned long long busy = go();
            int pick = 63 - __builtin_clzll((left & ~busy) != 0 ? left & ~busy : left);
            unsigned tile = __builtin_amdgcn_readlane(tab, pick);
            load_w(tile * (64u * 8192u) + h * 128, sl + W_OFF, t < N1);
            if (__builtin_expect((left & ~busy) == 0, 0)) {
                do { __builtin_amdgcn_s_sleep(1); busy = __builtin_amdgcn_ballot_w64((int)asm_lds_ld(watch) < thr); } while ((left & ~busy) == 0);
                if ((busy >> pick) & 1) {
                    pick = 63 - __builtin_clzll(left & ~busy);
                    tile = __builtin_amdgcn_readlane(tab, pick);
                    asm volatile("s_waitcnt vmcnt(40)" ::: "memory");
                    load_w(tile * (64u * 8192u) + h * 128, sl + W_OFF, t < N1);
                }
            }
            left &= ~(1ull << pick);
            const unsigned sw = tile * (64u * 8192u) + h * 128, sa = tile * 65536u + par;
            load_a(sa, sl);
            landed(full_a);
            asm volatile("s_waitcnt vmcnt(40)" ::: "memory");
            watch += d1;
            go();
            watch += d2;
            load_w(sw + 32u * 8192u, sl + W_OFF + 4096, t < N1);
            load_a(sa + 32768, sl + 8192);
            landed(full_a + 512);
            full_a -= 4;
        }
    } else if (wid == 6) {
        if (slot < 32) {
            __builtin_amdgcn_s_setprio(3);
            const unsigned tstart = (unsigned)__builtin_amdgcn_s_memrealtime();
            const int dest = (rank + 1 + di) & 7, slice = slot >> 2, tm = (slot - ((6 - di) >> 1)) & 3;
            GAS unsigned* cnt = (GAS unsigned*)(myws + WS_CNT + bid * 128);
            const auto rx = mkrsrc(p.x, 4096 * KC * 2);
            const unsigned vx = (unsigned)(dest * 512 + tm * 128 + ((lane >> 2) & 7)) * 1024u + slice * 128 + (lane >> 5) * 64;
            const unsigned vx0 = vx + ((lane & 3) ^ aswz((lane >> 4) & 1)) * 16, vx1 = vx + ((lane & 3) ^ aswz(2 + ((lane >> 4) & 1))) * 16;
            if (slot >= LATE) {
                __builtin_amdgcn_s_barrier();
                while (__builtin_amdgcn_readfirstlane(asm_lds_ld(sig_at(SIG_FULL, 1))) != 0) __builtin_amdgcn_s_sleep(2);
            }
            u32x4 v[16];
#pragma unroll
            for (int i = 0; i < 16; ++i) v[i] = __builtin_amdgcn_raw_buffer_load_b128(rx, (i & 1) ? vx1 : vx0, i * 8192, 0);
            const unsigned c0 = __hip_atomic_load(cnt, __ATOMIC_RELAXED, __HIP_MEMORY_SCOPE_AGENT);
            if (slot < LATE) __builtin_amdgcn_s_barrier();
            char* const dws = p.peer[di];
            __builtin_amdgcn_s_waitcnt(0x0F70);
            SB();
            const unsigned call = __builtin_amdgcn_readfirstlane(c0) + 1;
            char* const dst = dws + WS_STG + (call & 1) * STG_PAR + (rank * 8 + slice) * 65536 + (lane >> 5) * 32768 + tm * 8192 + (lane & 31) * 16 + 4096;
            asm volatile("" :: "v"(dst));
            for (int left; (left = departs(slot) - (int)((unsigned)__builtin_amdgcn_s_memrealtime() - tstart)) > 0;) {
                if (left > 80) __builtin_amdgcn_s_sleep(16); else __builtin_amdgcn_s_sleep(1);
            }
            static_for<16>([&](auto ic) {
                constexpr int i = decltype(ic)::value;
                char* const d = dst; const u32x4 vi = v[i];
                asm volatile("global_store_dwordx4 %0, %1, off offset:%2 sc0 sc1\n\ts_nop 1" :: "v"(d), "v"(vi), "i"(i * 512 - 4096) : "memory");
            });
            if (slot < HOLD && lane == 0) lds_st(misc + 2, 0u);
            asm volatile("s_waitcnt vmcnt(0)" ::: "memory");
            if (lane == 0) {
                __hip_atomic_store((GAS unsigned*)(dws + WS_FLAG + ((tm * 8 + slice) * 8 + rank) * 128), call, __ATOMIC_RELAXED, __HIP_MEMORY_SCOPE_SYSTEM);
                __hip_atomic_store(cnt, call, __ATOMIC_RELAXED, __HIP_MEMORY_SCOPE_AGENT);
            }
        } else {
            __builtin_amdgcn_s_barrier();
        }
    } else if (wid == 7) {
        __builtin_amdgcn_s_setprio(3);
        const unsigned long long t0 = __builtin_amdgcn_s_memrealtime();
        GAS unsigned* cnt = (GAS unsigned*)(myws + WS_CNT + bid * 128 + 64);
        const auto rc = mkrsrc(myws, WS_STG);
        const unsigned c0 = __hip_atomic_load(cnt, __ATOMIC_RELAXED, __HIP_MEMORY_SCOPE_AGENT);
        const int st = 63 - lane, rt = tile_table(lane, rank, mt, h);
        const unsigned foff = st < NT ? WS_FLAG + ((mt * 8 + (rt & 7)) * 8 + (rt >> 3)) * 128 : WS_TMO;
        const int fslot = (rt & 7) * 4 + ((mt + ((((rt >> 3) - rank - 1) & 7) >> 1)) & 3);
        const unsigned due = (unsigned)t0 + departs(fslot) + (fslot < HOLD ? GATE0 : GATE);
        START_BARRIER();
        if (lane == 0) __builtin_amdgcn_raw_ptr_buffer_load_lds(rc, (LAS void*)misc, 4, (unsigned)(WS_CNT + bid * 128 + 64), 0, 0, 16);
        asm volatile("s_waitcnt vmcnt(0)" ::: "memory");
        const unsigned call = __builtin_amdgcn_readfirstlane(c0) + 1;
        if (lane == 0) __hip_atomic_store(cnt, call, __ATOMIC_RELAXED, __HIP_MEMORY_SCOPE_AGENT);
        bool bad = false;
        for (unsigned n = 1;; ++n) {
            const unsigned v = asm_lds_ld(lds0 + FLG_OFF + lane * 4);
            const bool here = v >= call || st < 4 || st >= NT;
            const unsigned long long seen = __builtin_amdgcn_ballot_w64(here), now = __builtin_amdgcn_s_memrealtime();
            bad = __builtin_amdgcn_readfirstlane(v) == call || now - t0 > DEADLINE;
            if (bad || seen == ~0ull) break;
            const int have = min(__builtin_popcountll(seen & entries(0, N1, 0)), __builtin_popcountll(seen & entries(0, N1, 1))) - 3;
            if (have < 0 || __builtin_amdgcn_readfirstlane(asm_lds_ld(sig_at(SIG_FULL, have))) == 0) {
                if ((lane == 0 && (n & 15) == 0) || (!here && (int)((unsigned)now - due) >= 0))
                    __builtin_amdgcn_raw_ptr_buffer_load_lds(rc, (LAS void*)(lds + FLG_OFF), 4, foff, 0, 0, 17);
                asm volatile("s_waitcnt vmcnt(0)" ::: "memory");
            }
            __builtin_amdgcn_s_sleep(4);
        }
        if (bad) {
            if (lane == 0) { __hip_atomic_store((GAS unsigned*)(myws + WS_TMO), call, __ATOMIC_RELAXED, __HIP_MEMORY_SCOPE_AGENT); lds_st(misc + 1, 0u); }
            lds_st((LAS unsigned*)(lds + FLG_OFF) + lane, call);
        }
    } else {
        __builtin_amdgcn_s_barrier();
        const char* const wq = p.w + (size_t)(mt * 16 + (wid - 8) * 4) * 8192 + nt * 256;
        unsigned junk = 0;
#pragma unroll 1
        for (int gi = G0 + h; gi < 64; gi += gi + 2 < 2 * N1 ? 2 : gi < 2 * N1 ? 2 * N1 - gi : 1) {
            const int u = gi - 8, slice = u / 7, src = (rank + 1 + u - slice * 7) & 7;
            const char* const a = wq + (size_t)(src * 8 + slice) * (64 * 8192);
#define PF2(r) "s_load_dword %0, %1, " #r "*8192\n\ts_load_dword %0, %1, " #r "*8192+64\n\t"
            asm volatile(PF2(0) PF2(1) PF2(2) PF2(3) "s_nop 0" : "+s"(junk) : "s"(a + h * 128) : "memory");
            if (gi < 2 * N1) asm volatile(PF2(0) PF2(1) PF2(2) PF2(3) "s_nop 0" : "+s"(junk) : "s"(a + (h ^ 1) * 128) : "memory");
        }
        asm volatile("s_waitcnt lgkmcnt(0)" : "+s"(junk) :: "memory");
    }
}
}

extern "C" void kernel_launch(int rank, int nranks, ncclComm_t comm,
                              hipStream_t stream,
                              void* const* d_in, const int* in_sizes, int n_in,
                              void* d_out, int out_size,
                              const int* in_dtypes, int out_dtype,
                              void* const* d_out_all,
                              void* const* d_ws_all, size_t ws_bytes) {
    using namespace a2a;
    if (nranks != NR || (unsigned)rank >= (unsigned)NR || n_in < 2 || ws_bytes < WS_NEED ||
        in_sizes[0] != 4096 * KC || in_sizes[1] != KTOT * NTOT || out_size != MLOC * NTOT) {
        fprintf(stderr, "a2a_gemm: unexpected problem shape\n");
        return;
    }
    Params p;
    p.x = (const char*)d_in[0]; p.w = (const char*)d_in[1]; p.out = (char*)d_out;
    p.ws = (char*)d_ws_all[rank];
    for (int i = 0; i < NR - 1; ++i) p.peer[i] = (char*)d_ws_all[(rank + 1 + i) % NR];
    p.rank = rank; p.pad = 0;
    hipLaunchKernelGGL(a2a_gemm, dim3(NWG), dim3(NTHREADS), 0, stream, p);
    HIP_CHECK(hipGetLastError());
}
```

```cpp
#include <hip/hip_runtime.h>
#include <rccl/rccl.h>
#include <cstdio>
#include <cstdint>
#include <type_traits>
#include <utility>

#define HIP_CHECK(expr) do { hipError_t _e = (expr); if (_e != hipSuccess) { \
    fprintf(stderr, "HIP error %s at %s:%d\n", hipGetErrorString(_e), __FILE__, __LINE__); \
    return; } } while (0)

namespace a2a {
#define LAS __attribute__((address_space(3)))
#define GAS __attribute__((address_space(1)))
typedef unsigned u32x4 __attribute__((ext_vector_type(4)));
typedef float f32x4 __attribute__((ext_vector_type(4)));
typedef __bf16 bf16x8 __attribute__((ext_vector_type(8)));
typedef __bf16 bf16x4 __attribute__((ext_vector_type(4)));
typedef __bf16 bf16x2 __attribute__((ext_vector_type(2)));
typedef float f32x2 __attribute__((ext_vector_type(2)));

constexpr int T0 = 80;
constexpr int TAU = 32;
constexpr int HOLD = 9;
constexpr int LATE = 12;
constexpr int GATE0 = 170, GATE = 220;
constexpr int U0 = 10;

constexpr int NR = 8;
constexpr int MLOC = 512, KTOT = 4096, NTOT = 4096, KC = 512;
constexpr int NWG = 256, NTHREADS = 12 * 64;
constexpr int NT = 32, NL = NT - 4;
constexpr int SLOT = 16384, W_OFF = 4 * SLOT;
constexpr int SIG_OFF = 8 * SLOT;
constexpr int SIG_FULL = 0;
constexpr int SIG_FREE = 4;
constexpr int NSIG = 12;
constexpr int MISC_OFF = SIG_OFF + NSIG * 512;
constexpr int FLG_OFF = MISC_OFF + 64;
constexpr int LDS_BYTES = FLG_OFF + 256;
static_assert(LDS_BYTES <= 163840 && LDS_BYTES > 81920, "one workgroup per CU");
constexpr size_t WS_CNT = 0;
constexpr size_t WS_TMO = 32768;
constexpr size_t WS_FLAG = 65536;
constexpr size_t WS_STG = 1u << 20;
constexpr size_t STG_PAR = 4u << 20;
constexpr size_t WS_XBUF = WS_STG + 2 * STG_PAR;
constexpr size_t XBUF_BYTES = 5u << 20;
constexpr size_t WS_NEED = WS_XBUF + XBUF_BYTES;
constexpr unsigned long long DEADLINE = 50000000ull;

struct Params {
    const char* x; const char* w; char* out;
    char* ws;
    int rank; int pad;
    char* peer[NR - 1];
};

__device__ __forceinline__ __amdgpu_buffer_rsrc_t mkrsrc(const void* ptr, unsigned bytes) {
    return __builtin_amdgcn_make_buffer_rsrc((void*)ptr, 0, bytes, 0x00020000);
}
__device__ __forceinline__ unsigned lds_ld(LAS unsigned* a) { return __hip_atomic_load(a, __ATOMIC_RELAXED, __HIP_MEMORY_SCOPE_WORKGROUP); }
__device__ __forceinline__ void lds_st(LAS unsigned* a, unsigned v) { __hip_atomic_store(a, v, __ATOMIC_RELAXED, __HIP_MEMORY_SCOPE_WORKGROUP); }
__device__ __forceinline__ int aswz(int i) { return ((i << 1) ^ ((i >> 1) * 3)) & 3; }
__device__ __forceinline__ unsigned pack2(float a, float b) { f32x2 v = {a, b}; return __builtin_bit_cast(unsigned, __builtin_convertvector(v, bf16x2)); }
__device__ __forceinline__ unsigned asm_lds_ld(unsigned addr) {
    unsigned v;
    asm volatile("ds_read_b32 %0, %1\n\ts_waitcnt lgkmcnt(0)" : "=&v"(v) : "v"(addr) : "memory");
    return v;
}
template <int N, class F, int... I> __device__ __forceinline__ void static_for_impl(F f, std::integer_sequence<int, I...>) { (f(std::integral_constant<int, I>{}), ...); }
template <int N, class F> __device__ __forceinline__ void static_for(F f) { static_for_impl<N>(f, std::make_integer_sequence<int, N>{}); }
__device__ __forceinline__ int tile_table(int lane, int rank, int mt, int h) {
    const int j = lane >> 3, pi = 2 * ((((lane >> 1) & 3) - mt) & 3) + (lane & 1);
    const bool mine = pi < 7 && ((j + pi) & 1) == h;
    const int n = __builtin_popcountll(__builtin_amdgcn_ballot_w64(mine) & ((1ull << lane) - 1));
    return __builtin_amdgcn_ds_permute((mine ? 59 - n : 0) * 4, (((rank + 1 + pi) & 7) << 3) | j);
}
constexpr unsigned long long entries(int from, int parity) {
    unsigned long long m = 0;
    for (int t = from; t < NT; ++t) if ((t & 1) == parity) m |= 1ull << (63 - t);
    return m;
}
__device__ __forceinline__ unsigned top4(unsigned long long m) { unsigned hi = (unsigned)(m >> 32); asm("" : "+s"(hi)); return hi >> 28; }
__device__ __forceinline__ int departs(int slot) { return T0 + slot * TAU; }
#define START_BARRIER() do { asm volatile("s_waitcnt lgkmcnt(0)" ::: "memory"); __builtin_amdgcn_s_barrier(); asm volatile("" ::: "memory"); } while (0)
#define SB() __builtin_amdgcn_sched_barrier(0)

__global__ __launch_bounds__(NTHREADS, 3) void a2a_gemm(const Params p) {
    __shared__ __attribute__((aligned(16))) char smem[LDS_BYTES];
    LAS char* lds = (LAS char*)smem;
    const unsigned lds0 = (unsigned)(size_t)lds, sig0 = lds0 + SIG_OFF;
    LAS unsigned* misc = (LAS unsigned*)(lds + MISC_OFF);
    auto sig_at = [&](int arr, int e) { return sig0 + arr * 512 + (63 - e) * 4; };
    const int bid = blockIdx.x, lane = threadIdx.x & 63;
    const int wid = __builtin_amdgcn_readfirstlane(threadIdx.x >> 6);
    const int rank = p.rank;
    asm volatile("" :: "s"(p.x), "s"(p.w), "s"(p.out), "s"(p.ws), "s"(rank));
    const int h = bid & 1, mt = (bid >> 3) & 3, nt = (bid >> 5) * 4 + ((bid >> 1) & 3);
    const int slot = (bid & 7) == 3 ? 32 : bid >> 3, di = (bid & 7) - ((bid & 7) > 3);
    char* const myws = p.ws;
    const u32x4 rz = {(unsigned)(size_t)p.x, (unsigned)((size_t)p.x >> 32) & 0xffffu, 0u, 0x00020000u};

    if (wid < 4) {
        for (int i = threadIdx.x; i < (NSIG * 512 + 64) / 16; i += 256) *(LAS u32x4*)(lds + SIG_OFF + i * 16) = (u32x4){~0u, ~0u, ~0u, ~0u};
        if (wid == 3) *(LAS unsigned*)(lds + FLG_OFF + lane * 4) = 0u;
        START_BARRIER();
        __builtin_amdgcn_s_setprio(3);
        const int wm = wid >> 1, wn = wid & 1, q = (lane >> 2) & 3, g = lane >> 4, r16 = lane & 15, f = (q >> 1) | ((g & 1) << 1);
        const unsigned a0 = lds0 + (wm * 64 + r16) * 64 + ((g ^ aswz(r16 >> 2)) * 16);
        const unsigned w0 = lds0 + W_OFF + (8 * g + q) * 128 + (((wn * 2) ^ f) * 32) + (lane & 3) * 8, w1 = w0 ^ 32;
        unsigned fbase = sig_at(SIG_FULL, 2);
        unsigned free_a = sig_at(SIG_FREE + wid, 0);
        char* outp = p.out;
        asm volatile("" : "+s"(outp));
        auto wait_zero = [&](unsigned a) {
            while (__builtin_amdgcn_ballot_w64(*(volatile LAS unsigned*)(size_t)a != 0) != 0) __builtin_amdgcn_s_sleep(1);
            asm volatile("" ::: "memory");
        };
        bf16x8 af[2][4], wf[2][4];
        bf16x4 wl;
        f32x4 acc[4][4];
        unsigned fv;
#pragma unroll
        for (int i = 0; i < 4; ++i) { af[1][i] = (bf16x8)(__bf16)0.f; wf[1][i] = (bf16x8)(__bf16)0.f; }
#pragma unroll
        for (int i = 0; i < 16; ++i) acc[i >> 2][i & 3] = (f32x4){0.f, 0.f, 0.f, 0.f};
#define FULL_AT(U) (fbase + ((U) & 3) * 512 + (2 - ((U) >> 2)) * 4)
#define FV() do { if constexpr (more) { fv = *(volatile LAS unsigned*)(size_t)FULL_AT(U + 1); SB(); } } while (0)
#define LGKM0() do { __builtin_amdgcn_s_waitcnt(0xC07F); SB(); } while (0)
#define HAVE() do { if constexpr (more) { miss = __builtin_amdgcn_ballot_w64(fv != 0); asm volatile("" : "+s"(miss)); SB(); } } while (0)
#define M0SET() do { if constexpr (sig) { \
            if constexpr (H) asm volatile("s_mov_b32 m0, %0\n\ts_sub_u32 %0, %0, 2052" : "+s"(free_a) :: "memory", "scc"); \
            else asm volatile("s_mov_b32 m0, %0\n\ts_add_u32 %0, %0, 2048" : "+s"(free_a) :: "memory", "scc"); \
            SB(); } } while (0)
#define FREE() do { if constexpr (sig) { const u32x4 rz_ = rz; asm volatile("buffer_load_dword off, %0, 0 lds" :: "s"(rz_) : "memory"); SB(); } } while (0)
#define RDW(cb, t2) do { const bf16x4 w_ = __builtin_amdgcn_ds_read_tr16_b64_v4bf16((LAS bf16x4*)(size_t)(((cb & 1) ? w1 : w0) + S * SLOT + H * 4096 + ((cb >> 1) + IMG) * 8192 + t2 * 512)); \
            if (t2 == 0) wl = w_; else wf[H][cb] = __builtin_shufflevector(wl, w_, 0, 1, 2, 3, 4, 5, 6, 7); SB(); } while (0)
#define RDA(rb) do { af[H][rb] = *(LAS bf16x8*)(size_t)(a0 + S * SLOT + H * 8192 + rb * 1024); SB(); } while (0)
#define MM(rb, cb) do { f32x4& c_ = acc[rb][C0 + cb]; const bf16x8 w_ = wf[H ^ 1][cb], a_ = af[H ^ 1][rb]; \
            asm volatile("v_mfma_f32_16x16x32_bf16 %0, %1, %2, %0" : "+v"(c_) : "v"(w_), "v"(a_)); SB(); } while (0)
#define MMEND() do { asm volatile("s_nop 7\n\ts_nop 1" ::: "memory"); SB(); } while (0)
        auto half = [&](auto uc, auto kc, auto more) {
            constexpr int U = decltype(uc)::value, S = U >> 1, H = U & 1, K = decltype(kc)::value, IMG = K == 1 || K == 2, C0 = K == 2 ? 2 : 0;
            constexpr bool sig = K == 0;
            unsigned long long miss = 0;
            if constexpr (K == 0) {
                MM(0, 0); RDW(0, 0); MM(0, 1); RDW(0, 1); RDA(0); MM(0, 2); RDW(1, 0); MM(0, 3); RDW(1, 1);
                MM(1, 0); RDW(2, 0); RDW(2, 1); MM(1, 1); RDW(3, 0); MM(1, 2); RDW(3, 1); MM(1, 3); RDA(1);
                MM(2, 0); RDA(2); MM(2, 1); RDA(3); FV(); MM(2, 2); M0SET(); MM(2, 3);
                MM(3, 0); MM(3, 1); LGKM0(); HAVE(); MM(3, 2); FREE(); MM(3, 3);
            } else if constexpr (K == 1) {
                MM(0, 0); RDW(0, 0); MM(0, 1); RDW(0, 1); MM(0, 2); RDA(0); MM(0, 3); RDW(1, 0);
                MM(1, 0); RDW(1, 1); MM(1, 1); RDA(1); MM(1, 2); RDA(2); MM(1, 3); RDA(3);
                MM(2, 0); MM(2, 1); FV(); MM(2, 2); MM(2, 3);
                MM(3, 0); MM(3, 1); LGKM0(); HAVE(); MM(3, 2); MM(3, 3);
            } else if constexpr (K == 3) {
                RDW(0, 0); RDW(0, 1); RDA(0); RDW(1, 0); RDW(1, 1); RDA(1); RDA(2); RDA(3);
            } else {
                FV(); MM(0, 0); RDW(0, 0); RDW(0, 1); MM(0, 1); RDA(0); MM(1, 0); RDW(1, 0); RDW(1, 1); MM(1, 1); RDA(1);
                MM(2, 0); RDA(2); MM(2, 1); RDA(3); MM(3, 0); HAVE(); MM(3, 1);
            }
            if constexpr (more) {
                if (__builtin_expect(miss != 0, 0)) wait_zero(FULL_AT(U + 1));
                if constexpr (U == 7) { fbase -= 8; asm volatile("" : "+v"(fbase)); }
            }
        };
        using F_ = std::integral_constant<bool, false>;
        using T_ = std::integral_constant<bool, true>;
        __builtin_amdgcn_s_waitcnt(0x0F70);
        wait_zero(FULL_AT(0));
#pragma unroll 1
        for (int i = 0; i < NL / 4; ++i) {
            static_for<8>([&](auto uc) { half(uc, std::integral_constant<int, 0>{}, T_{}); });
        }
        constexpr int NX = 5;
        const auto rxb = mkrsrc(myws + WS_XBUF, XBUF_BYTES);
        u32x4 xp[NX];
        unsigned call;
        int tid = threadIdx.x;
        auto fetch = [&]() {
            asm volatile("" ::: "memory");
#pragma unroll
            for (int k = 0; k < NX; ++k) xp[k] = __builtin_amdgcn_raw_buffer_load_b128(rxb, (bid * 4 + wid) * (NX * 1024) + (tid & 63) * 16 + k * 1024, 0, 16);
        };
        static_for<8>([&](auto uc) { half(uc, std::integral_constant<int, decltype(uc)::value == 0 ? 1 : 2>{}, std::integral_constant<bool, (decltype(uc)::value < 7)>{}); });
        asm volatile("" : "+v"(tid));
        call = __builtin_amdgcn_readfirstlane(lds_ld(misc)) + 1;
        SB();
#pragma unroll
        for (int rb = 0; rb < 4; ++rb) { constexpr int H = 0, C0 = 2; MM(rb, 0); MM(rb, 1); }
        MMEND();
        {
            unsigned pk[16];
#pragma unroll
            for (int k = 0; k < 16; ++k) pk[k] = pack2(acc[k >> 2][2 + ((k >> 1) & 1)][(k & 1) * 2], acc[k >> 2][2 + ((k >> 1) & 1)][(k & 1) * 2 + 1]);
#pragma unroll
            for (int k = 0; k < NX; ++k)
                __builtin_amdgcn_raw_buffer_store_b128((u32x4){pk[3 * k], pk[3 * k + 1], pk[3 * k + 2], (call & 0xffffu) | (k == 3 ? pk[15] << 16 : k == 4 ? pk[15] & 0xffff0000u : 0u)},
                                                       rxb, ((bid ^ 1) * 4 + wid) * (NX * 1024) + (tid & 63) * 16 + k * 1024, 0, 16);
            SB();
        }
        wait_zero(sig_at(SIG_FULL + (tid & 1) * 2, NT / 2));
        static_for<8>([&](auto uc) {
            half(uc, std::integral_constant<int, decltype(uc)::value == 0 ? 3 : 4>{}, F_{});
            if constexpr (decltype(uc)::value == 5) { fetch(); SB(); }
        });
        unsigned vo = (unsigned)(mt * 128 + wm * 64 + (tid & 15)) * 8192u + nt * 256 + h * 128 + wn * 64 + ((tid >> 4) & 1) * 32 + ((tid >> 5) & 1) * 16;
        bool bad = __builtin_amdgcn_readfirstlane(lds_ld(misc + 1)) == 0;
        asm volatile("" : "+v"(vo));
        SB();
#pragma unroll
        for (int rb = 0; rb < 4; ++rb) { constexpr int H = 0, C0 = 0; MM(rb, 0); MM(rb, 1); }
        MMEND();
        auto stale = [&]() { return __builtin_amdgcn_ballot_w64((((xp[0][3] ^ call) | (xp[1][3] ^ call) | (xp[2][3] ^ call) | (xp[3][3] ^ call) | (xp[4][3] ^ call)) & 0xffffu) != 0) != 0; };
        if (stale()) {
            GAS unsigned* tmo = (GAS unsigned*)(myws + WS_TMO);
            const unsigned long long t0 = __builtin_amdgcn_s_memrealtime();
            for (unsigned n = 1;; ++n) {
                fetch();
                if (!stale()) break;
                if ((n & 15) == 0 && ((unsigned)__builtin_amdgcn_readfirstlane(__hip_atomic_load(tmo, __ATOMIC_RELAXED, __HIP_MEMORY_SCOPE_AGENT)) == call
                                      || __builtin_amdgcn_s_memrealtime() - t0 > DEADLINE)) {
                    __hip_atomic_store(tmo, call, __ATOMIC_RELAXED, __HIP_MEMORY_SCOPE_AGENT);
                    bad = true; break;
                }
                __builtin_amdgcn_s_sleep(1);
            }
        }
        const auto ro = mkrsrc(outp, MLOC * NTOT * 2);
#pragma unroll
        for (int rb = 0; rb < 4; ++rb) {
            unsigned x_[4];
#pragma unroll
            for (int k = 0; k < 4; ++k) x_[k] = rb * 4 + k < 15 ? xp[(rb * 4 + k) / 3][(rb * 4 + k) % 3] : (xp[3][3] >> 16) | (xp[4][3] & 0xffff0000u);
            const f32x4 a = acc[rb][0] + __builtin_bit_cast(f32x4, (u32x4){x_[0] << 16, x_[0] & 0xffff0000u, x_[1] << 16, x_[1] & 0xffff0000u});
            const f32x4 b = acc[rb][1] + __builtin_bit_cast(f32x4, (u32x4){x_[2] << 16, x_[2] & 0xffff0000u, x_[3] << 16, x_[3] & 0xffff0000u});
            const auto s0 = __builtin_amdgcn_permlane16_swap(pack2(a[0], a[1]), pack2(b[0], b[1]), false, false);
            const auto s1 = __builtin_amdgcn_permlane16_swap(pack2(a[2], a[3]), pack2(b[2], b[3]), false, false);
            u32x4 o = {s0[0], s1[0], s0[1], s1[1]};
            if (bad) o = (u32x4){0x7fc07fc0u, 0x7fc07fc0u, 0x7fc07fc0u, 0x7fc07fc0u};
            __builtin_amdgcn_raw_buffer_store_b128(o, ro, vo + rb * 16 * 8192, 0, 0);
        }
    } else if (wid < 6) {
        const int j = wid & 1;
        unsigned full_a = sig_at(SIG_FULL + j * 2, 2);
        int tab;
        const u32x4 rw4 = {(unsigned)(size_t)p.w, (unsigned)((size_t)p.w >> 32) & 0xffffu, KTOT * NTOT * 2, 0x00020000u};
        const unsigned vwb = (unsigned)(lane >> 3) * 8192u + nt * 256;
        const unsigned vw0 = vwb + (((lane & 7) ^ (((lane >> 4) & 1) * 2)) * 16), vw1 = vwb + (((lane & 7) ^ ((((lane >> 4) & 1) | 2) * 2)) * 16) + (65536 - 1024);
        const unsigned vw2 = vw0 + 2 * (65536 - 1024), vw3 = vw1 + 2 * (65536 - 1024);
#define LV4(aux) "buffer_load_dwordx4 %1, %5, %0 offen" aux " lds\n\tbuffer_load_dwordx4 %2, %5, %0 offen offset:1024" aux " lds\n\t" \
                 "buffer_load_dwordx4 %3, %5, %0 offen offset:2048" aux " lds\n\tbuffer_load_dwordx4 %4, %5, %0 offen offset:3072" aux " lds\n\t"
        auto load_w1 = [&](unsigned so, unsigned m) {
            asm volatile("s_mov_b32 m0, %6\n\ts_nop 0\n\t" LV4("") :: "s"(so), "v"(vw0), "v"(vw1), "v"(vw2), "v"(vw3), "s"(rw4), "s"(m) : "memory");
        };
        auto load_w = [&](unsigned so, unsigned m, bool both) { load_w1(so ^ 128, m + 8192); if (both) load_w1(so, m); };
        auto landed = [&](unsigned a) {
            asm volatile("s_mov_b32 m0, %0\n\ts_mov_b64 exec, 1\n\tbuffer_load_dword off, %1, 0 lds\n\ts_mov_b64 exec, -1" :: "s"(a), "s"(rz) : "memory");
        };
        {
            const u32x4 rx4 = {rz[0], rz[1], 4096 * KC * 2, 0x00020000u};
            const unsigned vx0 = (unsigned)(rank * 512 + mt * 128 + (lane >> 2)) * 1024u + ((lane & 3) ^ aswz(lane >> 4)) * 16;
            const unsigned vx1 = vx0 + (16384 - 1024), vx2 = vx0 + 2 * (16384 - 1024), vx3 = vx0 + 3 * (16384 - 1024);
            const int hold = slot < HOLD ? max(slot - 1, 0) / 2 : 4;
            auto local = [&](int t, int sl_) {
                load_w((rank * 8 + sl_) * (64u * 8192u) + j * (32u * 8192u) + h * 128, lds0 + W_OFF + t * SLOT + j * 4096, true);
                unsigned so = sl_ * 128 + j * 64;
                asm volatile("s_mov_b32 m0, %6\n\ts_nop 0\n\t" LV4("") "s_add_u32 m0, m0, 0x1000\n\ts_add_u32 %0, %0, 0x10000\n\t" LV4("")
                             : "+s"(so) : "v"(vx0), "v"(vx1), "v"(vx2), "v"(vx3), "s"(rx4), "s"(lds0 + t * SLOT + j * 8192) : "memory", "scc");
            };
            auto burst_out = [&]() { while (__builtin_amdgcn_readfirstlane(asm_lds_ld(lds0 + MISC_OFF + 8)) != 0) __builtin_amdgcn_s_sleep(1); };
            const bool first = slot >= LATE;
            if (!first) {
                START_BARRIER();
                if (hold == 0) burst_out();
            }
            local(0, h);
            tab = tile_table(lane, rank, mt, h);
            if (first) START_BARRIER();
            landed(sig_at(SIG_FULL + j, 0));
#pragma unroll 1
            for (int t = 1; t < 4; ++t) {
                if (t == hold) burst_out();
                asm volatile("s_waitcnt vmcnt(40)" ::: "memory");
                local(t, 2 * (1 + (t - 1 + mt + (bid >> 5)) % 3) + h);
                landed(sig_at(SIG_FULL + (t & 1) * 2 + j, t >> 1));
            }
        }
        const size_t sb = (size_t)(myws + WS_STG) + mt * 8192;
        const u32x4 rs4 = {(unsigned)sb, ((unsigned)(sb >> 32) & 0xffffu) | (16u << 16), (unsigned)(2 * STG_PAR) - mt * 8192, 0x00800000u};
        unsigned c0;
        while ((c0 = __builtin_amdgcn_readfirstlane(asm_lds_ld(lds0 + MISC_OFF))) == ~0u) __builtin_amdgcn_s_sleep(1);
        asm volatile("" : "+s"(c0));
        const unsigned call = c0 + 1, par = (call & 1) * (unsigned)STG_PAR;
        unsigned watch = lane < 60 ? lds0 + FLG_OFF + lane * 4 : sig_at(SIG_FREE + (lane & 3), j);
        const int thr = lane < 60 ? (int)call : 0;
        const unsigned d1 = lane < 60 ? 0 : 2048, d2 = lane < 60 ? 0 : -2056;
        auto go = [&]() {
            unsigned long long busy;
            do busy = __builtin_amdgcn_ballot_w64((int)asm_lds_ld(watch) < thr); while (top4(busy) != 0);
            return busy;
        };
        unsigned long long left = j ? entries(4, 1) : entries(4, 0);
#define LA4 "buffer_load_dwordx4 off, %1, %0 sc0 sc1 lds\n\tbuffer_load_dwordx4 off, %1, %0 offset:1024 sc0 sc1 lds\n\t" \
            "buffer_load_dwordx4 off, %1, %0 offset:2048 sc0 sc1 lds\n\tbuffer_load_dwordx4 off, %1, %0 offset:3072 sc0 sc1 lds\n\t"
        auto load_a = [&](unsigned so, unsigned m) {
            asm volatile("s_mov_b32 m0, %2\n\ts_nop 0\n\t" LA4 "s_add_u32 m0, m0, 0x1000\n\ts_add_u32 %0, %0, 0x1000\n\t" LA4
                         : "+s"(so) : "s"(rs4), "s"(m) : "memory", "scc");
        };
        unsigned swp = 0, swl = 0;
#pragma unroll 1
        for (int t = 4 + j; t < NT; t += 2) {
            const unsigned sl = lds0 + (t & 3) * SLOT;
            asm volatile("s_waitcnt vmcnt(40)" ::: "memory");
            unsigned long long busy = go();
            int pick = 63 - __builtin_clzll((left & ~busy) != 0 ? left & ~busy : left);
            unsigned tile = __builtin_amdgcn_readlane(tab, pick);
            load_w(tile * (64u * 8192u) + h * 128, sl + W_OFF, t < NL);
            if (__builtin_expect((left & ~busy) == 0, 0)) {
                do { __builtin_amdgcn_s_sleep(1); busy = __builtin_amdgcn_ballot_w64((int)asm_lds_ld(watch) < thr); } while ((left & ~busy) == 0);
                if ((busy >> pick) & 1) {
                    pick = 63 - __builtin_clzll(left & ~busy);
                    tile = __builtin_amdgcn_readlane(tab, pick);
                    asm volatile("s_waitcnt vmcnt(40)" ::: "memory");
                    load_w(tile * (64u * 8192u) + h * 128, sl + W_OFF, t < NL);
                }
            }
            left &= ~(1ull << pick);
            const unsigned sw = tile * (64u * 8192u) + h * 128, sa = tile * 65536u + par;
            load_a(sa, sl);
            landed(full_a);
            asm volatile("s_waitcnt vmcnt(40)" ::: "memory");
            watch += d1;
            go();
            watch += d2;
            load_w(sw + 32u * 8192u, sl + W_OFF + 4096, t < NL);
            load_a(sa + 32768, sl + 8192);
            landed(full_a + 512);
            full_a -= 4;
            swp = swl; swl = sw;
        }
        asm volatile("s_waitcnt vmcnt(40)" ::: "memory");
        load_w1(swp, lds0 + W_OFF + j * SLOT); load_w1(swp + 32u * 8192u, lds0 + W_OFF + j * SLOT + 4096);
        load_w1(swl, lds0 + W_OFF + (2 + j) * SLOT); load_w1(swl + 32u * 8192u, lds0 + W_OFF + (2 + j) * SLOT + 4096);
        landed(full_a);
    } else if (wid == 6) {
        if (slot < 32) {
            __builtin_amdgcn_s_setprio(3);
            const unsigned tstart = (unsigned)__builtin_amdgcn_s_memrealtime();
            const int dest = (rank + 1 + di) & 7, slice = slot >> 2, tm = (slot - ((6 - di) >> 1)) & 3;
            GAS unsigned* cnt = (GAS unsigned*)(myws + WS_CNT + bid * 128);
            const auto rx = mkrsrc(p.x, 4096 * KC * 2);
            const unsigned vx = (unsigned)(dest * 512 + tm * 128 + ((lane >> 2) & 7)) * 1024u + slice * 128 + (lane >> 5) * 64;
            const unsigned vx0 = vx + ((lane & 3) ^ aswz((lane >> 4) & 1)) * 16, vx1 = vx + ((lane & 3) ^ aswz(2 + ((lane >> 4) & 1))) * 16;
            if (slot >= LATE) {
                __builtin_amdgcn_s_barrier();
                while (__builtin_amdgcn_readfirstlane(asm_lds_ld(sig_at(SIG_FULL, 1))) != 0) __builtin_amdgcn_s_sleep(2);
            }
            u32x4 v[16];
#pragma unroll
            for (int i = 0; i < 16; ++i) v[i] = __builtin_amdgcn_raw_buffer_load_b128(rx, (i & 1) ? vx1 : vx0, i * 8192, 0);
            const unsigned c0 = __hip_atomic_load(cnt, __ATOMIC_RELAXED, __HIP_MEMORY_SCOPE_AGENT);
            if (slot < LATE) __builtin_amdgcn_s_barrier();
            char* const dws = p.peer[di];
            __builtin_amdgcn_s_waitcnt(0x0F70);
            SB();
            const unsigned call = __builtin_amdgcn_readfirstlane(c0) + 1;
            char* const dst = dws + WS_STG + (call & 1) * STG_PAR + (rank * 8 + slice) * 65536 + (lane >> 5) * 32768 + tm * 8192 + (lane & 31) * 16 + 4096;
            asm volatile("" :: "v"(dst));
            for (int left; (left = departs(slot) - (int)((unsigned)__builtin_amdgcn_s_memrealtime() - tstart)) > 0;) {
                if (left > 80) __builtin_amdgcn_s_sleep(16); else __builtin_amdgcn_s_sleep(1);
            }
            static_for<16>([&](auto ic) {
                constexpr int i = decltype(ic)::value;
                char* const d = dst; const u32x4 vi = v[i];
                asm volatile("global_store_dwordx4 %0, %1, off offset:%2 sc0 sc1\n\ts_nop 1" :: "v"(d), "v"(vi), "i"(i * 512 - 4096) : "memory");
            });
            if (slot < HOLD && lane == 0) lds_st(misc + 2, 0u);
            asm volatile("s_waitcnt vmcnt(0)" ::: "memory");
            if (lane == 0) {
                __hip_atomic_store((GAS unsigned*)(dws + WS_FLAG + ((tm * 8 + slice) * 8 + rank) * 128), call, __ATOMIC_RELAXED, __HIP_MEMORY_SCOPE_SYSTEM);
                __hip_atomic_store(cnt, call, __ATOMIC_RELAXED, __HIP_MEMORY_SCOPE_AGENT);
            }
        } else {
            __builtin_amdgcn_s_barrier();
        }
    } else if (wid == 7) {
        __builtin_amdgcn_s_setprio(3);
        const unsigned long long t0 = __builtin_amdgcn_s_memrealtime();
        GAS unsigned* cnt = (GAS unsigned*)(myws + WS_CNT + bid * 128 + 64);
        const auto rc = mkrsrc(myws, WS_STG);
        const unsigned c0 = __hip_atomic_load(cnt, __ATOMIC_RELAXED, __HIP_MEMORY_SCOPE_AGENT);
        const int st = 63 - lane, rt = tile_table(lane, rank, mt, h);
        const unsigned foff = st < NT ? WS_FLAG + ((mt * 8 + (rt & 7)) * 8 + (rt >> 3)) * 128 : WS_TMO;
        const int fslot = (rt & 7) * 4 + ((mt + ((((rt >> 3) - rank - 1) & 7) >> 1)) & 3);
        const unsigned due = (unsigned)t0 + departs(fslot) + (fslot < HOLD ? GATE0 : GATE);
        START_BARRIER();
        if (lane == 0) __builtin_amdgcn_raw_ptr_buffer_load_lds(rc, (LAS void*)misc, 4, (unsigned)(WS_CNT + bid * 128 + 64), 0, 0, 16);
        asm volatile("s_waitcnt vmcnt(0)" ::: "memory");
        const unsigned call = __builtin_amdgcn_readfirstlane(c0) + 1;
        if (lane == 0) __hip_atomic_store(cnt, call, __ATOMIC_RELAXED, __HIP_MEMORY_SCOPE_AGENT);
        bool bad = false;
        for (unsigned n = 1;; ++n) {
            const unsigned v = asm_lds_ld(lds0 + FLG_OFF + lane * 4);
            const bool here = v >= call || st < 4 || st >= NT;
            const unsigned long long seen = __builtin_amdgcn_ballot_w64(here), now = __builtin_amdgcn_s_memrealtime();
            bad = __builtin_amdgcn_readfirstlane(v) == call || now - t0 > DEADLINE;
            if (bad || seen == ~0ull) break;
            const int have = min(__builtin_popcountll(seen & entries(0, 0)), __builtin_popcountll(seen & entries(0, 1))) - 3;
            if (have < 0 || __builtin_amdgcn_readfirstlane(asm_lds_ld(sig_at(SIG_FULL, have))) == 0) {
                if ((lane == 0 && (n & 15) == 0) || (!here && (int)((unsigned)now - due) >= 0))
                    __builtin_amdgcn_raw_ptr_buffer_load_lds(rc, (LAS void*)(lds + FLG_OFF), 4, foff, 0, 0, 17);
                asm volatile("s_waitcnt vmcnt(0)" ::: "memory");
            }
            __builtin_amdgcn_s_sleep(4);
        }
        if (bad) {
            if (lane == 0) { __hip_atomic_store((GAS unsigned*)(myws + WS_TMO), call, __ATOMIC_RELAXED, __HIP_MEMORY_SCOPE_AGENT); lds_st(misc + 1, 0u); }
            lds_st((LAS unsigned*)(lds + FLG_OFF) + lane, call);
        }
    } else {
        __builtin_amdgcn_s_barrier();
        const char* const wq = p.w + (size_t)(mt * 16 + (wid - 8) * 4) * 8192 + nt * 256;
        unsigned junk = 0;
#pragma unroll 1
        for (int u = U0 + h; u < 56; u += 2) {
            const int slice = u / 7, src = (rank + 1 + u - slice * 7) & 7;
#define PF4(r) "s_load_dword %0, %1, " #r "*8192\n\ts_load_dword %0, %1, " #r "*8192+64\n\ts_load_dword %0, %1, " #r "*8192+128\n\ts_load_dword %0, %1, " #r "*8192+192\n\t"
            asm volatile(PF4(0) PF4(1) PF4(2) PF4(3) "s_nop 0" : "+s"(junk) : "s"(wq + (size_t)(src * 8 + slice) * (64 * 8192)) : "memory");
        }
        asm volatile("s_waitcnt lgkmcnt(0)" : "+s"(junk) :: "memory");
    }
}
}

extern "C" void kernel_launch(int rank, int nranks, ncclComm_t comm,
                              hipStream_t stream,
                              void* const* d_in, const int* in_sizes, int n_in,
                              void* d_out, int out_size,
                              const int* in_dtypes, int out_dtype,
                              void* const* d_out_all,
                              void* const* d_ws_all, size_t ws_bytes) {
    using namespace a2a;
    if (nranks != NR || (unsigned)rank >= (unsigned)NR || n_in < 2 || ws_bytes < WS_NEED ||
        in_sizes[0] != 4096 * KC || in_sizes[1] != KTOT * NTOT || out_size != MLOC * NTOT) {
        fprintf(stderr, "a2a_gemm: unexpected problem shape\n");
        return;
    }
    Params p;
    p.x = (const char*)d_in[0]; p.w = (const char*)d_in[1]; p.out = (char*)d_out;
    p.ws = (char*)d_ws_all[rank];
    for (int i = 0; i < NR - 1; ++i) p.peer[i] = (char*)d_ws_all[(rank + 1 + i) % NR];
    p.rank = rank; p.pad = 0;
    hipLaunchKernelGGL(a2a_gemm, dim3(NWG), dim3(NTHREADS), 0, stream, p);
    HIP_CHECK(hipGetLastError());
}
```

```cpp
#include <hip/hip_runtime.h>
#include <rccl/rccl.h>
#include <cstdio>
#include <cstdint>
#include <type_traits>
#include <utility>

#define HIP_CHECK(expr) do { hipError_t _e = (expr); if (_e != hipSuccess) { \
    fprintf(stderr, "HIP error %s at %s:%d\n", hipGetErrorString(_e), __FILE__, __LINE__); \
    return; } } while (0)

namespace a2a {
#define LAS __attribute__((address_space(3)))
#define GAS __attribute__((address_space(1)))
typedef unsigned u32x4 __attribute__((ext_vector_type(4)));
typedef float f32x4 __attribute__((ext_vector_type(4)));
typedef __bf16 bf16x8 __attribute__((ext_vector_type(8)));
typedef __bf16 bf16x4 __attribute__((ext_vector_type(4)));
typedef __bf16 bf16x2 __attribute__((ext_vector_type(2)));
typedef float f32x2 __attribute__((ext_vector_type(2)));

constexpr int T0 = 80;
constexpr int TAU = 32;
constexpr int HOLD = 9;
constexpr int LATE = 12;
constexpr int GATE0 = 170, GATE = 220;
constexpr int U0 = 10;

constexpr int NR = 8;
constexpr int MLOC = 512, KTOT = 4096, NTOT = 4096, KC = 512;
constexpr int NWG = 256, NTHREADS = 12 * 64;
constexpr int NT = 32, NL = NT - 4;
constexpr int SLOT = 16384, W_OFF = 4 * SLOT;
constexpr int SIG_OFF = 8 * SLOT;
constexpr int SIG_FULL = 0;
constexpr int SIG_FREE = 4;
constexpr int NSIG = 12;
constexpr int MISC_OFF = SIG_OFF + NSIG * 512;
constexpr int FLG_OFF = MISC_OFF + 64;
constexpr int LDS_BYTES = FLG_OFF + 256;
static_assert(LDS_BYTES <= 163840 && LDS_BYTES > 81920, "one workgroup per CU");
constexpr size_t WS_CNT = 0;
constexpr size_t WS_TMO = 32768;
constexpr size_t WS_FLAG = 65536;
constexpr size_t WS_STG = 1u << 20;
constexpr size_t STG_PAR = 4u << 20;
constexpr size_t WS_XBUF = WS_STG + 2 * STG_PAR;
constexpr size_t XBUF_BYTES = 5u << 20;
constexpr size_t WS_NEED = WS_XBUF + XBUF_BYTES;
constexpr unsigned long long DEADLINE = 50000000ull;

struct Params {
    const char* x; const char* w; char* out;
    char* ws;
    int rank; int pad;
    char* peer[NR - 1];
};

__device__ __forceinline__ __amdgpu_buffer_rsrc_t mkrsrc(const void* ptr, unsigned bytes) {
    return __builtin_amdgcn_make_buffer_rsrc((void*)ptr, 0, bytes, 0x00020000);
}
__device__ __forceinline__ unsigned lds_ld(LAS unsigned* a) { return __hip_atomic_load(a, __ATOMIC_RELAXED, __HIP_MEMORY_SCOPE_WORKGROUP); }
__device__ __forceinline__ void lds_st(LAS unsigned* a, unsigned v) { __hip_atomic_store(a, v, __ATOMIC_RELAXED, __HIP_MEMORY_SCOPE_WORKGROUP); }
__device__ __forceinline__ int aswz(int i) { return ((i << 1) ^ ((i >> 1) * 3)) & 3; }
__device__ __forceinline__ unsigned pack2(float a, float b) { f32x2 v = {a, b}; return __builtin_bit_cast(unsigned, __builtin_convertvector(v, bf16x2)); }
__device__ __forceinline__ unsigned asm_lds_ld(unsigned addr) {
    unsigned v;
    asm volatile("ds_read_b32 %0, %1\n\ts_waitcnt lgkmcnt(0)" : "=&v"(v) : "v"(addr) : "memory");
    return v;
}
template <int N, class F, int... I> __device__ __forceinline__ void static_for_impl(F f, std::integer_sequence<int, I...>) { (f(std::integral_constant<int, I>{}), ...); }
template <int N, class F> __device__ __forceinline__ void static_for(F f) { static_for_impl<N>(f, std::make_integer_sequence<int, N>{}); }
__device__ __forceinline__ int tile_table(int lane, int rank, int mt, int h) {
    const int j = lane >> 3, pi = 2 * ((((lane >> 1) & 3) - mt) & 3) + (lane & 1);
    const bool mine = pi < 7 && ((j + pi) & 1) == h;
    const int n = __builtin_popcountll(__builtin_amdgcn_ballot_w64(mine) & ((1ull << lane) - 1));
    return __builtin_amdgcn_ds_permute((mine ? 59 - n : 0) * 4, (((rank + 1 + pi) & 7) << 3) | j);
}
constexpr unsigned long long entries(int from, int parity) {
    unsigned long long m = 0;
    for (int t = from; t < NT; ++t) if ((t & 1) == parity) m |= 1ull << (63 - t);
    return m;
}
__device__ __forceinline__ unsigned top4(unsigned long long m) { unsigned hi = (unsigned)(m >> 32); asm("" : "+s"(hi)); return hi >> 28; }
__device__ __forceinline__ int departs(int slot) { return T0 + slot * TAU; }
#define START_BARRIER() do { asm volatile("s_waitcnt lgkmcnt(0)" ::: "memory"); __builtin_amdgcn_s_barrier(); asm volatile("" ::: "memory"); } while (0)
#define SB() __builtin_amdgcn_sched_barrier(0)

__global__ __launch_bounds__(NTHREADS, 3) void a2a_gemm(const Params p) {
    __shared__ __attribute__((aligned(16))) char smem[LDS_BYTES];
    LAS char* lds = (LAS char*)smem;
    const unsigned lds0 = (unsigned)(size_t)lds, sig0 = lds0 + SIG_OFF;
    LAS unsigned* misc = (LAS unsigned*)(lds + MISC_OFF);
    auto sig_at = [&](int arr, int e) { return sig0 + arr * 512 + (63 - e) * 4; };
    const int bid = blockIdx.x, lane = threadIdx.x & 63;
    const int wid = __builtin_amdgcn_readfirstlane(threadIdx.x >> 6);
    const int rank = p.rank;
    asm volatile("" :: "s"(p.x), "s"(p.w), "s"(p.out), "s"(p.ws), "s"(rank));
    const int h = bid & 1, mt = (bid >> 3) & 3, nt = (bid >> 5) * 4 + ((bid >> 1) & 3);
    const int slot = (bid & 7) == 3 ? 32 : bid >> 3, di = (bid & 7) - ((bid & 7) > 3);
    char* const myws = p.ws;
    const u32x4 rz = {(unsigned)(size_t)p.x, (unsigned)((size_t)p.x >> 32) & 0xffffu, 0u, 0x00020000u};

    if (wid < 4) {
        for (int i = threadIdx.x; i < (NSIG * 512 + 64) / 16; i += 256) *(LAS u32x4*)(lds + SIG_OFF + i * 16) = (u32x4){~0u, ~0u, ~0u, ~0u};
        if (wid == 3) *(LAS unsigned*)(lds + FLG_OFF + lane * 4) = 0u;
        START_BARRIER();
        __builtin_amdgcn_s_setprio(3);
        const int wm = wid >> 1, wn = wid & 1, q = (lane >> 2) & 3, g = lane >> 4, r16 = lane & 15, f = (q >> 1) | ((g & 1) << 1);
        const unsigned a0 = lds0 + (wm * 64 + r16) * 64 + ((g ^ aswz(r16 >> 2)) * 16);
        const unsigned w0 = lds0 + W_OFF + (8 * g + q) * 128 + (((wn * 2) ^ f) * 32) + (lane & 3) * 8, w1 = w0 ^ 32;
        unsigned fbase = sig_at(SIG_FULL, 2);
        unsigned free_a = sig_at(SIG_FREE + wid, 0);
        char* outp = p.out;
        asm volatile("" : "+s"(outp));
        auto wait_zero = [&](unsigned a) {
            while (__builtin_amdgcn_ballot_w64(*(volatile LAS unsigned*)(size_t)a != 0) != 0) __builtin_amdgcn_s_sleep(1);
            asm volatile("" ::: "memory");
        };
        bf16x8 af[2][4], wf[2][4];
        bf16x4 wl;
        f32x4 acc[4][4];
        unsigned fv;
#pragma unroll
        for (int i = 0; i < 4; ++i) { af[1][i] = (bf16x8)(__bf16)0.f; wf[1][i] = (bf16x8)(__bf16)0.f; }
#pragma unroll
        for (int i = 0; i < 16; ++i) acc[i >> 2][i & 3] = (f32x4){0.f, 0.f, 0.f, 0.f};
#define FULL_AT(U) (fbase + ((U) & 3) * 512 + (2 - ((U) >> 2)) * 4)
#define FV() do { if constexpr (more) { fv = *(volatile LAS unsigned*)(size_t)FULL_AT(U + 1); SB(); } } while (0)
#define LGKM0() do { __builtin_amdgcn_s_waitcnt(0xC07F); SB(); } while (0)
#define HAVE() do { if constexpr (more) { miss = __builtin_amdgcn_ballot_w64(fv != 0); asm volatile("" : "+s"(miss)); SB(); } } while (0)
#define M0SET() do { if constexpr (sig) { \
            if constexpr (H) asm volatile("s_mov_b32 m0, %0\n\ts_sub_u32 %0, %0, 2052" : "+s"(free_a) :: "memory", "scc"); \
            else asm volatile("s_mov_b32 m0, %0\n\ts_add_u32 %0, %0, 2048" : "+s"(free_a) :: "memory", "scc"); \
            SB(); } } while (0)
#define FREE() do { if constexpr (sig) { const u32x4 rz_ = rz; asm volatile("buffer_load_dword off, %0, 0 lds" :: "s"(rz_) : "memory"); SB(); } } while (0)
#define RDW(cb, t2) do { const bf16x4 w_ = __builtin_amdgcn_ds_read_tr16_b64_v4bf16((LAS bf16x4*)(size_t)(((cb & 1) ? w1 : w0) + S * SLOT + H * 4096 + ((cb >> 1) + IMG) * 8192 + t2 * 512)); \
            if (t2 == 0) wl = w_; else wf[H][cb] = __builtin_shufflevector(wl, w_, 0, 1, 2, 3, 4, 5, 6, 7); SB(); } while (0)
#define RDA(rb) do { af[H][rb] = *(LAS bf16x8*)(size_t)(a0 + S * SLOT + H * 8192 + rb * 1024); SB(); } while (0)
#define MM(rb, cb) do { f32x4& c_ = acc[rb][C0 + cb]; const bf16x8 w_ = wf[H ^ 1][cb], a_ = af[H ^ 1][rb]; \
            asm volatile("v_mfma_f32_16x16x32_bf16 %0, %1, %2, %0" : "+v"(c_) : "v"(w_), "v"(a_)); SB(); } while (0)
#define MMEND() do { asm volatile("s_nop 7\n\ts_nop 1" ::: "memory"); SB(); } while (0)
        auto half = [&](auto uc, auto kc, auto more) {
            constexpr int U = decltype(uc)::value, S = U >> 1, H = U & 1, K = decltype(kc)::value, IMG = K == 1 || K == 2, C0 = K == 2 ? 2 : 0;
            constexpr bool sig = K == 0;
            unsigned long long miss = 0;
            if constexpr (K == 0) {
                MM(0, 0); RDW(0, 0); MM(0, 1); RDW(0, 1); RDA(0); MM(0, 2); RDW(1, 0); MM(0, 3); RDW(1, 1);
                MM(1, 0); RDW(2, 0); RDW(2, 1); MM(1, 1); RDW(3, 0); MM(1, 2); RDW(3, 1); MM(1, 3); RDA(1);
                MM(2, 0); RDA(2); MM(2, 1); RDA(3); FV(); MM(2, 2); M0SET(); MM(2, 3);
                MM(3, 0); MM(3, 1); LGKM0(); HAVE(); MM(3, 2); FREE(); MM(3, 3);
            } else if constexpr (K == 1) {
                MM(0, 0); RDW(0, 0); MM(0, 1); RDW(0, 1); MM(0, 2); RDA(0); MM(0, 3); RDW(1, 0);
                MM(1, 0); RDW(1, 1); MM(1, 1); RDA(1); MM(1, 2); RDA(2); MM(1, 3); RDA(3);
                MM(2, 0); MM(2, 1); FV(); MM(2, 2); MM(2, 3);
                MM(3, 0); MM(3, 1); LGKM0(); HAVE(); MM(3, 2); MM(3, 3);
            } else if constexpr (K == 3) {
                RDW(0, 0); RDW(0, 1); RDA(0); RDW(1, 0); RDW(1, 1); RDA(1); RDA(2); RDA(3);
            } else {
                FV(); MM(0, 0); RDW(0, 0); RDW(0, 1); MM(0, 1); RDA(0); MM(1, 0); RDW(1, 0); RDW(1, 1); MM(1, 1); RDA(1);
                MM(2, 0); RDA(2); MM(2, 1); RDA(3); MM(3, 0); HAVE(); MM(3, 1);
            }
            if constexpr (more) {
                if (__builtin_expect(miss != 0, 0)) wait_zero(FULL_AT(U + 1));
                if constexpr (U == 7) { fbase -= 8; asm volatile("" : "+v"(fbase)); }
            }
        };
        using F_ = std::integral_constant<bool, false>;
        using T_ = std::integral_constant<bool, true>;
        __builtin_amdgcn_s_waitcnt(0x0F70);
        wait_zero(FULL_AT(0));
#pragma unroll 1
        for (int i = 0; i < NL / 4; ++i) {
            static_for<8>([&](auto uc) { half(uc, std::integral_constant<int, 0>{}, T_{}); });
        }
        constexpr int NX = 5;
        const auto rxb = mkrsrc(myws + WS_XBUF, XBUF_BYTES);
        u32x4 xp[NX];
        unsigned call;
        int tid = threadIdx.x;
        auto fetch = [&]() {
            asm volatile("" ::: "memory");
#pragma unroll
            for (int k = 0; k < NX; ++k) xp[k] = __builtin_amdgcn_raw_buffer_load_b128(rxb, (bid * 4 + wid) * (NX * 1024) + (tid & 63) * 16 + k * 1024, 0, 16);
        };
        static_for<8>([&](auto uc) { half(uc, std::integral_constant<int, decltype(uc)::value == 0 ? 1 : 2>{}, std::integral_constant<bool, (decltype(uc)::value < 7)>{}); });
        asm volatile("" : "+v"(tid));
        call = __builtin_amdgcn_readfirstlane(lds_ld(misc)) + 1;
        SB();
#pragma unroll
        for (int rb = 0; rb < 4; ++rb) { constexpr int H = 0, C0 = 2; MM(rb, 0); MM(rb, 1); }
        MMEND();
        {
            unsigned pk[16];
#pragma unroll
            for (int k = 0; k < 16; ++k) pk[k] = pack2(acc[k >> 2][2 + ((k >> 1) & 1)][(k & 1) * 2], acc[k >> 2][2 + ((k >> 1) & 1)][(k & 1) * 2 + 1]);
#pragma unroll
            for (int k = 0; k < NX; ++k)
                __builtin_amdgcn_raw_buffer_store_b128((u32x4){pk[3 * k], pk[3 * k + 1], pk[3 * k + 2], (call & 0xffffu) | (k == 3 ? pk[15] << 16 : k == 4 ? pk[15] & 0xffff0000u : 0u)},
                                                       rxb, ((bid ^ 1) * 4 + wid) * (NX * 1024) + (tid & 63) * 16 + k * 1024, 0, 16);
            SB();
        }
        wait_zero(sig_at(SIG_FULL + (tid & 1) * 2, NT / 2));
        static_for<8>([&](auto uc) {
            half(uc, std::integral_constant<int, decltype(uc)::value == 0 ? 3 : 4>{}, F_{});
            if constexpr (decltype(uc)::value == 5) { fetch(); SB(); }
        });
        unsigned vo = (unsigned)(mt * 128 + wm * 64 + (tid & 15)) * 8192u + nt * 256 + h * 128 + wn * 64 + ((tid >> 4) & 1) * 32 + ((tid >> 5) & 1) * 16;
        bool bad = __builtin_amdgcn_readfirstlane(lds_ld(misc + 1)) == 0;
        asm volatile("" : "+v"(vo));
        SB();
#pragma unroll
        for (int rb = 0; rb < 4; ++rb) { constexpr int H = 0, C0 = 0; MM(rb, 0); MM(rb, 1); }
        MMEND();
        auto stale = [&]() { return __builtin_amdgcn_ballot_w64((((xp[0][3] ^ call) | (xp[1][3] ^ call) | (xp[2][3] ^ call) | (xp[3][3] ^ call) | (xp[4][3] ^ call)) & 0xffffu) != 0) != 0; };
        if (stale()) {
            GAS unsigned* tmo = (GAS unsigned*)(myws + WS_TMO);
            const unsigned long long t0 = __builtin_amdgcn_s_memrealtime();
            for (unsigned n = 1;; ++n) {
                fetch();
                if (!stale()) break;
                if ((n & 15) == 0 && ((unsigned)__builtin_amdgcn_readfirstlane(__hip_atomic_load(tmo, __ATOMIC_RELAXED, __HIP_MEMORY_SCOPE_AGENT)) == call
                                      || __builtin_amdgcn_s_memrealtime() - t0 > DEADLINE)) {
                    __hip_atomic_store(tmo, call, __ATOMIC_RELAXED, __HIP_MEMORY_SCOPE_AGENT);
                    bad = true; break;
                }
                __builtin_amdgcn_s_sleep(1);
            }
        }
        const auto ro = mkrsrc(outp, MLOC * NTOT * 2);
#pragma unroll
        for (int rb = 0; rb < 4; ++rb) {
            unsigned x_[4];
#pragma unroll
            for (int k = 0; k < 4; ++k) x_[k] = rb * 4 + k < 15 ? xp[(rb * 4 + k) / 3][(rb * 4 + k) % 3] : (xp[3][3] >> 16) | (xp[4][3] & 0xffff0000u);
            const f32x4 a = acc[rb][0] + __builtin_bit_cast(f32x4, (u32x4){x_[0] << 16, x_[0] & 0xffff0000u, x_[1] << 16, x_[1] & 0xffff0000u});
            const f32x4 b = acc[rb][1] + __builtin_bit_cast(f32x4, (u32x4){x_[2] << 16, x_[2] & 0xffff0000u, x_[3] << 16, x_[3] & 0xffff0000u});
            const auto s0 = __builtin_amdgcn_permlane16_swap(pack2(a[0], a[1]), pack2(b[0], b[1]), false, false);
            const auto s1 = __builtin_amdgcn_permlane16_swap(pack2(a[2], a[3]), pack2(b[2], b[3]), false, false);
            u32x4 o = {s0[0], s1[0], s0[1], s1[1]};
            if (bad) o = (u32x4){0x7fc07fc0u, 0x7fc07fc0u, 0x7fc07fc0u, 0x7fc07fc0u};
            __builtin_amdgcn_raw_buffer_store_b128(o, ro, vo + rb * 16 * 8192, 0, 0);
        }
    } else if (wid < 6) {
        const int j = wid & 1;
        unsigned full_a = sig_at(SIG_FULL + j * 2, 2);
        int tab;
        const u32x4 rw4 = {(unsigned)(size_t)p.w, (unsigned)((size_t)p.w >> 32) & 0xffffu, KTOT * NTOT * 2, 0x00020000u};
        const unsigned vwb = (unsigned)(lane >> 3) * 8192u + nt * 256;
        const unsigned vw0 = vwb + (((lane & 7) ^ (((lane >> 4) & 1) * 2)) * 16), vw1 = vwb + (((lane & 7) ^ ((((lane >> 4) & 1) | 2) * 2)) * 16) + (65536 - 1024);
        const unsigned vw2 = vw0 + 2 * (65536 - 1024), vw3 = vw1 + 2 * (65536 - 1024);
#define LV4(aux) "buffer_load_dwordx4 %1, %5, %0 offen" aux " lds\n\tbuffer_load_dwordx4 %2, %5, %0 offen offset:1024" aux " lds\n\t" \
                 "buffer_load_dwordx4 %3, %5, %0 offen offset:2048" aux " lds\n\tbuffer_load_dwordx4 %4, %5, %0 offen offset:3072" aux " lds\n\t"
        auto load_w1 = [&](unsigned so, unsigned m) {
            asm volatile("s_mov_b32 m0, %6\n\ts_nop 0\n\t" LV4("") :: "s"(so), "v"(vw0), "v"(vw1), "v"(vw2), "v"(vw3), "s"(rw4), "s"(m) : "memory");
        };
        auto load_w = [&](unsigned so, unsigned m, bool both) { load_w1(so ^ 128, m + 8192); if (both) load_w1(so, m); };
        auto landed = [&](unsigned a) {
            asm volatile("s_mov_b32 m0, %0\n\ts_mov_b64 exec, 1\n\tbuffer_load_dword off, %1, 0 lds\n\ts_mov_b64 exec, -1" :: "s"(a), "s"(rz) : "memory");
        };
        {
            const u32x4 rx4 = {rz[0], rz[1], 4096 * KC * 2, 0x00020000u};
            const unsigned vx0 = (unsigned)(rank * 512 + mt * 128 + (lane >> 2)) * 1024u + ((lane & 3) ^ aswz(lane >> 4)) * 16;
            const unsigned vx1 = vx0 + (16384 - 1024), vx2 = vx0 + 2 * (16384 - 1024), vx3 = vx0 + 3 * (16384 - 1024);
            const int hold = slot < HOLD ? max(slot - 1, 0) / 2 : 4;
            auto local = [&](int t, int sl_) {
                load_w((rank * 8 + sl_) * (64u * 8192u) + j * (32u * 8192u) + h * 128, lds0 + W_OFF + t * SLOT + j * 4096, true);
                unsigned so = sl_ * 128 + j * 64;
                asm volatile("s_mov_b32 m0, %6\n\ts_nop 0\n\t" LV4("") "s_add_u32 m0, m0, 0x1000\n\ts_add_u32 %0, %0, 0x10000\n\t" LV4("")
                             : "+s"(so) : "v"(vx0), "v"(vx1), "v"(vx2), "v"(vx3), "s"(rx4), "s"(lds0 + t * SLOT + j * 8192) : "memory", "scc");
            };
            auto burst_out = [&]() { while (__builtin_amdgcn_readfirstlane(asm_lds_ld(lds0 + MISC_OFF + 8)) != 0) __builtin_amdgcn_s_sleep(1); };
            const bool first = slot >= LATE;
            if (!first) {
                START_BARRIER();
                if (hold == 0) burst_out();
            }
            local(0, h);
            tab = tile_table(lane, rank, mt, h);
            if (first) START_BARRIER();
            landed(sig_at(SIG_FULL + j, 0));
#pragma unroll 1
            for (int t = 1; t < 4; ++t) {
                if (t == hold) burst_out();
                asm volatile("s_waitcnt vmcnt(40)" ::: "memory");
                local(t, 2 * (1 + (t - 1 + mt + (bid >> 5)) % 3) + h);
                landed(sig_at(SIG_FULL + (t & 1) * 2 + j, t >> 1));
            }
        }
        const size_t sb = (size_t)(myws + WS_STG) + mt * 8192;
        const u32x4 rs4 = {(unsigned)sb, ((unsigned)(sb >> 32) & 0xffffu) | (16u << 16), (unsigned)(2 * STG_PAR) - mt * 8192, 0x00800000u};
        unsigned c0;
        while ((c0 = __builtin_amdgcn_readfirstlane(asm_lds_ld(lds0 + MISC_OFF))) == ~0u) __builtin_amdgcn_s_sleep(1);
        asm volatile("" : "+s"(c0));
        const unsigned call = c0 + 1, par = (call & 1) * (unsigned)STG_PAR;
        unsigned watch = lane < 60 ? lds0 + FLG_OFF + lane * 4 : sig_at(SIG_FREE + (lane & 3), j);
        const int thr = lane < 60 ? (int)call : 0;
        const unsigned d1 = lane < 60 ? 0 : 2048, d2 = lane < 60 ? 0 : -2056;
        auto go = [&]() {
            unsigned long long busy;
            do busy = __builtin_amdgcn_ballot_w64((int)asm_lds_ld(watch) < thr); while (top4(busy) != 0);
            return busy;
        };
        unsigned long long left = j ? entries(4, 1) : entries(4, 0);
#define LA4 "buffer_load_dwordx4 off, %1, %0 sc0 sc1 lds\n\tbuffer_load_dwordx4 off, %1, %0 offset:1024 sc0 sc1 lds\n\t" \
            "buffer_load_dwordx4 off, %1, %0 offset:2048 sc0 sc1 lds\n\tbuffer_load_dwordx4 off, %1, %0 offset:3072 sc0 sc1 lds\n\t"
        auto load_a = [&](unsigned so, unsigned m) {
            asm volatile("s_mov_b32 m0, %2\n\ts_nop 0\n\t" LA4 "s_add_u32 m0, m0, 0x1000\n\ts_add_u32 %0, %0, 0x1000\n\t" LA4
                         : "+s"(so) : "s"(rs4), "s"(m) : "memory", "scc");
        };
        unsigned swp = 0, swl = 0;
#pragma unroll 1
        for (int t = 4 + j; t < NT; t += 2) {
            const unsigned sl = lds0 + (t & 3) * SLOT;
            asm volatile("s_waitcnt vmcnt(40)" ::: "memory");
            unsigned long long busy = go();
            int pick = 63 - __builtin_clzll((left & ~busy) != 0 ? left & ~busy : left);
            unsigned tile = __builtin_amdgcn_readlane(tab, pick);
            load_w(tile * (64u * 8192u) + h * 128, sl + W_OFF, t < NL);
            if (__builtin_expect((left & ~busy) == 0, 0)) {
                do { __builtin_amdgcn_s_sleep(1); busy = __builtin_amdgcn_ballot_w64((int)asm_lds_ld(watch) < thr); } while ((left & ~busy) == 0);
                if ((busy >> pick) & 1) {
                    pick = 63 - __builtin_clzll(left & ~busy);
                    tile = __builtin_amdgcn_readlane(tab, pick);
                    asm volatile("s_waitcnt vmcnt(40)" ::: "memory");
                    load_w(tile * (64u * 8192u) + h * 128, sl + W_OFF, t < NL);
                }
            }
            left &= ~(1ull << pick);
            const unsigned sw = tile * (64u * 8192u) + h * 128, sa = tile * 65536u + par;
            load_a(sa, sl);
            landed(full_a);
            asm volatile("s_waitcnt vmcnt(40)" ::: "memory");
            watch += d1;
            go();
            watch += d2;
            load_w(sw + 32u * 8192u, sl + W_OFF + 4096, t < NL);
            load_a(sa + 32768, sl + 8192);
            landed(full_a + 512);
            full_a -= 4;
            swp = swl; swl = sw;
        }
        asm volatile("s_waitcnt vmcnt(40)" ::: "memory");
        load_w1(swp, lds0 + W_OFF + j * SLOT); load_w1(swp + 32u * 8192u, lds0 + W_OFF + j * SLOT + 4096);
        load_w1(swl, lds0 + W_OFF + (2 + j) * SLOT); load_w1(swl + 32u * 8192u, lds0 + W_OFF + (2 + j) * SLOT + 4096);
        landed(full_a);
    } else if (wid == 6) {
        if (slot < 32) {
            __builtin_amdgcn_s_setprio(3);
            const unsigned tstart = (unsigned)__builtin_amdgcn_s_memrealtime();
            const int dest = (rank + 1 + di) & 7, slice = slot >> 2, tm = (slot - ((6 - di) >> 1)) & 3;
            GAS unsigned* cnt = (GAS unsigned*)(myws + WS_CNT + bid * 128);
            const auto rx = mkrsrc(p.x, 4096 * KC * 2);
            const unsigned vx = (unsigned)(dest * 512 + tm * 128 + ((lane >> 2) & 7)) * 1024u + slice * 128 + (lane >> 5) * 64;
            const unsigned vx0 = vx + ((lane & 3) ^ aswz((lane >> 4) & 1)) * 16, vx1 = vx + ((lane & 3) ^ aswz(2 + ((lane >> 4) & 1))) * 16;
            if (slot >= LATE) {
                __builtin_amdgcn_s_barrier();
                while (__builtin_amdgcn_readfirstlane(asm_lds_ld(sig_at(SIG_FULL, 1))) != 0) __builtin_amdgcn_s_sleep(2);
            }
            u32x4 v[16];
#pragma unroll
            for (int i = 0; i < 16; ++i) v[i] = __builtin_amdgcn_raw_buffer_load_b128(rx, (i & 1) ? vx1 : vx0, i * 8192, 0);
            const unsigned c0 = __hip_atomic_load(cnt, __ATOMIC_RELAXED, __HIP_MEMORY_SCOPE_AGENT);
            if (slot < LATE) __builtin_amdgcn_s_barrier();
            char* const dws = p.peer[di];
            __builtin_amdgcn_s_waitcnt(0x0F70);
            SB();
            const unsigned call = __builtin_amdgcn_readfirstlane(c0) + 1;
            char* const dst = dws + WS_STG + (call & 1) * STG_PAR + (rank * 8 + slice) * 65536 + (lane >> 5) * 32768 + tm * 8192 + (lane & 31) * 16 + 4096;
            asm volatile("" :: "v"(dst));
            for (int left; (left = departs(slot) - (int)((unsigned)__builtin_amdgcn_s_memrealtime() - tstart)) > 0;) {
                if (left > 80) __builtin_amdgcn_s_sleep(16); else __builtin_amdgcn_s_sleep(1);
            }
            static_for<16>([&](auto ic) {
                constexpr int i = decltype(ic)::value;
                char* const d = dst; const u32x4 vi = v[i];
                asm volatile("global_store_dwordx4 %0, %1, off offset:%2 sc0 sc1\n\ts_nop 1" :: "v"(d), "v"(vi), "i"(i * 512 - 4096) : "memory");
            });
            if (slot < HOLD && lane == 0) lds_st(misc + 2, 0u);
            asm volatile("s_waitcnt vmcnt(0)" ::: "memory");
            if (lane == 0) {
                __hip_atomic_store((GAS unsigned*)(dws + WS_FLAG + ((tm * 8 + slice) * 8 + rank) * 128), call, __ATOMIC_RELAXED, __HIP_MEMORY_SCOPE_SYSTEM);
                __hip_atomic_store(cnt, call, __ATOMIC_RELAXED, __HIP_MEMORY_SCOPE_AGENT);
            }
        } else {
            __builtin_amdgcn_s_barrier();
        }
    } else if (wid == 7) {
        __builtin_amdgcn_s_setprio(3);
        const unsigned long long t0 = __builtin_amdgcn_s_memrealtime();
        GAS unsigned* cnt = (GAS unsigned*)(myws + WS_CNT + bid * 128 + 64);
        const auto rc = mkrsrc(myws, WS_STG);
        const unsigned c0 = __hip_atomic_load(cnt, __ATOMIC_RELAXED, __HIP_MEMORY_SCOPE_AGENT);
        const int st = 63 - lane, rt = tile_table(lane, rank, mt, h);
        const unsigned foff = st < NT ? WS_FLAG + ((mt * 8 + (rt & 7)) * 8 + (rt >> 3)) * 128 : WS_TMO;
        const int fslot = (rt & 7) * 4 + ((mt + ((((rt >> 3) - rank - 1) & 7) >> 1)) & 3);
        const unsigned due = (unsigned)t0 + departs(fslot) + (fslot < HOLD ? GATE0 : GATE);
        START_BARRIER();
        if (lane == 0) __builtin_amdgcn_raw_ptr_buffer_load_lds(rc, (LAS void*)misc, 4, (unsigned)(WS_CNT + bid * 128 + 64), 0, 0, 16);
        asm volatile("s_waitcnt vmcnt(0)" ::: "memory");
        const unsigned call = __builtin_amdgcn_readfirstlane(c0) + 1;
        if (lane == 0) __hip_atomic_store(cnt, call, __ATOMIC_RELAXED, __HIP_MEMORY_SCOPE_AGENT);
        bool bad = false;
        for (unsigned n = 1;; ++n) {
            const unsigned v = asm_lds_ld(lds0 + FLG_OFF + lane * 4);
            const bool here = v >= call || st < 4 || st >= NT;
            const unsigned long long seen = __builtin_amdgcn_ballot_w64(here), now = __builtin_amdgcn_s_memrealtime();
            bad = __builtin_amdgcn_readfirstlane(v) == call || now - t0 > DEADLINE;
            if (bad || seen == ~0ull) break;
            const int have = min(__builtin_popcountll(seen & entries(0, 0)), __builtin_popcountll(seen & entries(0, 1))) - 3;
            if (have < 0 || __builtin_amdgcn_readfirstlane(asm_lds_ld(sig_at(SIG_FULL, have))) == 0) {
                if ((lane == 0 && (n & 15) == 0) || (!here && (int)((unsigned)now - due) >= 0))
                    __builtin_amdgcn_raw_ptr_buffer_load_lds(rc, (LAS void*)(lds + FLG_OFF), 4, foff, 0, 0, 17);
                asm volatile("s_waitcnt vmcnt(0)" ::: "memory");
            }
            __builtin_amdgcn_s_sleep(4);
        }
        if (bad) {
            if (lane == 0) { __hip_atomic_store((GAS unsigned*)(myws + WS_TMO), call, __ATOMIC_RELAXED, __HIP_MEMORY_SCOPE_AGENT); lds_st(misc + 1, 0u); }
            lds_st((LAS unsigned*)(lds + FLG_OFF) + lane, call);
        }
    } else {
        unsigned junk = 0, jv = 0;
        if (wid == 8 && slot >= 24 && slot < 32) {
            const char* const d = p.peer[di];
            const char* const d0 = d + WS_STG + (rank * 8 + slot - 24) * 65536;
            asm volatile("s_load_dword %0, %1, 0\n\ts_load_dword %0, %2, 0\n\ts_load_dword %0, %3, 0"
                         : "+s"(junk) : "s"(d0), "s"(d0 + STG_PAR), "s"(d + WS_FLAG + (slot - 24) * 4096 + ((rank + 1 + di) & 7) * 128) : "memory");
        }
        if (wid == 9 && bid >= NWG - 8 && lane < 50) {
            static_assert(WS_NEED == 14u << 20, "lanes");
            const char* const a = lane < 14 ? myws + 0xfff80 + (lane << 20) : lane < 18 ? p.out + ((lane - 14) << 20) : p.w + ((lane - 18) << 20);
            asm volatile("global_load_dword %0, %1, off sc1" : "+v"(jv) : "v"(a) : "memory");
        }
        __builtin_amdgcn_s_barrier();
        const char* const wq = p.w + (size_t)(mt * 16 + (wid - 8) * 4) * 8192 + nt * 256;
#pragma unroll 1
        for (int u = U0 + h; u < 56; u += 2) {
            const int slice = u / 7, src = (rank + 1 + u - slice * 7) & 7;
#define PF4(r) "s_load_dword %0, %1, " #r "*8192\n\ts_load_dword %0, %1, " #r "*8192+64\n\ts_load_dword %0, %1, " #r "*8192+128\n\ts_load_dword %0, %1, " #r "*8192+192\n\t"
            asm volatile(PF4(0) PF4(1) PF4(2) PF4(3) "s_nop 0" : "+s"(junk) : "s"(wq + (size_t)(src * 8 + slice) * (64 * 8192)) : "memory");
        }
        asm volatile("s_waitcnt vmcnt(0) lgkmcnt(0)" : "+s"(junk), "+v"(jv) :: "memory");
    }
}
}

extern "C" void kernel_launch(int rank, int nranks, ncclComm_t comm,
                              hipStream_t stream,
                              void* const* d_in, const int* in_sizes, int n_in,
                              void* d_out, int out_size,
                              const int* in_dtypes, int out_dtype,
                              void* const* d_out_all,
                              void* const* d_ws_all, size_t ws_bytes) {
    using namespace a2a;
    if (nranks != NR || (unsigned)rank >= (unsigned)NR || n_in < 2 || ws_bytes < WS_NEED ||
        in_sizes[0] != 4096 * KC || in_sizes[1] != KTOT * NTOT || out_size != MLOC * NTOT) {
        fprintf(stderr, "a2a_gemm: unexpected problem shape\n");
        return;
    }
    Params p;
    p.x = (const char*)d_in[0]; p.w = (const char*)d_in[1]; p.out = (char*)d_out;
    p.ws = (char*)d_ws_all[rank];
    for (int i = 0; i < NR - 1; ++i) p.peer[i] = (char*)d_ws_all[(rank + 1 + i) % NR];
    p.rank = rank; p.pad = 0;
    hipLaunchKernelGGL(a2a_gemm, dim3(NWG), dim3(NTHREADS), 0, stream, p);
    HIP_CHECK(hipGetLastError());
}
```
